```python
import numpy as np

kernel_name = "gpu_gemm_ar_m8192_k8192_n8192_bf16_none"


ABSMAX_RTOL = 0.008

COMPUTE_DTYPE = "bfloat16"
EPILOGUE = "none"
IN_QUANT = None
OUT_QUANT = None
IN_DTYPES = {"x": "bfloat16", "w_mat": "bfloat16"}
WORLD_SIZE = 8

def _to_bf16(x):
    u = np.ascontiguousarray(x, dtype=np.float32).view(np.uint32)
    return ((u + 0x7FFF + ((u >> 16) & 1)) & 0xFFFF0000).view(np.float32)

def setup_inputs():

    x = _to_bf16(np.random.randn(8192, 8192).astype(np.float32))
    w_mat = _to_bf16((np.random.randn(8192, 8192) / np.sqrt(8192)).astype(np.float32))
    return {"x": x, "w_mat": w_mat}

def _epilogue(y):
    return y

def reference(x, w_mat):
    y = (x @ w_mat).astype(np.float32)
    return _epilogue(y)
```

```cpp
#include <hip/hip_runtime.h>
#include <rccl/rccl.h>
#include <cstdio>

#define HIP_CHECK(expr) do { hipError_t _e = (expr); if (_e != hipSuccess) { \
    fprintf(stderr, "HIP error %s at %s:%d\n", hipGetErrorString(_e), __FILE__, __LINE__); \
    return; } } while (0)

namespace {

#define LAS __attribute__((address_space(3)))
typedef short bf16x8 __attribute__((ext_vector_type(8)));
typedef short s16x4 __attribute__((ext_vector_type(4)));
typedef float f32x4 __attribute__((ext_vector_type(4)));
typedef unsigned u32x4 __attribute__((ext_vector_type(4)));

constexpr int NR = 8;
constexpr int GM = 8192, GN = 8192;
constexpr int KS = 1024;
constexpr int BR = 4096, BC = 2048;
constexpr int NPX = BR / 256, NPW = BC / 256;
constexpr int NTILE = NPX * NPW;

constexpr size_t CTL_BYTES = 1u << 20;
constexpr size_t XS_OFF = CTL_BYTES;
constexpr size_t XS_SLOT = (size_t)BR * KS * 2;
constexpr size_t WS_OFF = XS_OFF + NR * XS_SLOT;
constexpr size_t WS_ARR = (size_t)KS * GN * 2;
constexpr size_t WS_END = WS_OFF + 2 * WS_ARR;
constexpr size_t OUT_BYTES = (size_t)GM * GN * 2;
constexpr size_t X_BYTES = (size_t)GM * KS * 2, W_BYTES = (size_t)KS * GN * 2;

constexpr int L_TICKET = 0, L_TMO = 1, L_DONE = 8  , L_PANEL = 16  , L_TILE = 512  ;
constexpr int CE_WGS = 8;
constexpr int CE_WAVES = CE_WGS * 8;
constexpr int SLOTS = CE_WGS + NTILE / 8;
constexpr int GRID = 8 * SLOTS;
constexpr unsigned long long DEADLINE_TICKS = 50000000ull;

constexpr int BK = 64, HALF = 128, HTB = HALF * BK * 2, STAGE_BYTES = 8 * HTB;

struct Params {
    const void* x; const void* w;
    void* out[NR]; void* ws[NR];
    int rank;
};

__device__ __forceinline__ int lds_byte(int r, int c) { const int st = (r >> 4) * 2 + (c >> 5), rr = r & 15, cc = c & 31, ob = rr * 64 + cc * 2; return st * 1024 + (ob ^ (((ob >> 9) & 1) << 5)); }
__device__ __forceinline__ void stage_rc(int b, int& R, int& C) { const int st = b / 1024, sb = b % 1024, swz = sb ^ (((sb >> 9) & 1) << 5); R = (st >> 1) * 16 + swz / 64; C = (st & 1) * 32 + (swz % 64) / 2; }
__device__ __forceinline__ unsigned wswz(unsigned row) { return ((row & 3u) << 2) | ((row >> 2) & 3u); }
__device__ __forceinline__ unsigned cvt_pk_bf16(float lo, float hi) { unsigned r; asm volatile("v_cvt_pk_bf16_f32 %0, %1, %2" : "=v"(r) : "v"(lo), "v"(hi)); return r; }

__device__ __forceinline__ unsigned ld_relaxed_sys(const unsigned* p) { return __hip_atomic_load(p, __ATOMIC_RELAXED, __HIP_MEMORY_SCOPE_SYSTEM); }
__device__ __forceinline__ unsigned ld_relaxed_agent(const unsigned* p) { return __hip_atomic_load(p, __ATOMIC_RELAXED, __HIP_MEMORY_SCOPE_AGENT); }

__device__ __forceinline__ bool wave_wait_ge(const unsigned* word, unsigned need, unsigned* tmo, unsigned call) {
    const unsigned long long t0 = __builtin_amdgcn_s_memrealtime();
    unsigned polls = 0;
    for (;;) {
        const unsigned v = ld_relaxed_sys(word);
        if (__all((int)(v - need) >= 0)) return true;
        if ((++polls & 63u) == 0u) {
            if ((unsigned)__builtin_amdgcn_readfirstlane(ld_relaxed_agent(tmo)) == call || __builtin_amdgcn_s_memrealtime() - t0 > DEADLINE_TICKS) {
                __hip_atomic_store(tmo, call, __ATOMIC_RELAXED, __HIP_MEMORY_SCOPE_AGENT);
                return false;
            }
        }
        __builtin_amdgcn_s_sleep(4);
    }
}

__device__ __forceinline__ void tile_of_index(int idx, int& i, int& j) {
    if (idx < 64) { int s = 0; while ((s + 1) * (s + 1) <= idx) ++s; const int rem = idx - s * s; if (rem < s) { i = rem; j = s; } else { i = s; j = rem - s; } }
    else { i = idx >> 3; j = idx & 7; }
}

template <int NI>
__device__ __forceinline__ void copy_rows(__amdgpu_buffer_rsrc_t src, unsigned soff, __amdgpu_buffer_rsrc_t dst, unsigned doff, unsigned step) {
    u32x4 v[NI];
#pragma unroll
    for (int u = 0; u < NI; ++u) v[u] = __builtin_amdgcn_raw_buffer_load_b128(src, soff + u * step, 0, 16  );
#pragma unroll
    for (int u = 0; u < NI; ++u) __builtin_amdgcn_raw_buffer_store_b128(v[u], dst, doff + u * step, 0, 17  );
}

__global__ __launch_bounds__(512, 2) void gemm_ar_kernel(Params P) {
    __shared__ __attribute__((aligned(16))) unsigned char shm[STAGE_BYTES + 64];
    LAS unsigned char* lds = (LAS unsigned char*)shm;
    const int tid = threadIdx.x, wid = __builtin_amdgcn_readfirstlane(tid >> 6), lane = tid & 63;
    const int rank = P.rank;
    const int xcd = blockIdx.x & 7, slot = blockIdx.x >> 3;
    char* myws = (char*)P.ws[rank];
    unsigned* ctl = (unsigned*)myws;
    unsigned* tmo = ctl + L_TMO * 32;
    const int row0 = (rank >> 2) * BR, col0 = (rank & 3) * BC;

    if (tid == 0) { const unsigned t = __hip_atomic_fetch_add(ctl + L_TICKET * 32, 1u, __ATOMIC_RELAXED, __HIP_MEMORY_SCOPE_AGENT); *(LAS unsigned*)(lds + STAGE_BYTES) = t / (unsigned)GRID + 1u; }

    if (slot < CE_WGS) {
        if (xcd == 7) return;
        const int q = (rank + 1 + xcd) & 7;
        const int w = slot * 8 + wid;
        char* qws = (char*)P.ws[q];
        unsigned* qctl = (unsigned*)qws;
        const int qrow0 = (q >> 2) * BR, qcol0 = (q & 3) * BC;
        const __amdgpu_buffer_rsrc_t rx = __builtin_amdgcn_make_buffer_rsrc((void*)P.x, 0, (int)X_BYTES, 0x00020000);
        const __amdgpu_buffer_rsrc_t rw = __builtin_amdgcn_make_buffer_rsrc((void*)P.w, 0, (int)W_BYTES, 0x00020000);
        const __amdgpu_buffer_rsrc_t rqx = __builtin_amdgcn_make_buffer_rsrc(qws + XS_OFF + (size_t)rank * XS_SLOT, 0, (int)XS_SLOT, 0x00020000);
        const __amdgpu_buffer_rsrc_t rqw = __builtin_amdgcn_make_buffer_rsrc(qws + WS_OFF + (size_t)(rank >> 2) * WS_ARR, 0, (int)WS_ARR, 0x00020000);
        const unsigned rlane = (unsigned)(lane >> 5) * 16384u + (unsigned)(lane & 31) * 16u;
        for (int s = 0; s < NPX + NPW; ++s) {
            const bool isw = (s < 2 * NPW) && !(s & 1);
            const int pidx = s < 2 * NPW ? (s >> 1) : s - NPW;
            if (isw) {
                const unsigned rb = (unsigned)w * 16u * 16384u + rlane;
                copy_rows<8>(rw, rb + (unsigned)(qcol0 + 256 * pidx) * 2u, rqw, rb + (unsigned)((rank & 3) * BC + 256 * pidx) * 2u, 32768u);
            } else {
                const unsigned lb = (unsigned)pidx * 524288u + (unsigned)w * 8192u + (unsigned)lane * 16u;
                copy_rows<8>(rx, (unsigned)qrow0 * 2048u + lb, rqx, lb, 1024u);
            }
            asm volatile("s_waitcnt vmcnt(0)" ::: "memory");
            if (lane == 0) __hip_atomic_fetch_add(qctl + (L_PANEL + rank * 32 + (isw ? 16 : 0) + pidx) * 32, 1u, __ATOMIC_RELAXED, __HIP_MEMORY_SCOPE_SYSTEM);
        }
        __syncthreads();
        const unsigned call = *(LAS unsigned*)(lds + STAGE_BYTES);
        const __amdgpu_buffer_rsrc_t ro = __builtin_amdgcn_make_buffer_rsrc(P.out[rank], 0, (int)OUT_BYTES, 0x00020000);
        const __amdgpu_buffer_rsrc_t rqo = __builtin_amdgcn_make_buffer_rsrc(P.out[q], 0, (int)OUT_BYTES, 0x00020000);
        bool ok = true;
        for (int idx = slot; idx < NTILE && ok; idx += CE_WGS) {
            int i, j; tile_of_index(idx, i, j);
            ok = wave_wait_ge(ctl + (L_TILE + i * NPW + j) * 32, 8u * call, tmo, call);
            if (!ok) break;
            __builtin_amdgcn_fence(__ATOMIC_ACQUIRE, "wavefront");
            const unsigned off = (unsigned)(row0 + 256 * i + 32 * wid) * 16384u + (unsigned)(col0 + 256 * j) * 2u + rlane;
            copy_rows<8>(ro, off, rqo, off, 32768u);
            copy_rows<8>(ro, off + 8u * 32768u, rqo, off + 8u * 32768u, 32768u);
        }
        asm volatile("s_waitcnt vmcnt(0)" ::: "memory");
        if (lane == 0) __hip_atomic_fetch_add(qctl + (L_DONE + rank) * 32, 1u, __ATOMIC_RELAXED, __HIP_MEMORY_SCOPE_SYSTEM);
        if (w == 0 && ok) ok = wave_wait_ge(ctl + (L_DONE + q) * 32, (unsigned)CE_WAVES * call, tmo, call);
        if (!ok && lane == 0) ((unsigned short*)P.out[rank])[(size_t)row0 * GN + col0] = 0x7fc0;
        return;
    }

    const int ts = slot - CE_WGS;
    const int ti = (xcd >> 1) * 4 + (ts >> 2), tj = (xcd & 1) * 4 + (ts & 3);
    const int wr = wid >> 2, wc = wid & 3, fr = lane & 15, fq = lane >> 4;
    unsigned voffA[2], voffB[2];
#pragma unroll
    for (int i = 0; i < 2; ++i) {
        int R, C; stage_rc(tid * 16 + i * 8192, R, C); voffA[i] = (unsigned)(R * KS + C) * 2u;
        const unsigned b = (unsigned)(tid * 16 + i * 8192), row = b >> 8, ch = ((b >> 4) & 15u) ^ wswz(row);
        voffB[i] = row * 16384u + ch * 16u;
    }
    constexpr size_t kstepA = BK * 2, hstepA = (size_t)HALF * KS * 2, kstepB = (size_t)BK * GN * 2, hstepB = HALF * 2;
    const unsigned ldsw = (unsigned)wid * 1024u;
    const int aoff = lds_byte(wr * 64 + fr, fq * 8);
    int boff[2][2];
#pragma unroll
    for (int n = 0; n < 2; ++n)
#pragma unroll
        for (int t = 0; t < 2; ++t) {
            const unsigned q4 = (unsigned)(lane & 15) >> 2, p4 = (unsigned)lane & 3u, row = 8u * fq + 4u * t + q4, ch = 4u * wc + 2u * n + (p4 >> 1);
            boff[n][t] = (int)((unsigned)(size_t)lds + 4u * HTB + 256u * row + 16u * (ch ^ wswz(row)) + 8u * (p4 & 1u));
        }
#define SA(b, h) (((b) * 2 + (h)) * HTB)
#define SB(b, h) ((4 + (b) * 2 + (h)) * HTB)
#define STAGE(bufoff, gbase, voff) do { _Pragma("unroll") for (int _i = 0; _i < 2; ++_i) \
        __builtin_amdgcn_global_load_lds((const unsigned*)((const char*)(gbase) + (voff)[_i]), (LAS unsigned*)(lds + (bufoff) + ldsw + _i * 8192), 16, 0, 0); } while (0)
#define LDA(dst, b, h) do { _Pragma("unroll") for (int m = 0; m < 4; ++m) _Pragma("unroll") for (int k = 0; k < 2; ++k) dst[m][k] = *(const LAS bf16x8*)(lds + SA(b, h) + aoff + m * 2048 + k * 1024); } while (0)
#define TRR(dst, base, off) asm volatile("ds_read_b64_tr_b16 %0, %1 offset:%2" : "=&v"(dst) : "v"(base), "i"(off) : "memory")
#define LDB(raw, b, h) do { \
        TRR(raw[0], boff[0][0], SB(b, h) - 4 * HTB); TRR(raw[1], boff[0][1], SB(b, h) - 4 * HTB); TRR(raw[2], boff[0][0], SB(b, h) - 4 * HTB + 8192); TRR(raw[3], boff[0][1], SB(b, h) - 4 * HTB + 8192); \
        TRR(raw[4], boff[1][0], SB(b, h) - 4 * HTB); TRR(raw[5], boff[1][1], SB(b, h) - 4 * HTB); TRR(raw[6], boff[1][0], SB(b, h) - 4 * HTB + 8192); TRR(raw[7], boff[1][1], SB(b, h) - 4 * HTB + 8192); } while (0)
#define WAIT_B(raw, dst) do { asm volatile("s_waitcnt lgkmcnt(0)" : "+v"(raw[0]), "+v"(raw[1]), "+v"(raw[2]), "+v"(raw[3]), "+v"(raw[4]), "+v"(raw[5]), "+v"(raw[6]), "+v"(raw[7]) :: "memory"); \
        _Pragma("unroll") for (int n = 0; n < 2; ++n) _Pragma("unroll") for (int k = 0; k < 2; ++k) dst[n][k] = __builtin_shufflevector(raw[n * 4 + k * 2], raw[n * 4 + k * 2 + 1], 0, 1, 2, 3, 4, 5, 6, 7); } while (0)
#define MMA(ai, bj, At, Bt) do { __builtin_amdgcn_s_setprio(1); _Pragma("unroll") for (int m = 0; m < 4; ++m) _Pragma("unroll") for (int n = 0; n < 2; ++n) _Pragma("unroll") for (int k = 0; k < 2; ++k) \
        acc[ai][bj][m][n] = __builtin_amdgcn_mfma_f32_16x16x32_bf16(Bt[n][k], At[m][k], acc[ai][bj][m][n], 0, 0, 0); __builtin_amdgcn_s_setprio(0); } while (0)
#define WAIT_V(n) asm volatile("s_waitcnt vmcnt(" #n ")" ::: "memory")
#define WAIT_L(n) asm volatile("s_waitcnt lgkmcnt(" #n ")" ::: "memory")
#define BAR __builtin_amdgcn_s_barrier()
#define SCHED __builtin_amdgcn_sched_barrier(0)

    __syncthreads();
    const unsigned call = (unsigned)__builtin_amdgcn_readfirstlane(*(LAS unsigned*)(lds + STAGE_BYTES));

    f32x4 acc[2][2][4][2];
#pragma unroll
    for (int a = 0; a < 2; ++a)
#pragma unroll
        for (int b = 0; b < 2; ++b)
#pragma unroll
            for (int m = 0; m < 4; ++m)
#pragma unroll
                for (int n = 0; n < 2; ++n) acc[a][b][m][n] = (f32x4){0.f, 0.f, 0.f, 0.f};
    bf16x8 At[4][2], B0[2][2], B1[2][2];
    s16x4 R0[8], R1[8];
    const char* ownA = (const char*)P.x + (size_t)(row0 + 256 * ti) * (KS * 2);
    const char* ownB = (const char*)P.w + (size_t)(col0 + 256 * tj) * 2;
    const char* cA = ownA; const char* cB = ownB;
    constexpr int nt = KS / BK;
    STAGE(SB(0, 0), cB, voffB); STAGE(SA(0, 0), cA, voffA); STAGE(SB(0, 1), cB + hstepB, voffB); STAGE(SA(0, 1), cA + hstepA, voffA);
    if (wr == 1) BAR;
    WAIT_V(4); BAR;
    STAGE(SB(1, 0), cB + kstepB, voffB); STAGE(SA(1, 0), cA + kstepA, voffA); STAGE(SB(1, 1), cB + hstepB + kstepB, voffB);
    WAIT_V(6); BAR;
    for (int s = 0; s < NR; ++s) {
        const bool has_next = s + 1 < NR;
        const int np = (rank + s + 1) & 7;
        const char* nA = has_next ? myws + XS_OFF + (size_t)np * XS_SLOT + (size_t)(256 * ti) * (KS * 2) : ownA;
        const char* nB = has_next ? myws + WS_OFF + (size_t)(np >> 2) * WS_ARR + (size_t)((np & 3) * BC + 256 * tj) * 2 : ownB;
        for (int t = 0; t < nt; t += 2) {
            const bool last = (t == nt - 2);
            const char* a1 = cA + (size_t)(t + 1) * kstepA;
            const char* a2 = last ? nA : cA + (size_t)(t + 2) * kstepA; const char* b2 = last ? nB : cB + (size_t)(t + 2) * kstepB;
            const char* a3 = a2 + kstepA; const char* b3 = b2 + kstepB;
            if (last && has_next) {
                if (wid == 0) {
                    wave_wait_ge(ctl + (L_PANEL + np * 32 + ((lane & 1) ? 16 + tj : ti)) * 32, (unsigned)CE_WAVES * call, tmo, call);
                    __builtin_amdgcn_fence(__ATOMIC_ACQUIRE, "");
                    asm volatile("s_waitcnt vmcnt(0)" ::: "memory");
                }
                asm volatile("" ::: "memory"); BAR; asm volatile("" ::: "memory");
            }
            LDB(R0, 0, 0); SCHED; LDA(At, 0, 0); STAGE(SA(1, 1), a1 + hstepA, voffA);
            WAIT_L(8); BAR; WAIT_B(R0, B0); MMA(0, 0, At, B0); BAR; SCHED;
            LDB(R1, 0, 1); STAGE(SB(0, 0), b2, voffB);
            BAR; WAIT_B(R1, B1); MMA(0, 1, At, B1); BAR;
            LDA(At, 0, 1); STAGE(SA(0, 0), a2, voffA);
            BAR; WAIT_L(0); MMA(1, 0, At, B0); BAR; SCHED;
            STAGE(SB(0, 1), b2 + hstepB, voffB);
            WAIT_V(6); BAR; MMA(1, 1, At, B1); BAR;
            LDB(R0, 1, 0); SCHED; LDA(At, 1, 0); STAGE(SA(0, 1), a2 + hstepA, voffA);
            WAIT_L(8); BAR; WAIT_B(R0, B0); MMA(0, 0, At, B0); BAR; SCHED;
            LDB(R1, 1, 1); STAGE(SB(1, 0), b3, voffB);
            BAR; WAIT_B(R1, B1); MMA(0, 1, At, B1); BAR;
            LDA(At, 1, 1); STAGE(SA(1, 0), a3, voffA);
            BAR; WAIT_L(0); MMA(1, 0, At, B0); BAR; SCHED;
            STAGE(SB(1, 1), b3 + hstepB, voffB);
            WAIT_V(6); BAR; MMA(1, 1, At, B1); BAR;
        }
        cA = nA; cB = nB;
    }
    WAIT_V(0);
    if (wr == 0) BAR;
    {
        const bool bad = (unsigned)__builtin_amdgcn_readfirstlane(ld_relaxed_agent(tmo)) == call;
        const __amdgpu_buffer_rsrc_t ro = __builtin_amdgcn_make_buffer_rsrc(P.out[rank], 0, (int)OUT_BYTES, 0x00020000);
        const unsigned obase = (unsigned)(row0 + 256 * ti + 64 * wr + fr) * 16384u + (unsigned)(col0 + 256 * tj + 32 * wc + (fq & 1) * 16 + (fq >> 1) * 8) * 2u;
#pragma unroll
        for (int ai = 0; ai < 2; ++ai)
#pragma unroll
            for (int m = 0; m < 4; ++m)
#pragma unroll
                for (int bj = 0; bj < 2; ++bj) {
                    const f32x4 v0 = acc[ai][bj][m][0], v1 = acc[ai][bj][m][1];
                    const unsigned a0 = cvt_pk_bf16(v0[0], v0[1]), a1 = cvt_pk_bf16(v0[2], v0[3]), b0 = cvt_pk_bf16(v1[0], v1[1]), b1 = cvt_pk_bf16(v1[2], v1[3]);
                    const auto r0 = __builtin_amdgcn_permlane16_swap(a0, b0, false, false);
                    const auto r1 = __builtin_amdgcn_permlane16_swap(a1, b1, false, false);
                    u32x4 wv = {r0[0], r1[0], r0[1], r1[1]};
                    if (bad) wv = (u32x4){0x7fc07fc0u, 0x7fc07fc0u, 0x7fc07fc0u, 0x7fc07fc0u};
                    __builtin_amdgcn_raw_buffer_store_b128(wv, ro, obase + (unsigned)(ai * HALF + m * 16) * 16384u + (unsigned)bj * 256u, 0, 16  );
                }
        asm volatile("s_waitcnt vmcnt(0)" ::: "memory");
        if (lane == 0) __hip_atomic_fetch_add(ctl + (L_TILE + ti * NPW + tj) * 32, 1u, __ATOMIC_RELAXED, __HIP_MEMORY_SCOPE_AGENT);
    }
}

}

extern "C" void kernel_launch(int rank, int nranks, ncclComm_t comm,
                              hipStream_t stream,
                              void* const* d_in, const int* in_sizes, int n_in,
                              void* d_out, int out_size,
                              const int* in_dtypes, int out_dtype,
                              void* const* d_out_all,
                              void* const* d_ws_all, size_t ws_bytes) {
    if (nranks != NR || (unsigned)rank >= (unsigned)NR || n_in < 2 || ws_bytes < WS_END || (size_t)out_size != (size_t)GM * GN ||
        (size_t)in_sizes[0] != (size_t)GM * KS || (size_t)in_sizes[1] != (size_t)KS * GN) {
        fprintf(stderr, "kernel_launch: unexpected problem shape\n");
        return;
    }
    Params P;
    P.x = d_in[0]; P.w = d_in[1]; P.rank = rank;
    for (int p = 0; p < NR; ++p) { P.out[p] = d_out_all[p]; P.ws[p] = d_ws_all[p]; }
    hipLaunchKernelGGL(gemm_ar_kernel, dim3(GRID), dim3(512), 0, stream, P);
    HIP_CHECK(hipGetLastError());
}
```

```cpp
#include <hip/hip_runtime.h>
#include <rccl/rccl.h>
#include <cstdio>

#define HIP_CHECK(expr) do { hipError_t _e = (expr); if (_e != hipSuccess) { \
    fprintf(stderr, "HIP error %s at %s:%d\n", hipGetErrorString(_e), __FILE__, __LINE__); \
    return; } } while (0)

namespace {

#define LAS __attribute__((address_space(3)))
typedef short bf16x8 __attribute__((ext_vector_type(8)));
typedef short s16x4 __attribute__((ext_vector_type(4)));
typedef float f32x4 __attribute__((ext_vector_type(4)));
typedef unsigned u32x4 __attribute__((ext_vector_type(4)));

constexpr int NR = 8;
constexpr int GM = 8192, GN = 8192;
constexpr int KS = 1024;
constexpr int NTILE = 256;

#define K_TR 18
#define K_NF 28
#define K_NG 28
#define K_LEAD 2
#define K_N0 16
#define K_M 7
#define K_MX 100
#define K_SHELLS {6, 7, 8, 10, 12, 14, 16}
#define K_SCR 1
#define K_CW 2
#define K_FT 1
#define K_STRICT 1
#define K_SLEEP 8
#define K_CE 14
constexpr int TR = K_TR, NF = K_NF, NG = K_NG, LEAD = K_LEAD, N0 = K_N0, MS = K_M, MX = K_MX;
static_assert(TR % 2 == 0 && TR <= 32 && NF % 4 == 0 && NG % 4 == 0 && NF + NG <= 64 && N0 % 4 == 0 && MS >= 1 && MS < 15 && MX % 4 == 0 && MX >= N0 && MX <= 256, "knobs");
constexpr int NF1 = (NF + NG) / 4;
constexpr int NS2 = NF1 + NF / 4;
constexpr int ND = 64 - NF - NG, NH = 64 - NF;

constexpr unsigned CTL_BYTES = 1u << 20;
constexpr unsigned XS_OFF = CTL_BYTES, XS_SLOT = 16u * 524288u;
constexpr unsigned WS_OFF = XS_OFF + 3u * XS_SLOT, WS_ARR = 1024u * 16384u;
constexpr unsigned RL_OFF = WS_OFF + 2u * WS_ARR;
constexpr unsigned PT_OFF = RL_OFF + 4u * 32u * 16u * 8192u;
constexpr unsigned RT_OFF = PT_OFF + 256u * 131072u;
constexpr unsigned FT_OFF = RT_OFF + 256u * 131072u;
constexpr unsigned WS_END = FT_OFF + 256u * 131072u;
constexpr unsigned OUT_BYTES = (unsigned)((size_t)GM * GN * 2);
constexpr unsigned IN_BYTES = (unsigned)((size_t)GM * KS * 2);

constexpr int L_TMO = 1, L_CEEXIT = 2, L_CALL = 3  , L_DONE = 24  , L_START = 32  ,
              L_HEAD = 64  , L_PANEL = 128  , L_RELAY = 256  , L_TILE = 512  ,
              L_SCNT = 1024  ;
constexpr int CE_WGS = K_CE;
constexpr int G_WGS = 32 - CE_WGS;
constexpr int GRID = 256;
constexpr unsigned long long DEADLINE_TICKS = 50000000ull;
constexpr unsigned long long WATCH_DELAY = 1000;

constexpr int Q1X_ITEMS = TR * (NF1 + NS2);
constexpr int Q2_ITEMS = 64 * 16;
constexpr int Q2_AFTER = 128;
constexpr int Q3I_ITEMS = (MX - N0) * MS + (NTILE - MX) * (MS + 1);
constexpr int Q3X_ITEMS = (N0 * 3 / 4) * 16 + ((MX - N0) * 3 / 4) * (16 - MS) + ((NTILE - MX) * 3 / 4) * (15 - MS);
constexpr int Q3L_ITEMS = NTILE * 4;

constexpr int BK = 64, HALF = 128, HTB = HALF * BK * 2, STAGE_BYTES = 8 * HTB;

#define STAMP(i) do {} while (0)
#define PROG(q, it) do {} while (0)

struct Params {
    const void* x; const void* w;
    void* out[8]; void* ws[8];
    int g, a;
    unsigned tab[448];
};
constexpr int T_ORD = 0  , T_ORD2 = 256  , T_ORD3 = 512  , T_ORDX = 1280  ,
              T_ORDF = 1536  ;
#define TAB(i) ((P.tab[(unsigned)(i) >> 2] >> (((unsigned)(i) & 3u) * 8u)) & 255u)

__device__ __forceinline__ int lds_byte(int r, int c) { const int st = (r >> 4) * 2 + (c >> 5), rr = r & 15, cc = c & 31, ob = rr * 64 + cc * 2; return st * 1024 + (ob ^ (((ob >> 9) & 1) << 5)); }
__device__ __forceinline__ void stage_rc(int b, int& R, int& C) { const int st = b / 1024, sb = b % 1024, swz = sb ^ (((sb >> 9) & 1) << 5); R = (st >> 1) * 16 + swz / 64; C = (st & 1) * 32 + (swz % 64) / 2; }
__device__ __forceinline__ unsigned wswz(unsigned row) { return ((row & 3u) << 2) | ((row >> 2) & 3u); }
__device__ __forceinline__ unsigned cvt_pk_bf16(float lo, float hi) { unsigned r; asm volatile("v_cvt_pk_bf16_f32 %0, %1, %2" : "=v"(r) : "v"(lo), "v"(hi)); return r; }
__device__ __forceinline__ unsigned panel_base(int g, int ab, int I) {
    const int bb = ab ^ (I & 1), q = g == 0 ? 2 * ab + bb : 2 * bb + ab, j = I >> 1;
    return (unsigned)((j << 2) | ((q + (0xb114 >> (2 * j))) & 3)) * 256u;
}

__device__ __forceinline__ unsigned ld_relaxed_sys(const unsigned* p) { return __hip_atomic_load(p, __ATOMIC_RELAXED, __HIP_MEMORY_SCOPE_SYSTEM); }
__device__ __forceinline__ unsigned ld_relaxed_agent(const unsigned* p) { return __hip_atomic_load(p, __ATOMIC_RELAXED, __HIP_MEMORY_SCOPE_AGENT); }

__device__ __forceinline__ bool gave_up(unsigned long long t0, unsigned* tmo, unsigned call) {
    if ((unsigned)__builtin_amdgcn_readfirstlane(ld_relaxed_agent(tmo)) == call || __builtin_amdgcn_s_memrealtime() - t0 > DEADLINE_TICKS) {
        __hip_atomic_store(tmo, call, __ATOMIC_RELAXED, __HIP_MEMORY_SCOPE_AGENT);
        return true;
    }
    return false;
}
__device__ __forceinline__ bool wave_wait_ge(const unsigned* word, unsigned need, unsigned* tmo, unsigned call) {
    if (__all((int)(ld_relaxed_sys(word) - need) >= 0)) return true;
    const unsigned long long t0 = __builtin_amdgcn_s_memrealtime();
    for (unsigned polls = 1;; ++polls) {
        __builtin_amdgcn_s_sleep(4);
        if (__all((int)(ld_relaxed_sys(word) - need) >= 0)) return true;
        if ((polls & 63u) == 0u && gave_up(t0, tmo, call)) return false;
    }
}

__device__ __forceinline__ unsigned claim(unsigned* ctr) {
    unsigned r; unsigned long long sv;
    asm volatile("s_nop 4\n\ts_mov_b64 %1, exec\n\ts_mov_b64 exec, 1\n\tglobal_atomic_add %0, %2, %3, %4 sc0\n\ts_waitcnt vmcnt(0)\n\ts_mov_b64 exec, %1"
                 : "=&v"(r), "=&s"(sv) : "v"(0u), "v"(1u), "s"(ctr) : "memory");
    return (unsigned)__builtin_amdgcn_readfirstlane(r);
}
__device__ __forceinline__ unsigned claim_poll(unsigned* ctr, const unsigned* base, unsigned off, unsigned& ticket) {
    unsigned r, v; unsigned long long sv;
    asm volatile("s_nop 4\n\ts_mov_b64 %2, exec\n\ts_mov_b64 exec, 1\n\tglobal_atomic_add %0, %3, %4, %5 sc0\n\ts_mov_b64 exec, %2\n\tglobal_load_dword %1, %6, %7 sc0 sc1\n\ts_waitcnt vmcnt(0)"
                 : "=&v"(r), "=&v"(v), "=&s"(sv) : "v"(0u), "v"(1u), "s"(ctr), "v"(off), "s"(base) : "memory");
    ticket = (unsigned)__builtin_amdgcn_readfirstlane(r);
    return v;
}
__device__ __forceinline__ void add_sys(unsigned* ctr, unsigned v) {
    unsigned long long sv;
    asm volatile("s_nop 4\n\ts_mov_b64 %0, exec\n\ts_mov_b64 exec, 1\n\tglobal_atomic_add %1, %2, %3 sc1\n\ts_mov_b64 exec, %0"
                 : "=&s"(sv) : "v"(0u), "v"(v), "s"(ctr) : "memory");
}
__device__ __forceinline__ void add_agent(unsigned* ctr, unsigned v) {
    unsigned long long sv;
    asm volatile("s_nop 4\n\ts_mov_b64 %0, exec\n\ts_mov_b64 exec, 1\n\tglobal_atomic_add %1, %2, %3\n\ts_mov_b64 exec, %0"
                 : "=&s"(sv) : "v"(0u), "v"(v), "s"(ctr) : "memory");
}
__device__ __forceinline__ unsigned add_bf16x2(unsigned a, unsigned b) {
    return cvt_pk_bf16(__uint_as_float(a << 16) + __uint_as_float(b << 16), __uint_as_float(a & 0xffff0000u) + __uint_as_float(b & 0xffff0000u));
}

__global__ __launch_bounds__(512, 2) void gemm_ar_kernel(const Params P) {
    __shared__ __attribute__((aligned(16))) unsigned char shm[STAGE_BYTES + 64];
    LAS unsigned char* lds = (LAS unsigned char*)shm;
    const int tid = threadIdx.x, wid = __builtin_amdgcn_readfirstlane(tid >> 6), lane = tid & 63;
    const int grp = P.g, a1 = P.a >> 1, a0 = P.a & 1;
    const int xl = blockIdx.x & 7, slot = blockIdx.x >> 3;
    char* myws = (char*)P.ws[3];
    unsigned* ctl = (unsigned*)myws;
    unsigned* tmo = ctl + L_TMO * 32;
    STAMP(0);

    if (slot < CE_WGS) {
        int e = (int)((unsigned)__builtin_amdgcn_s_getreg(20 | (3 << 11)) & 7u);
        bool active = wid < K_CW || e == 3;
        if (wid == 7) {
            if (slot == 0 && xl != 3) add_sys((unsigned*)P.ws[xl] + (L_START + xl) * 32, 1u);
            const unsigned long long t0 = __builtin_amdgcn_s_memrealtime();
            while (__builtin_amdgcn_s_memrealtime() - t0 < WATCH_DELAY) __builtin_amdgcn_s_sleep(16);
            if (__builtin_amdgcn_readfirstlane(ld_relaxed_agent(ctl + (L_HEAD + xl * 3 + (xl == 3 ? 2 : 0)) * 32)) == 0) { e = xl; active = true; }
        }
        char* qws = (char*)P.ws[e];
        unsigned* qctl = (unsigned*)qws;
        const __amdgpu_buffer_rsrc_t rq = __builtin_amdgcn_make_buffer_rsrc(qws, 0, (int)WS_END, 0x00020000);
        const __amdgpu_buffer_rsrc_t rl = __builtin_amdgcn_make_buffer_rsrc(myws, 0, (int)WS_END, 0x00020000);
        const __amdgpu_buffer_rsrc_t rqo = __builtin_amdgcn_make_buffer_rsrc(P.out[e], 0, (int)OUT_BYTES, 0x00020000);
        const unsigned llane = (unsigned)lane * 16u;
        const unsigned rlane = (unsigned)(lane >> 5) * 16384u + (unsigned)(lane & 31) * 16u;
        const int d = e + 1, c = (e - 4) & 3;
        const int nX = (d & 2) ? (d == 3 ? NH : ND) : 64, nW = (d & 1) ? (d == 3 ? NH : ND) : 64;
        const unsigned n1 = e < 3 ? (unsigned)((TR / 2) * (nX + nW) + (32 - TR) * 64) : (unsigned)Q1X_ITEMS;
        const unsigned n3 = e == 3 ? (unsigned)Q3L_ITEMS : (e < 3 ? (unsigned)Q3I_ITEMS : (unsigned)Q3X_ITEMS);
        unsigned* const head0 = ctl + (L_HEAD + e * 3) * 32;
        unsigned* const scnt = ctl + (L_SCNT + e * 8) * 32;
        const unsigned cold = (unsigned)(L_START + e) * 128u;
        int mq = e == 3 ? 2 : 0;
        unsigned pm = 0, call = 0;
        if (active) call = (unsigned)__builtin_amdgcn_readfirstlane(claim_poll(head0 + mq * 32, ctl, L_CALL * 128u, pm)) + 1u;
        else mq = 3;
        STAMP(4);
        constexpr unsigned NONE = 0xffffffffu;
        bool ok = true, fresh = true, mready = false, fin = false, ready2 = false, want2 = e > 3 && active;
        unsigned p2 = NONE, tl2 = 0, mine = 0, idle = 0;
        unsigned long long t0 = 0;
        int src = 0; unsigned so = 0, dO = 0, depA = 0, needA = 0, depB = 0, needB = 0, sig = 0, seg = 0, segn = 1; bool sstr = false, dstr = false;
        for (;;) {
            if (fresh) {
                fresh = false;
                if (mq == 0 && pm >= n1) { mq = 2; pm = claim(head0 + 64); STAMP(1); }
                if (mq == 2 && pm >= n3) mq = 3;
                needA = needB = 0; depA = depB = cold;
                const unsigned it = pm;
                if (mq == 0) {
                    int t, u, dd; bool first = false, second = false; int cp = 0, m = 0;
                    if (e < 3) {
                        dd = d;
                        const unsigned nrel = (unsigned)((TR / 2) * (nX + nW));
                        if (it < nrel) { const unsigned pr = it / (unsigned)(nX + nW), rem = it - pr * (unsigned)(nX + nW);
                            if (rem < (unsigned)nX) { t = 2 * (int)pr; u = 64 - nX + (int)rem; } else { t = 2 * (int)pr + 1; u = 64 - nW + (int)rem - nX; } }
                        else { t = TR + (int)((it - nrel) >> 6); u = (int)((it - nrel) & 63u); }
                        seg = (unsigned)t; segn = t >= TR ? 64u : (unsigned)((t & 1) ? nW : nX);
                    } else {
                        int r;
                        if (it < (unsigned)(LEAD * NF1)) { t = (int)it / NF1; r = (int)it - t * NF1; first = true; }
                        else { const unsigned i2 = it - (unsigned)(LEAD * NF1);
                            if (i2 < (unsigned)((TR - LEAD) * (NF1 + NS2))) { const int sl = (int)(i2 / (unsigned)(NF1 + NS2)); r = (int)i2 - sl * (NF1 + NS2);
                                if (r < NF1) { t = sl + LEAD; first = true; } else { t = sl; r -= NF1; } }
                            else { const unsigned i3 = i2 - (unsigned)((TR - LEAD) * (NF1 + NS2)); const int sl = (int)(i3 / (unsigned)NS2); t = TR - LEAD + sl; r = (int)i3 - sl * NS2; } }
                        second = !first;
                        if (first) { m = r; cp = c; dd = 3; }
                        else { if (r < NF1) { m = r; dd = (t & 1) ? 1 : 2; } else { m = r - NF1; dd = 3; } cp = c ^ dd; }
                        u = 4 * m + cp;
                        seg = (unsigned)(t + (first ? 0 : (dd == 3 ? 64 : 32))); segn = (second && dd == 3) ? (unsigned)(NF / 4) : (unsigned)NF1;
                    }
                    const bool isw = t & 1; const int I = t >> 1;
                    const unsigned rel = RL_OFF + (unsigned)((cp * 32 + t) * 16 + m) * 8192u;
                    if (second) { src = 2; so = rel; sstr = false; depA = (unsigned)(L_RELAY + cp * 32 + t) * 128u; needA = (unsigned)NF1 * call; }
                    else if (isw) { src = 1; so = (unsigned)u * 16u * 16384u + panel_base(grp, a0 ^ (dd & 1), I) * 2u; sstr = true; }
                    else { src = 0; so = panel_base(grp, a1 ^ (dd >> 1), I) * 2048u + (unsigned)u * 8192u; sstr = false; }
                    if (first) { dO = rel; dstr = false; sig = (unsigned)(L_RELAY + cp * 32 + t) * 32u; }
                    else {
                        const int sl = dd - 1;
                        if (isw) { dO = WS_OFF + (unsigned)(sl >> 1) * WS_ARR + (unsigned)u * 16u * 16384u + (unsigned)((sl & 1) * 4096 + 256 * I) * 2u; dstr = true; }
                        else { dO = XS_OFF + (unsigned)sl * XS_SLOT + (unsigned)I * 524288u + (unsigned)u * 8192u; dstr = false; }
                        sig = (unsigned)(L_PANEL + sl * 32 + t) * 32u;
                    }
                } else if (mq == 2) {
                    unsigned tl, st;
                    if (e == 3) { tl = T_ORD + (it >> 2); st = (it & 3u) * 4u; }
                    else if (e < 3) {
                        if (it < (unsigned)((MX - N0) * MS)) { const unsigned k = it / (unsigned)MS; tl = T_ORDF + N0 + k; st = it - k * (unsigned)MS; }
                        else { const unsigned i2 = it - (unsigned)((MX - N0) * MS), k = i2 / (unsigned)(MS + 1); tl = T_ORDF + MX + k; st = i2 - k * (unsigned)(MS + 1); }
                    } else {
                        constexpr unsigned B0 = (N0 * 3 / 4) * 16, B1 = B0 + ((MX - N0) * 3 / 4) * (16 - MS);
                        if (it < B0) { tl = it >> 4; st = it & 15u; }
                        else if (it < B1) { const unsigned i2 = it - B0, k = i2 / (unsigned)(16 - MS); tl = N0 * 3 / 4 + k; st = (unsigned)MS + i2 - k * (unsigned)(16 - MS); }
                        else { const unsigned i2 = it - B1, k = i2 / (unsigned)(15 - MS); tl = MX * 3 / 4 + k; st = (unsigned)(MS + 1) + i2 - k * (unsigned)(15 - MS); }
                        tl += (unsigned)(T_ORD3 + c * 192);
                    }
                    tl = TAB(tl);
                    depA = (unsigned)(L_TILE + tl) * 128u; needA = 8u * call; depB = depA + 64u; needB = 16u * call;
                    so = tl * 131072u + st * 8192u;
                    dO = (panel_base(grp, a1, (int)(tl >> 4)) + st * 16u) * 16384u + panel_base(grp, a0, (int)(tl & 15u)) * 2u;
                }
                mready = needA == 0u || !ok; fin = false;
            }
            bool look = false;
            if (want2 && (mq != 0 || pm >= (unsigned)Q2_AFTER)) {
                want2 = false; ready2 = !ok; look = K_STRICT && ok;
                p2 = claim(head0 + 32);
                if (p2 >= (unsigned)Q2_ITEMS) { p2 = NONE; STAMP(2); } else tl2 = TAB(T_ORD2 + c * 64 + (p2 >> 4));
            }
            if (mq == 3 && p2 == NONE) break;
            const int act = (p2 != NONE && ready2) ? 1 : ((mq != 3 && mready && !(look && p2 != NONE)) ? 2 : 0);
            __builtin_amdgcn_fence(__ATOMIC_ACQUIRE, "wavefront");
            unsigned sg = 0, sgc = 0, sgn = 1;
            if (act == 1) PROG(1, p2); else if (act == 2) PROG(mq, pm);
            if (act == 2 && mq == 2 && fin) {
                u32x4 v[8];
#pragma unroll
                for (int k = 0; k < 8; ++k) v[k] = __builtin_amdgcn_raw_buffer_load_b128(rl, FT_OFF + so + llane + k * 1024u, 0, 16);
#pragma unroll
                for (int k = 0; k < 8; ++k) __builtin_amdgcn_raw_buffer_store_b128(v[k], rqo, dO + rlane + k * 32768u, 0, 17);
                ++mine;
            } else if (act == 2 && mq == 2) {
                for (int sub = 0; sub < (e == 3 ? 4 : 1); ++sub) {
                    const unsigned sv = so + llane + (unsigned)sub * 8192u, dv = dO + rlane + (unsigned)sub * 262144u;
                    u32x4 va[8], vb[8];
#pragma unroll
                    for (int k = 0; k < 8; ++k) { va[k] = __builtin_amdgcn_raw_buffer_load_b128(rl, PT_OFF + sv + k * 1024u, 0, 16); vb[k] = __builtin_amdgcn_raw_buffer_load_b128(rl, RT_OFF + sv + k * 1024u, 0, 16); }
#pragma unroll
                    for (int k = 0; k < 8; ++k) {
                        u32x4 o = {add_bf16x2(va[k].x, vb[k].x), add_bf16x2(va[k].y, vb[k].y), add_bf16x2(va[k].z, vb[k].z), add_bf16x2(va[k].w, vb[k].w)};
                        if (!ok) o = (u32x4){0x7fc07fc0u, 0x7fc07fc0u, 0x7fc07fc0u, 0x7fc07fc0u};
                        __builtin_amdgcn_raw_buffer_store_b128(o, rqo, dv + k * 32768u, 0, 17);
                        if (e == 3) __builtin_amdgcn_raw_buffer_store_b128(o, rl, FT_OFF + sv + k * 1024u, 0, 16);
                    }
                }
                if (e == 3) { sg = (depA >> 2) + 8u; sgc = pm >> 2; sgn = 4u; }
                ++mine;
            } else if (act) {
                int csrc = src; unsigned cso = so, cdo = dO; bool cs = sstr, cd = dstr; sg = sig; sgc = seg; sgn = segn;
                if (act == 1) {
                    const unsigned o = tl2 * 131072u + (p2 & 15u) * 8192u;
                    csrc = 2; cso = PT_OFF + o; cdo = RT_OFF + o; cs = cd = false; sg = (unsigned)(L_TILE + tl2) * 32u + 16u; sgc = 128u + (p2 >> 4); sgn = 16u;
                    p2 = NONE; want2 = true;
                }
                const __amdgpu_buffer_rsrc_t rs = __builtin_amdgcn_make_buffer_rsrc(csrc == 0 ? (void*)P.x : csrc == 1 ? (void*)P.w : (void*)myws, 0, csrc == 2 ? (int)WS_END : (int)IN_BYTES, 0x00020000);
                const unsigned sv = cso + (cs ? rlane : llane), dv = cdo + (cd ? rlane : llane), ss = cs ? 32768u : 1024u, ds = cd ? 32768u : 1024u;
                u32x4 v[8];
#pragma unroll
                for (int k = 0; k < 8; ++k) v[k] = __builtin_amdgcn_raw_buffer_load_b128(rs, sv + k * ss, 0, 16  );
#pragma unroll
                for (int k = 0; k < 8; ++k) __builtin_amdgcn_raw_buffer_store_b128(v[k], rq, dv + k * ds, 0, 17  );
            } else {
                if (idle == 0u) t0 = __builtin_amdgcn_s_memrealtime(); else __builtin_amdgcn_s_sleep(K_SLEEP);
                if ((++idle & 63u) == 0u && gave_up(t0, tmo, call)) { ok = false; ready2 = true; mready = true; }
            }
            const bool p2w = p2 != NONE && !ready2;
            const unsigned m0 = lane == 0 ? ~0u : 0u, m1 = lane == 1 ? ~0u : 0u, m2 = (lane == 2 && p2w) ? ~0u : 0u, m3 = (lane == 3 && p2w) ? ~0u : 0u, m4 = lane == 4 ? ~0u : 0u;
            const unsigned poff = (depA & m0) | (depB & m1) | (((unsigned)(L_TILE + tl2) * 128u) & m2) | ((depA + 32u) & m4) | (cold & ~(m0 | m1 | m2 | m4));
            const unsigned pneed = (needA & m0) | (needB & m1) | ((needB >> 2) & m4) | ((8u * call) & m2) | (call & m3);
            unsigned pv;
            if (act == 2) { pv = claim_poll(head0 + mq * 32, ctl, poff, pm); fresh = true; }
            else { pv = __builtin_amdgcn_raw_buffer_load_b32(rl, poff, 0, 17); asm volatile("s_waitcnt vmcnt(0)" : "+v"(pv) :: "memory"); }
            if (sg) { if (claim(scnt + sgc) % sgn == sgn - 1u) add_sys(qctl + sg, sgn); }
            if (act) idle = 0;
            const unsigned okm = (unsigned)__ballot((int)(pv - pneed) >= 0);
            if (p2w && (okm & 12u) == 12u) ready2 = true;
            if (!fresh && (okm & 3u) == 3u) { mready = true; fin = K_FT && ok && e != 3 && (okm & 16u); }
        }
        if (mine && e != 3) add_sys(qctl + (L_DONE + e) * 32, mine);
        STAMP(3);
        __syncthreads();
        if (wid == 0 && claim(ctl + L_CEEXIT * 32) % (unsigned)(8 * CE_WGS) == (unsigned)(8 * CE_WGS) - 1u) {
            if (lane < 25) __hip_atomic_store(ctl + (lane == 24 ? L_CALL : L_HEAD + lane) * 32, lane == 24 ? call : 0u, __ATOMIC_RELAXED, __HIP_MEMORY_SCOPE_AGENT);
        }
        STAMP(30);
        if (blockIdx.x == 0 && wid == 0) {
            const int pe = (lane & 7) == 3 ? 0 : (lane & 7);
            if (ok) ok = wave_wait_ge(ctl + (L_DONE + pe) * 32, (pe < 3 ? (unsigned)Q3I_ITEMS : (unsigned)Q3X_ITEMS) * call, tmo, call);
        }
        STAMP(31);
        if (!ok && lane == 0) ((unsigned short*)P.out[3])[(size_t)panel_base(grp, a1, 0) * GN + panel_base(grp, a0, 0)] = 0x7fc0;
        return;
    }

    const unsigned call = (unsigned)__builtin_amdgcn_readfirstlane(ld_relaxed_agent(ctl + L_CALL * 32)) + 1u;
    const int wr = wid >> 2, wc = wid & 3, fr = lane & 15, fq = lane >> 4;
    unsigned voffA[2], voffB[2];
#pragma unroll
    for (int i = 0; i < 2; ++i) {
        int R, C; stage_rc(tid * 16 + i * 8192, R, C); voffA[i] = (unsigned)(R * KS + C) * 2u;
        const unsigned b = (unsigned)(tid * 16 + i * 8192), row = b >> 8, ch = ((b >> 4) & 15u) ^ wswz(row);
        voffB[i] = row * 16384u + ch * 16u;
    }
    constexpr size_t kstepA = BK * 2, hstepA = (size_t)HALF * KS * 2, kstepB = (size_t)BK * GN * 2, hstepB = HALF * 2;
    const unsigned ldsw = (unsigned)wid * 1024u;
    const int aoff = lds_byte(wr * 64 + fr, fq * 8);
    int boff[2][2];
#pragma unroll
    for (int n = 0; n < 2; ++n)
#pragma unroll
        for (int t = 0; t < 2; ++t) {
            const unsigned q4 = (unsigned)(lane & 15) >> 2, p4 = (unsigned)lane & 3u, row = 8u * fq + 4u * t + q4, ch = 4u * wc + 2u * n + (p4 >> 1);
            boff[n][t] = (int)((unsigned)(size_t)lds + 4u * HTB + 256u * row + 16u * (ch ^ wswz(row)) + 8u * (p4 & 1u));
        }
#define SA(b, h) (((b) * 2 + (h)) * HTB)
#define SB(b, h) ((4 + (b) * 2 + (h)) * HTB)
#define STAGE(bufoff, gbase, voff) do { _Pragma("unroll") for (int _i = 0; _i < 2; ++_i) \
        __builtin_amdgcn_global_load_lds((const unsigned*)((const char*)(gbase) + (voff)[_i]), (LAS unsigned*)(lds + (bufoff) + ldsw + _i * 8192), 16, 0, 0); } while (0)
#define LDA(dst, b, h) do { _Pragma("unroll") for (int m = 0; m < 4; ++m) _Pragma("unroll") for (int k = 0; k < 2; ++k) dst[m][k] = *(const LAS bf16x8*)(lds + SA(b, h) + aoff + m * 2048 + k * 1024); } while (0)
#define TRR(dst, base, off) asm volatile("ds_read_b64_tr_b16 %0, %1 offset:%2" : "=&v"(dst) : "v"(base), "i"(off) : "memory")
#define LDB(raw, b, h) do { \
        TRR(raw[0], boff[0][0], SB(b, h) - 4 * HTB); TRR(raw[1], boff[0][1], SB(b, h) - 4 * HTB); TRR(raw[2], boff[0][0], SB(b, h) - 4 * HTB + 8192); TRR(raw[3], boff[0][1], SB(b, h) - 4 * HTB + 8192); \
        TRR(raw[4], boff[1][0], SB(b, h) - 4 * HTB); TRR(raw[5], boff[1][1], SB(b, h) - 4 * HTB); TRR(raw[6], boff[1][0], SB(b, h) - 4 * HTB + 8192); TRR(raw[7], boff[1][1], SB(b, h) - 4 * HTB + 8192); } while (0)
#define WAIT_B(raw, dst) do { asm volatile("s_waitcnt lgkmcnt(0)" : "+v"(raw[0]), "+v"(raw[1]), "+v"(raw[2]), "+v"(raw[3]), "+v"(raw[4]), "+v"(raw[5]), "+v"(raw[6]), "+v"(raw[7]) :: "memory"); \
        _Pragma("unroll") for (int n = 0; n < 2; ++n) _Pragma("unroll") for (int k = 0; k < 2; ++k) dst[n][k] = __builtin_shufflevector(raw[n * 4 + k * 2], raw[n * 4 + k * 2 + 1], 0, 1, 2, 3, 4, 5, 6, 7); } while (0)
#define MMA(ai, bj, At, Bt) do { __builtin_amdgcn_s_setprio(1); _Pragma("unroll") for (int m = 0; m < 4; ++m) _Pragma("unroll") for (int n = 0; n < 2; ++n) _Pragma("unroll") for (int k = 0; k < 2; ++k) \
        acc[ai][bj][m][n] = __builtin_amdgcn_mfma_f32_16x16x32_bf16(Bt[n][k], At[m][k], acc[ai][bj][m][n], 0, 0, 0); __builtin_amdgcn_s_setprio(0); } while (0)
#define WAIT_V(n) asm volatile("s_waitcnt vmcnt(" #n ")" ::: "memory")
#define WAIT_L(n) asm volatile("s_waitcnt lgkmcnt(" #n ")" ::: "memory")
#define BAR __builtin_amdgcn_s_barrier()
#define SCHED __builtin_amdgcn_sched_barrier(0)

    for (int job = slot - CE_WGS; job < 32; job += G_WGS) {
        const int tl = (int)TAB(T_ORDX + xl * 32 + job), ti = tl >> 4, tj = tl & 15;
        f32x4 acc[2][2][4][2];
#pragma unroll
        for (int a = 0; a < 2; ++a)
#pragma unroll
            for (int b = 0; b < 2; ++b)
#pragma unroll
                for (int m = 0; m < 4; ++m)
#pragma unroll
                    for (int n = 0; n < 2; ++n) acc[a][b][m][n] = (f32x4){0.f, 0.f, 0.f, 0.f};
        bf16x8 At[4][2], B0[2][2], B1[2][2];
        s16x4 R0[8], R1[8];
        const char* ownA = (const char*)P.x + (size_t)panel_base(grp, a1, ti) * (KS * 2);
        const char* ownB = (const char*)P.w + (size_t)panel_base(grp, a0, tj) * 2;
        const char* cA = ownA; const char* cB = ownB;
        constexpr int nt = KS / BK;
        unsigned todo = 7u;
        STAGE(SB(0, 0), cB, voffB); STAGE(SA(0, 0), cA, voffA); STAGE(SB(0, 1), cB + hstepB, voffB); STAGE(SA(0, 1), cA + hstepA, voffA);
        if (wr == 1) BAR;
        WAIT_V(4); BAR;
        STAGE(SB(1, 0), cB + kstepB, voffB); STAGE(SA(1, 0), cA + kstepA, voffA); STAGE(SB(1, 1), cB + hstepB + kstepB, voffB);
        WAIT_V(6); BAR;
        for (int s = 0; s < 4; ++s) {
            const char* nA = ownA; const char* nB = ownB;
            for (int t = 0; t < nt; t += 2) {
                const bool last = (t == nt - 2);
                if (last && s < 3) {
                    if (wid == 0) {
                        STAMP(1 + 2 * s + (job >= G_WGS ? 8 : 0));
                        const unsigned* word = ctl + (L_PANEL + ((lane >> 1) % 3) * 32 + ((lane & 1) ? 2 * tj + 1 : 2 * ti)) * 32;
                        const unsigned long long t0 = __builtin_amdgcn_s_memrealtime();
                        unsigned pick;
                        for (unsigned polls = 1;; ++polls) {
                            const unsigned mk = (unsigned)__ballot((int)(ld_relaxed_sys(word) - 64u * call) >= 0);
                            const unsigned both = mk & (mk >> 1);
                            pick = ((both & 1u) | ((both >> 1) & 2u) | ((both >> 2) & 4u)) & todo;
                            if (pick) break;
                            if ((polls & 63u) == 0u && gave_up(t0, tmo, call)) { pick = todo; break; }
                            __builtin_amdgcn_s_sleep(4);
                        }
                        STAMP(2 + 2 * s + (job >= G_WGS ? 8 : 0));
                        *(LAS unsigned*)(lds + STAGE_BYTES + 16) = (unsigned)__builtin_ctz(pick);
                        __builtin_amdgcn_fence(__ATOMIC_ACQUIRE, "");
                        asm volatile("s_waitcnt vmcnt(0) lgkmcnt(0)" ::: "memory");
                    }
                    asm volatile("" ::: "memory"); BAR; asm volatile("" ::: "memory");
                    unsigned slv;
                    asm volatile("ds_read_b32 %0, %1\n\ts_waitcnt lgkmcnt(0)" : "=&v"(slv) : "v"((unsigned)(size_t)lds + STAGE_BYTES + 16) : "memory");
                    const unsigned sl = (unsigned)__builtin_amdgcn_readfirstlane(slv) & 3u;
                    todo &= ~(1u << sl);
                    nA = myws + XS_OFF + sl * XS_SLOT + (unsigned)ti * 524288u;
                    nB = myws + WS_OFF + (sl >> 1) * WS_ARR + ((sl & 1u) * 4096u + 256u * (unsigned)tj) * 2u;
                }
                const char* a1p = cA + (size_t)(t + 1) * kstepA;
                const char* a2 = last ? nA : cA + (size_t)(t + 2) * kstepA; const char* b2 = last ? nB : cB + (size_t)(t + 2) * kstepB;
                const char* a3 = a2 + kstepA; const char* b3 = b2 + kstepB;
                LDB(R0, 0, 0); SCHED; LDA(At, 0, 0); STAGE(SA(1, 1), a1p + hstepA, voffA);
                WAIT_L(8); BAR; WAIT_B(R0, B0); MMA(0, 0, At, B0); BAR; SCHED;
                LDB(R1, 0, 1); STAGE(SB(0, 0), b2, voffB);
                BAR; WAIT_B(R1, B1); MMA(0, 1, At, B1); BAR;
                LDA(At, 0, 1); STAGE(SA(0, 0), a2, voffA);
                BAR; WAIT_L(0); MMA(1, 0, At, B0); BAR; SCHED;
                STAGE(SB(0, 1), b2 + hstepB, voffB);
                WAIT_V(6); BAR; MMA(1, 1, At, B1); BAR;
                LDB(R0, 1, 0); SCHED; LDA(At, 1, 0); STAGE(SA(0, 1), a2 + hstepA, voffA);
                WAIT_L(8); BAR; WAIT_B(R0, B0); MMA(0, 0, At, B0); BAR; SCHED;
                LDB(R1, 1, 1); STAGE(SB(1, 0), b3, voffB);
                BAR; WAIT_B(R1, B1); MMA(0, 1, At, B1); BAR;
                LDA(At, 1, 1); STAGE(SA(1, 0), a3, voffA);
                BAR; WAIT_L(0); MMA(1, 0, At, B0); BAR; SCHED;
                STAGE(SB(1, 1), b3 + hstepB, voffB);
                WAIT_V(6); BAR; MMA(1, 1, At, B1); BAR;
            }
            cA = nA; cB = nB;
        }
        WAIT_V(0);
        if (wr == 0) BAR;
        STAMP(7 + (job >= G_WGS ? 8 : 0));
        {
            const bool bad = (unsigned)__builtin_amdgcn_readfirstlane(ld_relaxed_agent(tmo)) == call;
            const __amdgpu_buffer_rsrc_t ro = __builtin_amdgcn_make_buffer_rsrc(myws + PT_OFF + (unsigned)tl * 131072u, 0, 131072, 0x00020000);
            const unsigned obase = (unsigned)(64 * wr + fr) * 512u + (unsigned)(32 * wc + (fq & 1) * 16 + (fq >> 1) * 8) * 2u;
#pragma unroll
            for (int ai = 0; ai < 2; ++ai)
#pragma unroll
                for (int m = 0; m < 4; ++m)
#pragma unroll
                    for (int bj = 0; bj < 2; ++bj) {
                        const f32x4 v0 = acc[ai][bj][m][0], v1 = acc[ai][bj][m][1];
                        const unsigned x0 = cvt_pk_bf16(v0[0], v0[1]), x1 = cvt_pk_bf16(v0[2], v0[3]), y0 = cvt_pk_bf16(v1[0], v1[1]), y1 = cvt_pk_bf16(v1[2], v1[3]);
                        const auto r0 = __builtin_amdgcn_permlane16_swap(x0, y0, false, false);
                        const auto r1 = __builtin_amdgcn_permlane16_swap(x1, y1, false, false);
                        u32x4 wv = {r0[0], r1[0], r0[1], r1[1]};
                        if (bad) wv = (u32x4){0x7fc07fc0u, 0x7fc07fc0u, 0x7fc07fc0u, 0x7fc07fc0u};
                        __builtin_amdgcn_raw_buffer_store_b128(wv, ro, obase + (unsigned)(ai * HALF + m * 16) * 512u + (unsigned)bj * 256u, 0, 16  );
                    }
            asm volatile("s_waitcnt vmcnt(0)" ::: "memory");
            BAR;
            if (wid == 0) add_agent(ctl + (L_TILE + tl) * 32, 8u);
            STAMP(8 + (job >= G_WGS ? 8 : 0));
        }
    }
}

}

extern "C" void kernel_launch(int rank, int nranks, ncclComm_t comm,
                              hipStream_t stream,
                              void* const* d_in, const int* in_sizes, int n_in,
                              void* d_out, int out_size,
                              const int* in_dtypes, int out_dtype,
                              void* const* d_out_all,
                              void* const* d_ws_all, size_t ws_bytes) {
    if (nranks != NR || (unsigned)rank >= (unsigned)NR || n_in < 2 || ws_bytes < WS_END || (size_t)out_size != (size_t)GM * GN ||
        (size_t)in_sizes[0] != (size_t)GM * KS || (size_t)in_sizes[1] != (size_t)KS * GN) {
        fprintf(stderr, "kernel_launch: unexpected problem shape\n");
        return;
    }
    const int member[2][4] = {{0, 2, 3, 6}, {1, 4, 5, 7}};
    Params P;
    P.x = d_in[0]; P.w = d_in[1]; P.g = 0; P.a = 0;
    for (int g = 0; g < 2; ++g) for (int a = 0; a < 4; ++a) if (member[g][a] == rank) { P.g = g; P.a = a; }
    for (int e = 0; e < 8; ++e) {
        const int peer = e < 3 ? member[P.g][P.a ^ (e + 1)] : (e == 3 ? rank : member[1 - P.g][P.a ^ (e - 4)]);
        P.out[e] = d_out_all[peer]; P.ws[e] = d_ws_all[peer];
    }
    unsigned char* tb = (unsigned char*)P.tab;
    int n = 0, n2[4] = {0, 0, 0, 0}, n3[4] = {0, 0, 0, 0}, nx[8] = {0, 0, 0, 0, 0, 0, 0, 0};
    for (int k = 0; k < 16; ++k)
        for (int r = 0; r < 2 * k + 1; ++r) {
            const int I = r < k ? k : r - k, J = r < k ? r : k;
            const unsigned char tl = (unsigned char)(I << 4 | J);
            const int c = (I & 1) << 1 | (J & 1), x = (I & 3) << 1 | ((J >> 1) & 1);
            tb[T_ORDF + n] = tl; tb[T_ORD + n++] = tl; tb[T_ORD2 + c * 64 + n2[c]++] = tl; tb[T_ORDX + x * 32 + nx[x]++] = tl;
        }
    {
        const int shell[] = K_SHELLS;
        unsigned rng = 12345u;
        for (unsigned si = 0; si + 1 < sizeof(shell) / sizeof(shell[0]); ++si) {
            const int lo = shell[si] * shell[si], hi = shell[si + 1] * shell[si + 1];
            for (int i = hi - 1; i > lo; --i) { rng = rng * 1664525u + 1013904223u; const int j = lo + (int)((rng >> 8) % (unsigned)(i - lo + 1)); const unsigned char t = tb[T_ORDF + i]; tb[T_ORDF + i] = tb[T_ORDF + j]; tb[T_ORDF + j] = t; }
        }
        for (int i = 0; i < NTILE; ++i) { const unsigned char tl = tb[T_ORDF + i]; const int c = ((tl >> 4) & 1) << 1 | (tl & 1);
            for (int o = 0; o < 4; ++o) if (o != c) tb[T_ORD3 + o * 192 + n3[o]++] = tl; }
    }
    hipLaunchKernelGGL(gemm_ar_kernel, dim3(GRID), dim3(512), 0, stream, P);
    HIP_CHECK(hipGetLastError());
}
```

```cpp
#include <hip/hip_runtime.h>
#include <rccl/rccl.h>
#include <cstdio>

#define HIP_CHECK(expr) do { hipError_t _e = (expr); if (_e != hipSuccess) { \
    fprintf(stderr, "HIP error %s at %s:%d\n", hipGetErrorString(_e), __FILE__, __LINE__); \
    return; } } while (0)

namespace {

#define LAS __attribute__((address_space(3)))
typedef short bf16x8 __attribute__((ext_vector_type(8)));
typedef short s16x4 __attribute__((ext_vector_type(4)));
typedef float f32x4 __attribute__((ext_vector_type(4)));
typedef unsigned u32x4 __attribute__((ext_vector_type(4)));

constexpr int NR = 8;
constexpr int GM = 8192, GN = 8192;
constexpr int KS = 1024;
constexpr int NTILE = 256;

#define K_TR 12
#define K_NF 40
#define K_NG 12
#define K_LEAD 3
#define K_N0 36
#define K_M 8
#define K_NP 12
#define K_SMALL 64
#define K_MARK 200
#define K_SHELLS {6, 7, 8, 10, 12, 14, 16}
#define K_SCR 1
#define K_CW 1
#define K_FT 1
#define K_SLEEP 8
#define K_CE 16
constexpr int TR = K_TR, NF = K_NF, NG = K_NG, LEAD = K_LEAD, N0 = K_N0, MS = K_M, NP = K_NP, NS = 256 - 4 * NP;
static_assert(TR % 2 == 0 && TR <= 32 && NF % 4 == 0 && NG % 4 == 0 && NF + NG <= 64 && N0 % 4 == 0 && MS >= 1 && MS < 16 && NP >= 1 && NP <= 39, "knobs");
constexpr int NF1 = (NF + NG) / 4;
constexpr int NS2 = NF1 + NF / 4;
constexpr int ND = 64 - NF - NG, NH = 64 - NF;

constexpr unsigned CTL_BYTES = 1u << 20;
constexpr unsigned XS_OFF = CTL_BYTES, XS_SLOT = 16u * 524288u;
constexpr unsigned WS_OFF = XS_OFF + 3u * XS_SLOT;
constexpr unsigned RL_OFF = WS_OFF + 3u * XS_SLOT;
constexpr unsigned PT_OFF = RL_OFF + 4u * 32u * 16u * 8192u;
constexpr unsigned RT_OFF = PT_OFF + 256u * 131072u;
constexpr unsigned FT_OFF = RT_OFF + 256u * 131072u;
constexpr unsigned WS_END = FT_OFF + 256u * 131072u;
constexpr unsigned OUT_BYTES = (unsigned)((size_t)GM * GN * 2);
constexpr unsigned IN_BYTES = (unsigned)((size_t)GM * KS * 2);

constexpr int L_TMO = 1, L_EXIT = 2, L_CALL = 3  , L_NFIN = 4, L_ALLFIN = 5, L_DONE = 24  , L_START = 32  ,
              L_HEAD = 64  , L_MARK = 40  , L_SHARE = 48  , L_PANEL = 128  , L_RELAY = 256  , L_TILE = 512  ,
              L_SCNT = 1024  ;
constexpr int CE_WGS = K_CE;
constexpr int G_WGS = 32 - CE_WGS;
constexpr int GRID = 256;
constexpr unsigned long long DEADLINE_TICKS = 50000000ull;
constexpr unsigned long long WATCH_DELAY = 1000;

constexpr int Q1X_ITEMS = TR * (NF1 + NS2);
constexpr int Q2_ITEMS = 64 * 16;
constexpr int Q2_AFTER = 128;
constexpr int Q3I_ITEMS = (NS - N0) * MS;
constexpr int Q3X_ITEMS = (N0 * 3 / 4) * 16 + ((NS - N0) * 3 / 4) * (16 - MS);
constexpr int POOL_ITEMS = NP * 16;
constexpr int FIX_N = (TR / 2) * (64 + ND) + (32 - TR) * 64 + Q3I_ITEMS, FIX_D = TR * NH + (32 - TR) * 64 + Q3I_ITEMS, FIX_X = Q1X_ITEMS + Q2_ITEMS + Q3X_ITEMS;
constexpr int MARK_AT = (FIX_N < FIX_D ? (FIX_N < FIX_X ? FIX_N : FIX_X) : (FIX_D < FIX_X ? FIX_D : FIX_X)) - K_MARK;
constexpr int Q3L_ITEMS = NTILE * 4;

constexpr int BK = 64, HALF = 128, HTB = HALF * BK * 2, STAGE_BYTES = 8 * HTB;

#define STAMP(i) do {} while (0)
#define PROG(q, it) do {} while (0)

struct Params {
    const void* x; const void* w;
    void* out[8]; void* ws[8];
    int g, a;
    unsigned njob[8], job[256];
    unsigned tab[488];
};
constexpr int T_ORD = 0  , T_ORD2 = 256  , T_ORD3 = 512  , T_ORDX = 1280  ,
              T_ORDF = 1536  ,
              T_POOL = 1792  ;
#define TAB(i) ((P.tab[(unsigned)(i) >> 2] >> (((unsigned)(i) & 3u) * 8u)) & 255u)

__device__ __forceinline__ int lds_byte(int r, int c) { const int st = (r >> 4) * 2 + (c >> 5), rr = r & 15, cc = c & 31, ob = rr * 64 + cc * 2; return st * 1024 + (ob ^ (((ob >> 9) & 1) << 5)); }
__device__ __forceinline__ void stage_rc(int b, int& R, int& C) { const int st = b / 1024, sb = b % 1024, swz = sb ^ (((sb >> 9) & 1) << 5); R = (st >> 1) * 16 + swz / 64; C = (st & 1) * 32 + (swz % 64) / 2; }
__device__ __forceinline__ unsigned wswz(unsigned row) { return ((row & 3u) << 2) | ((row >> 2) & 3u); }
__device__ __forceinline__ unsigned cvt_pk_bf16(float lo, float hi) { unsigned r; asm volatile("v_cvt_pk_bf16_f32 %0, %1, %2" : "=v"(r) : "v"(lo), "v"(hi)); return r; }
__device__ __forceinline__ unsigned panel_base(int g, int ab, int I) {
    const int bb = ab ^ (I & 1), q = g == 0 ? 2 * ab + bb : 2 * bb + ab, j = I >> 1;
    return (unsigned)((j << 2) | ((q + (0xb114 >> (2 * j))) & 3)) * 256u;
}

__device__ __forceinline__ unsigned ld_relaxed_sys(const unsigned* p) { return __hip_atomic_load(p, __ATOMIC_RELAXED, __HIP_MEMORY_SCOPE_SYSTEM); }
__device__ __forceinline__ unsigned ld_relaxed_agent(const unsigned* p) { return __hip_atomic_load(p, __ATOMIC_RELAXED, __HIP_MEMORY_SCOPE_AGENT); }

__device__ __forceinline__ bool gave_up(unsigned long long t0, unsigned* tmo, unsigned call) {
    if ((unsigned)__builtin_amdgcn_readfirstlane(ld_relaxed_agent(tmo)) == call || __builtin_amdgcn_s_memrealtime() - t0 > DEADLINE_TICKS) {
        __hip_atomic_store(tmo, call, __ATOMIC_RELAXED, __HIP_MEMORY_SCOPE_AGENT);
        return true;
    }
    return false;
}
__device__ __forceinline__ unsigned add_bf16x2(unsigned a, unsigned b) {
    return cvt_pk_bf16(__uint_as_float(a << 16) + __uint_as_float(b << 16), __uint_as_float(a & 0xffff0000u) + __uint_as_float(b & 0xffff0000u));
}

__device__ __forceinline__ void wg_exit(unsigned* ctl, unsigned call, int wid, int lane) {
    if (wid != 0) return;
    const unsigned t = (unsigned)__builtin_amdgcn_raw_ptr_buffer_atomic_add_i32(1, __builtin_amdgcn_make_buffer_rsrc(ctl, 0, (int)CTL_BYTES, 0x00020000), lane == 0 ? (unsigned)L_EXIT * 128u : 0x7fffff00u, 0, 0);
    if (((unsigned)__builtin_amdgcn_readfirstlane((int)t) & (unsigned)(GRID - 1)) == (unsigned)(GRID - 1)) {
        if (lane < 25) __hip_atomic_store(ctl + (lane == 24 ? L_CALL : L_HEAD + lane) * 32, lane == 24 ? call : 0u, __ATOMIC_RELAXED, __HIP_MEMORY_SCOPE_AGENT);
    }
}

template <bool SMALL>
__device__ __forceinline__ int gemm_job(const Params& P, LAS unsigned char* lds, char* myws, unsigned call, int tl, int qd, int xl, int round) {
    int tid = threadIdx.x; asm volatile("" : "+v"(tid));
    const int wid = __builtin_amdgcn_readfirstlane(tid >> 6), lane = tid & 63;
    const int grp = P.g, a1 = P.a >> 1, a0 = P.a & 1, ti = tl >> 4, tj = tl & 15;
    unsigned* ctl = (unsigned*)myws;
    unsigned* tmo = ctl + L_TMO * 32;
    const int wr = wid >> 2, wc = wid & 3, fr = lane & 15, fq = lane >> 4;
    unsigned voffA[2], voffB[2];
#pragma unroll
    for (int i = 0; i < 2; ++i) { int R, C; stage_rc(tid * 16 + i * 8192, R, C); voffA[i] = (unsigned)(R * KS + C) * 2u; }
#define SET_VOFFB(sh) do { int tz = tid; asm volatile("" : "+v"(tz)); _Pragma("unroll") for (int i = 0; i < 2; ++i) { \
        const unsigned row = (unsigned)(tz >> 4) + 32u * i, ch = ((unsigned)tz & 15u) ^ wswz(row); voffB[i] = (row << (sh)) + ch * 16u; } \
        kstepB = (size_t)(SMALL ? 2 * BK : BK) << (sh); hstepB = SMALL ? (size_t)BK << (sh) : (size_t)HALF * 2; } while (0)
    size_t kstepB, hstepB;
    constexpr size_t kstepA = SMALL ? BK * 4 : BK * 2, hstepA = SMALL ? BK * 2 : (size_t)HALF * KS * 2;
    const unsigned qa = SMALL ? (unsigned)(qd >> 1) * (unsigned)(HALF * KS * 2) : 0u, qb = SMALL ? (unsigned)(qd & 1) * 256u : 0u;
    const unsigned ldsw = (unsigned)wid * 1024u;
    const int aoff = lds_byte(wr * 64 + fr, fq * 8);
    int boff[2][2];
#pragma unroll
    for (int n = 0; n < 2; ++n)
#pragma unroll
        for (int t = 0; t < 2; ++t) {
            const unsigned q4 = (unsigned)(lane & 15) >> 2, p4 = (unsigned)lane & 3u, row = 8u * fq + 4u * t + q4, ch = 4u * wc + 2u * n + (p4 >> 1);
            boff[n][t] = (int)((unsigned)(size_t)lds + 4u * HTB + 256u * row + 16u * (ch ^ wswz(row)) + 8u * (p4 & 1u));
        }
#define SA(b, h) (((b) * 2 + (h)) * HTB)
#define SB(b, h) ((4 + (b) * 2 + (h)) * HTB)
#define STAGE(bufoff, gbase, voff) do { _Pragma("unroll") for (int _i = 0; _i < 2; ++_i) \
        __builtin_amdgcn_global_load_lds((const unsigned*)((const char*)(gbase) + (voff)[_i]), (LAS unsigned*)(lds + (bufoff) + ldsw + _i * 8192), 16, 0, 0); } while (0)
#define LDA(dst, b, h) do { _Pragma("unroll") for (int m = 0; m < 4; ++m) _Pragma("unroll") for (int k = 0; k < 2; ++k) dst[m][k] = *(const LAS bf16x8*)(lds + SA(b, h) + aoff + m * 2048 + k * 1024); } while (0)
#define TRR(dst, base, off) asm volatile("ds_read_b64_tr_b16 %0, %1 offset:%2" : "=&v"(dst) : "v"(base), "i"(off) : "memory")
#define LDB(raw, b, h) do { \
        TRR(raw[0], boff[0][0], SB(b, h) - 4 * HTB); TRR(raw[1], boff[0][1], SB(b, h) - 4 * HTB); TRR(raw[2], boff[0][0], SB(b, h) - 4 * HTB + 8192); TRR(raw[3], boff[0][1], SB(b, h) - 4 * HTB + 8192); \
        TRR(raw[4], boff[1][0], SB(b, h) - 4 * HTB); TRR(raw[5], boff[1][1], SB(b, h) - 4 * HTB); TRR(raw[6], boff[1][0], SB(b, h) - 4 * HTB + 8192); TRR(raw[7], boff[1][1], SB(b, h) - 4 * HTB + 8192); } while (0)
#define WAIT_B(raw, dst) do { asm volatile("s_waitcnt lgkmcnt(0)" : "+v"(raw[0]), "+v"(raw[1]), "+v"(raw[2]), "+v"(raw[3]), "+v"(raw[4]), "+v"(raw[5]), "+v"(raw[6]), "+v"(raw[7]) :: "memory"); \
        _Pragma("unroll") for (int n = 0; n < 2; ++n) _Pragma("unroll") for (int k = 0; k < 2; ++k) dst[n][k] = __builtin_shufflevector(raw[n * 4 + k * 2], raw[n * 4 + k * 2 + 1], 0, 1, 2, 3, 4, 5, 6, 7); } while (0)
#define MMA(ai, bj, At, Bt) do { if (!SMALL || (ai) == (bj)) { __builtin_amdgcn_s_setprio(1); _Pragma("unroll") for (int m = 0; m < 4; ++m) _Pragma("unroll") for (int n = 0; n < 2; ++n) _Pragma("unroll") for (int k = 0; k < 2; ++k) \
        acc[SMALL ? 0 : ai][SMALL ? 0 : bj][m][n] = __builtin_amdgcn_mfma_f32_16x16x32_bf16(Bt[n][k], At[m][k], acc[SMALL ? 0 : ai][SMALL ? 0 : bj][m][n], 0, 0, 0); __builtin_amdgcn_s_setprio(0); } } while (0)
#define WAIT_V(n) asm volatile("s_waitcnt vmcnt(" #n ")" ::: "memory")
#define WAIT_L(n) asm volatile("s_waitcnt lgkmcnt(" #n ")" ::: "memory")
#define BAR __builtin_amdgcn_s_barrier()
#define SCHED __builtin_amdgcn_sched_barrier(0)
    constexpr int NH2 = SMALL ? 1 : 2;
    f32x4 acc[NH2][NH2][4][2];
#pragma unroll
    for (int a = 0; a < NH2; ++a)
#pragma unroll
        for (int b = 0; b < NH2; ++b)
#pragma unroll
            for (int m = 0; m < 4; ++m)
#pragma unroll
                for (int n = 0; n < 2; ++n) acc[a][b][m][n] = (f32x4){0.f, 0.f, 0.f, 0.f};
    bf16x8 At[4][2], B0[2][2], B1[2][2];
    s16x4 R0[8], R1[8];
    const char* cA = (const char*)P.x + (size_t)panel_base(grp, a1, ti) * (KS * 2) + qa;
    const char* cB = (const char*)P.w + (size_t)panel_base(grp, a0, tj) * 2 + qb;
    constexpr int nt = SMALL ? KS / BK / 2 : KS / BK;
    unsigned todo = 7u;
    SET_VOFFB(14);
    STAGE(SB(0, 0), cB, voffB); STAGE(SA(0, 0), cA, voffA); STAGE(SB(0, 1), cB + hstepB, voffB); STAGE(SA(0, 1), cA + hstepA, voffA);
    if (wr == 1) BAR;
    WAIT_V(4); BAR;
    STAGE(SB(1, 0), cB + kstepB, voffB); STAGE(SA(1, 0), cA + kstepA, voffA); STAGE(SB(1, 1), cB + hstepB + kstepB, voffB);
    WAIT_V(6); BAR;
    for (int s = 0; s < 4; ++s) {
        const char* nA = cA; const char* nB = cB;
        for (int t = 0; t < nt; t += 2) {
            const bool last = (t == nt - 2);
            if (last && s < 3) {
                if (wid == 0) {
                    STAMP(1 + 2 * s + 8 * round);
                    const unsigned* word = ctl + (L_PANEL + ((lane >> 1) % 3) * 32 + ((lane & 1) ? 2 * tj + 1 : 2 * ti)) * 32;
                    const unsigned long long t0 = __builtin_amdgcn_s_memrealtime();
                    unsigned pick;
                    for (unsigned polls = 1;; ++polls) {
                        const unsigned mk = (unsigned)__ballot((int)(ld_relaxed_sys(word) - 64u * call) >= 0);
                        const unsigned both = mk & (mk >> 1);
                        pick = ((both & 1u) | ((both >> 1) & 2u) | ((both >> 2) & 4u)) & todo;
                        if (pick) break;
                        if ((polls & 63u) == 0u && gave_up(t0, tmo, call)) { pick = todo; break; }
                        __builtin_amdgcn_s_sleep(4);
                    }
                    STAMP(2 + 2 * s + 8 * round);
                    *(LAS unsigned*)(lds + STAGE_BYTES + 16) = (unsigned)__builtin_ctz(pick);
                    __builtin_amdgcn_fence(__ATOMIC_ACQUIRE, "");
                    asm volatile("s_waitcnt vmcnt(0) lgkmcnt(0)" ::: "memory");
                }
                asm volatile("" ::: "memory"); BAR; asm volatile("" ::: "memory");
                unsigned slv;
                asm volatile("ds_read_b32 %0, %1\n\ts_waitcnt lgkmcnt(0)" : "=&v"(slv) : "v"((unsigned)(size_t)lds + STAGE_BYTES + 16) : "memory");
                const unsigned sl = (unsigned)__builtin_amdgcn_readfirstlane(slv) & 3u;
                todo &= ~(1u << sl);
                nA = myws + XS_OFF + sl * XS_SLOT + (unsigned)ti * 524288u + qa;
                nB = myws + WS_OFF + sl * XS_SLOT + (unsigned)tj * 524288u + qb;
                if (s == 0) SET_VOFFB(9);
            }
            const char* a1p = cA + (size_t)(t + 1) * kstepA;
            const char* a2 = last ? nA : cA + (size_t)(t + 2) * kstepA; const char* b2 = last ? nB : cB + (size_t)(t + 2) * kstepB;
            const char* a3 = a2 + kstepA; const char* b3 = b2 + kstepB;
            LDB(R0, 0, 0); SCHED; LDA(At, 0, 0); STAGE(SA(1, 1), a1p + hstepA, voffA);
            WAIT_L(8); BAR; WAIT_B(R0, B0); MMA(0, 0, At, B0); BAR; SCHED;
            LDB(R1, 0, 1); STAGE(SB(0, 0), b2, voffB);
            BAR; WAIT_B(R1, B1); MMA(0, 1, At, B1); BAR; if (SMALL) SCHED;
            LDA(At, 0, 1); STAGE(SA(0, 0), a2, voffA);
            BAR; WAIT_L(0); MMA(1, 0, At, B0); BAR; SCHED;
            STAGE(SB(0, 1), b2 + hstepB, voffB);
            WAIT_V(6); BAR; MMA(1, 1, At, B1); BAR;
            LDB(R0, 1, 0); SCHED; LDA(At, 1, 0); STAGE(SA(0, 1), a2 + hstepA, voffA);
            WAIT_L(8); BAR; WAIT_B(R0, B0); MMA(0, 0, At, B0); BAR; SCHED;
            LDB(R1, 1, 1); STAGE(SB(1, 0), b3, voffB);
            BAR; WAIT_B(R1, B1); MMA(0, 1, At, B1); BAR; if (SMALL) SCHED;
            LDA(At, 1, 1); STAGE(SA(1, 0), a3, voffA);
            BAR; WAIT_L(0); MMA(1, 0, At, B0); BAR; SCHED;
            STAGE(SB(1, 1), b3 + hstepB, voffB);
            WAIT_V(6); BAR; MMA(1, 1, At, B1); BAR;
        }
        cA = nA; cB = nB;
    }
    WAIT_V(0);
    if (wr == 0) BAR;
    STAMP(7 + 8 * round);
    const __amdgpu_buffer_rsrc_t rc = __builtin_amdgcn_make_buffer_rsrc(myws, 0, (int)CTL_BYTES, 0x00020000);
    const unsigned nj = (unsigned)__builtin_amdgcn_raw_ptr_buffer_atomic_add_i32(1, rc, tid == 0 ? (unsigned)(L_HEAD + 16 + xl) * 128u : 0x7fffff00u, 0, 0);
    const bool bad = (unsigned)__builtin_amdgcn_readfirstlane(ld_relaxed_agent(tmo)) == call;
    const __amdgpu_buffer_rsrc_t ro = __builtin_amdgcn_make_buffer_rsrc(myws + PT_OFF + (unsigned)tl * 131072u, 0, 131072, 0x00020000);
    const unsigned obase = (unsigned)(64 * wr + fr + (SMALL ? (qd >> 1) * HALF : 0)) * 512u + (unsigned)(32 * wc + (fq & 1) * 16 + (fq >> 1) * 8) * 2u + qb;
#pragma unroll
    for (int ai = 0; ai < NH2; ++ai)
#pragma unroll
        for (int m = 0; m < 4; ++m)
#pragma unroll
            for (int bj = 0; bj < NH2; ++bj) {
                const f32x4 v0 = acc[ai][bj][m][0], v1 = acc[ai][bj][m][1];
                const unsigned x0 = cvt_pk_bf16(v0[0], v0[1]), x1 = cvt_pk_bf16(v0[2], v0[3]), y0 = cvt_pk_bf16(v1[0], v1[1]), y1 = cvt_pk_bf16(v1[2], v1[3]);
                const auto r0 = __builtin_amdgcn_permlane16_swap(x0, y0, false, false);
                const auto r1 = __builtin_amdgcn_permlane16_swap(x1, y1, false, false);
                u32x4 wv = {r0[0], r1[0], r0[1], r1[1]};
                if (bad) wv = (u32x4){0x7fc07fc0u, 0x7fc07fc0u, 0x7fc07fc0u, 0x7fc07fc0u};
                __builtin_amdgcn_raw_buffer_store_b128(wv, ro, obase + (unsigned)(ai * HALF + m * 16) * 512u + (unsigned)bj * 256u, 0, 16  );
            }
    if (wid == 0) *(LAS unsigned*)(lds + STAGE_BYTES) = (unsigned)__builtin_amdgcn_readfirstlane((int)nj);
    asm volatile("s_waitcnt vmcnt(0) lgkmcnt(0)" ::: "memory");
    BAR;
    (void)__builtin_amdgcn_raw_ptr_buffer_atomic_add_i32(SMALL ? 2 : 8, rc, tid == 0 ? (unsigned)(L_TILE + tl) * 128u : 0x7fffff00u, 0, 0);
    STAMP(8 + 8 * round);
    unsigned njv;
    asm volatile("ds_read_b32 %0, %1\n\ts_waitcnt lgkmcnt(0)" : "=&v"(njv) : "v"((unsigned)(size_t)lds + STAGE_BYTES) : "memory");
    return __builtin_amdgcn_readfirstlane((int)njv) + G_WGS;
}

__global__ __launch_bounds__(512, 2) void gemm_ar_kernel(const Params P) {
    __shared__ __attribute__((aligned(16))) unsigned char shm[STAGE_BYTES + 64];
    LAS unsigned char* lds = (LAS unsigned char*)shm;
    const int tid = threadIdx.x, wid = __builtin_amdgcn_readfirstlane(tid >> 6), lane = tid & 63;
    const int grp = P.g, a1 = P.a >> 1, a0 = P.a & 1;
    const int xl = blockIdx.x & 7, slot = blockIdx.x >> 3;
    char* myws = (char*)P.ws[3];
    unsigned* ctl = (unsigned*)myws;
    unsigned* tmo = ctl + L_TMO * 32;
    STAMP(0);

    if (slot < CE_WGS) {
        int e = (int)((unsigned)__builtin_amdgcn_s_getreg(20 | (3 << 11)) & 7u);
        const bool broker = blockIdx.x == 0 && wid == 6;
        bool active = (wid < K_CW || e == 3) && !broker;
        const __amdgpu_buffer_rsrc_t rl = __builtin_amdgcn_make_buffer_rsrc(myws, 0, (int)WS_END, 0x00020000);
        const unsigned l0 = lane == 0 ? 0u : 0x7fffff00u;
#define ATOM(rs, off, v, aux) (unsigned)__builtin_amdgcn_raw_ptr_buffer_atomic_add_i32((int)(v), rs, l0, off, aux)
#define ATOMV(rs, off, v) (unsigned)__builtin_amdgcn_raw_ptr_buffer_atomic_add_i32((int)(v), rs, l0 | (off), 0, 16)
#define UNI(x) (unsigned)__builtin_amdgcn_readfirstlane((int)(x))
        if (wid == 7) {
            if (slot == 0 && xl != 3) (void)ATOM(__builtin_amdgcn_make_buffer_rsrc(P.ws[xl], 0, (int)CTL_BYTES, 0x00020000), (L_START + xl) * 128, 1, 16);
            const unsigned long long t0 = __builtin_amdgcn_s_memrealtime();
            while (__builtin_amdgcn_s_memrealtime() - t0 < WATCH_DELAY) __builtin_amdgcn_s_sleep(16);
            if (UNI(ld_relaxed_agent(ctl + (L_HEAD + xl * 2) * 32)) == 0u) { e = xl; active = true; }
        }
        const __amdgpu_buffer_rsrc_t rq = __builtin_amdgcn_make_buffer_rsrc(P.ws[e], 0, (int)WS_END, 0x00020000);
        const __amdgpu_buffer_rsrc_t rqo = __builtin_amdgcn_make_buffer_rsrc(P.out[e], 0, (int)OUT_BYTES, 0x00020000);
        const unsigned llane = (unsigned)lane * 16u;
        const unsigned rlane = (unsigned)(lane >> 5) * 16384u + (unsigned)(lane & 31) * 16u;
        const int head = (L_HEAD + e * 2) * 128, scnt = (L_SCNT + e * 8) * 128;
        const unsigned cold = (unsigned)(L_START + e) * 128u;
        unsigned tkv = 0, call = 0;
        if (active) tkv = ATOM(rl, head, e == 3 ? 1 : 2, 0);
        if (active || broker) call = UNI(__builtin_amdgcn_raw_buffer_load_b32(rl, L_CALL * 128u, 0, 16)) + 1u;
        STAMP(4);
        const unsigned tag = call % 4095u + 1u;
        bool ok = true;
        unsigned idle = 0; unsigned long long t0 = 0;
        if (active && e == 3) {
            for (unsigned it = UNI(tkv); it < (unsigned)Q3L_ITEMS; it = UNI(tkv)) {
                const unsigned tl = TAB(T_ORD + (it >> 2)), st = (it & 3u) * 4u, dep = (unsigned)(L_TILE + tl) * 128u;
                t0 = __builtin_amdgcn_s_memrealtime();
                for (idle = 1; ok; ++idle) {
                    const unsigned v = __builtin_amdgcn_raw_buffer_load_b32(rl, dep + (unsigned)(lane & 1) * 64u, 0, 17);
                    if (__all((int)(v - ((lane & 1) ? 16u : 8u) * call) >= 0)) break;
                    if ((idle & 63u) == 0u && gave_up(t0, tmo, call)) ok = false;
                    __builtin_amdgcn_s_sleep(K_SLEEP);
                }
                __builtin_amdgcn_fence(__ATOMIC_ACQUIRE, "wavefront");
                PROG(2, it);
                const unsigned so = tl * 131072u + st * 8192u + llane;
                const unsigned dO = (panel_base(grp, a1, (int)(tl >> 4)) + st * 16u) * 16384u + panel_base(grp, a0, (int)(tl & 15u)) * 2u + rlane;
                for (int sub = 0; sub < 4; ++sub) {
                    const unsigned sv = so + (unsigned)sub * 8192u, dv = dO + (unsigned)sub * 262144u;
                    u32x4 va[8], vb[8];
#pragma unroll
                    for (int k = 0; k < 8; ++k) { va[k] = __builtin_amdgcn_raw_buffer_load_b128(rl, PT_OFF + sv + k * 1024u, 0, 16); vb[k] = __builtin_amdgcn_raw_buffer_load_b128(rl, RT_OFF + sv + k * 1024u, 0, 16); }
#pragma unroll
                    for (int k = 0; k < 8; ++k) {
                        u32x4 o = {add_bf16x2(va[k].x, vb[k].x), add_bf16x2(va[k].y, vb[k].y), add_bf16x2(va[k].z, vb[k].z), add_bf16x2(va[k].w, vb[k].w)};
                        if (!ok) o = (u32x4){0x7fc07fc0u, 0x7fc07fc0u, 0x7fc07fc0u, 0x7fc07fc0u};
                        __builtin_amdgcn_raw_buffer_store_b128(o, rl, FT_OFF + sv + k * 1024u, 0, 16);
                        __builtin_amdgcn_raw_buffer_store_b128(o, rqo, dv + k * 32768u, 0, 16);
                    }
                }
                tkv = ATOM(rl, head, 1, 0);
                asm volatile("s_waitcnt vmcnt(0)" ::: "memory");
                if ((UNI(ATOM(rl, scnt + (int)(it >> 2) * 4, 1, 0)) & 3u) == 3u) {
                    (void)ATOM(rl, (int)dep + 32, 4, 0);
                    if ((UNI(ATOM(rl, L_NFIN * 128, 1, 0)) & 255u) == 255u) (void)ATOM(rl, L_ALLFIN * 128, 1, 0);
                }
            }
        } else if (active) {
            const int d = e + 1, c = (e - 4) & 3;
            const int nX = (d & 2) ? (d == 3 ? NH : ND) : 64, nW = (d & 1) ? (d == 3 ? NH : ND) : 64;
            const unsigned n1 = e < 3 ? (unsigned)((TR / 2) * (nX + nW) + (32 - TR) * 64) : (unsigned)Q1X_ITEMS;
            const unsigned n13 = n1 + (e < 3 ? (unsigned)Q3I_ITEMS : (unsigned)Q3X_ITEMS);
            bool more = true, spare = true, shk = false, markp = false;
            int tks = 2, tk2s = 0;
            unsigned tk = UNI(tkv), tk2 = 0, sh0 = 0, sh1 = 0, sh2 = 0, sh3 = 0;
            bool q2 = e > 3, allfin = false, started = false;
            int nk = 0; unsigned nso = 0, ndo = 0, ndep = cold, nneed = 0, nsig = 0, nseg = 0, nsegn = 1; bool nrdy = false, nfin = false;
            bool h2 = false, r2 = false; unsigned p2 = 0, tl2 = 0;
            unsigned psig = 0, pseg = 0, psegn = 1;
            unsigned mine = 0; bool pfin = false;
            const unsigned lm0 = lane == 0 ? ~0u : 0u, lm1 = lane == 1 ? ~0u : 0u, lm2 = lane == 2 ? ~0u : 0u, lm3 = lane == 3 ? ~0u : 0u, lm4 = lane == 4 ? ~0u : 0u, lm5 = lane == 5 ? ~0u : 0u, lms = (lane >> 2) == 2 ? ~0u : 0u, lmx = (lane > 5 && (lane >> 2) != 2) ? 0x7fff0000u : 0u;
            const unsigned pneed = (lm1 & (16u * call)) | (lm2 & (8u * call)) | (lm4 & (4u * call)) | ((lm3 | lm5) & call);
            for (;;) {
                int ck = 0; unsigned cso = 0, cdo = 0, csig = 0, cseg = 0, csegn = 1; bool cfin = false;
#pragma nounroll
                for (int pass = 0; pass < 2; ++pass) {
                    if (nk == 0 && tks == 2 && (tk < n13 || shk)) {
                        tks = 0; nneed = 0; ndep = cold; nfin = false; nsig = 0; nseg = 0;
                        if (tk == (unsigned)(e < 3 ? MARK_AT : MARK_AT - Q2_ITEMS)) markp = true;
                        if (tk < n1) {
                            const unsigned it = tk;
                            int t, u, dd; bool first = false, second = false; int cp = 0, m = 0;
                            if (e < 3) {
                                dd = d;
                                const unsigned nrel = (unsigned)((TR / 2) * (nX + nW));
                                if (it < nrel) { const unsigned pr = d == 3 ? it / (unsigned)(2 * NH) : it / (unsigned)(64 + ND), rem = it - pr * (unsigned)(nX + nW);
                                    if (rem < (unsigned)nX) { t = 2 * (int)pr; u = 64 - nX + (int)rem; } else { t = 2 * (int)pr + 1; u = 64 - nW + (int)rem - nX; } }
                                else { t = TR + (int)((it - nrel) >> 6); u = (int)((it - nrel) & 63u); }
                                nseg = (unsigned)t; nsegn = t >= TR ? 64u : (unsigned)((t & 1) ? nW : nX);
                            } else {
                                int r;
                                if (it < (unsigned)(LEAD * NF1)) { t = (int)it / NF1; r = (int)it - t * NF1; first = true; }
                                else { const unsigned i2 = it - (unsigned)(LEAD * NF1);
                                    if (i2 < (unsigned)((TR - LEAD) * (NF1 + NS2))) { const int sl = (int)(i2 / (unsigned)(NF1 + NS2)); r = (int)i2 - sl * (NF1 + NS2);
                                        if (r < NF1) { t = sl + LEAD; first = true; } else { t = sl; r -= NF1; } }
                                    else { const unsigned i3 = i2 - (unsigned)((TR - LEAD) * (NF1 + NS2)); const int sl = (int)(i3 / (unsigned)NS2); t = TR - LEAD + sl; r = (int)i3 - sl * NS2; } }
                                second = !first;
                                if (first) { m = r; cp = c; dd = 3; }
                                else { if (r < NF1) { m = r; dd = (t & 1) ? 1 : 2; } else { m = r - NF1; dd = 3; } cp = c ^ dd; }
                                u = 4 * m + cp;
                                nseg = (unsigned)(t + (first ? 0 : (dd == 3 ? 64 : 32))); nsegn = (second && dd == 3) ? (unsigned)(NF / 4) : (unsigned)NF1;
                            }
                            const bool isw = t & 1; const int I = t >> 1;
                            const unsigned rel = RL_OFF + (unsigned)((cp * 32 + t) * 16 + m) * 8192u;
                            if (second) { nk = 3; nso = rel; ndep = (unsigned)(L_RELAY + cp * 32 + t) * 128u; nneed = (unsigned)NF1 * call; }
                            else if (isw) { nk = 2; nso = (unsigned)u * 16u * 16384u + panel_base(grp, a0 ^ (dd & 1), I) * 2u; }
                            else { nk = 1; nso = panel_base(grp, a1 ^ (dd >> 1), I) * 2048u + (unsigned)u * 8192u; }
                            if (first) { ndo = rel; nsig = (unsigned)(L_RELAY + cp * 32 + t) * 128u; }
                            else { ndo = (isw ? WS_OFF : XS_OFF) + (unsigned)(dd - 1) * XS_SLOT + (unsigned)I * 524288u + (unsigned)u * 8192u; nsig = (unsigned)(L_PANEL + (dd - 1) * 32 + t) * 128u; }
                            nrdy = !second || !ok;
                        } else {
                            const unsigned it = tk - n1;
                            unsigned tl, st; bool none = false;
                            if (tk >= n13) {
                                unsigned r = tk - n13, ct = 0, w = sh0;
                                if (r >= (w & 1023u)) { r -= w & 1023u; ct = 1; w = sh1;
                                    if (r >= (w & 1023u)) { r -= w & 1023u; ct = 2; w = sh2;
                                        if (r >= (w & 1023u)) { r -= w & 1023u; ct = 3; w = sh3; if (r >= (w & 1023u)) { none = true; r = 0; w = 0; } } } }
                                r += (w >> 10) & 1023u;
                                tl = (unsigned)T_POOL + ct * 40u + (r % (unsigned)NP); st = r / (unsigned)NP;
                            }
                            else if (e < 3) { const unsigned k = it / (unsigned)MS; tl = T_ORDF + N0 + k; st = it - k * (unsigned)MS; }
                            else {
                                constexpr unsigned B0 = (N0 * 3 / 4) * 16;
                                if (it < B0) { tl = it >> 4; st = it & 15u; }
                                else { const unsigned i2 = it - B0, k = i2 / (unsigned)(16 - MS); tl = N0 * 3 / 4 + k; st = (unsigned)MS + i2 - k * (unsigned)(16 - MS); }
                                tl += (unsigned)(T_ORD3 + c * 192);
                            }
                            tl = TAB(tl);
                            nk = 4; ndep = (unsigned)(L_TILE + tl) * 128u; nneed = 8u * call;
                            nso = tl * 131072u + st * 8192u;
                            ndo = (panel_base(grp, a1, (int)(tl >> 4)) + st * 16u) * 16384u + panel_base(grp, a0, (int)(tl & 15u)) * 2u;
                            nrdy = allfin || !ok; nfin = allfin;
                            if (none) { nk = 0; more = false; spare = false; STAMP(29); }
                        }
                        if (spare) { spare = false; ++tk; tks = 2; }
                    }
                    if (!h2 && tk2s == 2) { tk2s = 0; h2 = true; p2 = tk2; tl2 = TAB(T_ORD2 + c * 64 + (p2 >> 4)); r2 = !ok; }
                    if (pass == 0) {
                        if (h2 && r2) {
                            const unsigned o = tl2 * 131072u + (p2 & 15u) * 8192u;
                            ck = 3; cso = PT_OFF + o; cdo = RT_OFF + o; csig = (unsigned)(L_TILE + tl2) * 128u + 64u; cseg = 128u + (p2 >> 4); csegn = 16u; h2 = false;
                            PROG(1, p2);
                        } else if (nk && nrdy) {
                            ck = nk; cso = nso; cdo = ndo; csig = nsig; cseg = nseg & 255u; csegn = nsegn; cfin = nfin && ok; nk = 0;
                            PROG(0, nseg >> 8);
                        }
                    }
                }
                if (ck == 0 && nk == 0 && !more && !h2 && tk2s == 0 && !q2 && psig == 0u && !pfin && !markp) break;
                __builtin_amdgcn_fence(__ATOMIC_ACQUIRE, "wavefront");
                constexpr unsigned OOR = 0x7fff0000u;
                const bool add2 = ck == 4 && !cfin;
                if (ck) idle = 0;
                else {
                    if (idle == 0u) t0 = __builtin_amdgcn_s_memrealtime(); else __builtin_amdgcn_s_sleep(K_SLEEP);
                    if ((++idle & 63u) == 0u && ok && gave_up(t0, tmo, call)) { ok = false; nrdy = true; r2 = true; shk = true; }
                }
                u32x4 va[8], vb[8];
                {
                    const __amdgpu_buffer_rsrc_t rs = __builtin_amdgcn_make_buffer_rsrc(ck == 1 ? (void*)P.x : ck == 2 ? (void*)P.w : (void*)myws, 0, ck >= 3 ? (int)WS_END : (int)IN_BYTES, 0x00020000);
                    const unsigned sv = (ck ? cso + (ck == 4 ? (cfin ? FT_OFF : PT_OFF) : 0u) : OOR) + (ck == 2 ? rlane : llane), ss = ck == 2 ? 32768u : 1024u;
                    const unsigned sb = (add2 ? RT_OFF + cso : OOR) + llane;
#pragma unroll
                    for (int k = 0; k < 8; ++k) va[k] = __builtin_amdgcn_raw_buffer_load_b128(rs, sv + k * ss, 0, 16  );
#pragma unroll
                    for (int k = 0; k < 8; ++k) vb[k] = __builtin_amdgcn_raw_buffer_load_b128(rl, sb + k * 1024u, 0, 16);
                }
                const bool cl_own = tks == 0 && more, cl2 = q2 && tk2s == 0 && tk >= (unsigned)Q2_AFTER, ws = !shk && tk + 64u >= n13;
                const unsigned tkv1 = ATOMV(rl, cl_own ? (unsigned)head : OOR, 1);
                const unsigned tkv2 = ATOMV(rl, cl2 ? (unsigned)head + 128u : OOR, 1);
                const bool nw = nk && !(nrdy && (nfin || nk != 4)), w2 = h2 && !r2, f4 = nw && nk == 4;
                const unsigned poff = (lm0 & (nw ? ndep : OOR)) | (lm1 & (f4 ? ndep + 64u : OOR)) | (lm2 & (w2 ? (unsigned)(L_TILE + tl2) * 128u : OOR)) | (lm3 & ((w2 && !started) ? cold : OOR)) |
                                      (lm4 & (f4 ? ndep + 32u : OOR)) | (lm5 & (f4 ? (unsigned)(L_ALLFIN * 128) : OOR)) | (lms & (ws ? (unsigned)(L_SHARE + e) * 128u + (unsigned)(lane & 3) * 4u : OOR)) | lmx;
                const unsigned pv = __builtin_amdgcn_raw_buffer_load_b32(rl, poff, 0, 17);
                __builtin_amdgcn_sched_barrier(0);
                const unsigned okm = (unsigned)__ballot((int)(pv - ((lm0 & nneed) | pneed)) >= 0);
                if (okm & 32u) allfin = true;
                if (okm & 8u) started = true;
                if (w2 && started && (okm & 4u)) r2 = true;
                if (nw && (okm & 1u) && (!f4 || (okm & 2u))) { nrdy = true; nfin = K_FT && (okm & 16u); }
                if (allfin && nk == 4) nrdy = nfin = true;
                if (ws && (((unsigned)__ballot((pv >> 20) == tag) >> 8) & 15u) == 15u) {
                    shk = true; sh0 = (unsigned)__builtin_amdgcn_readlane((int)pv, 8); sh1 = (unsigned)__builtin_amdgcn_readlane((int)pv, 9);
                    sh2 = (unsigned)__builtin_amdgcn_readlane((int)pv, 10); sh3 = (unsigned)__builtin_amdgcn_readlane((int)pv, 11);
                }
                if (cl_own) { tk = UNI(tkv1); tks = 2; }
                if (cl2) { tk2 = UNI(tkv2); tk2s = 2; if (tk2 >= (unsigned)Q2_ITEMS) { tk2s = 0; q2 = false; STAMP(2); } }
                unsigned one = 1u;
                asm volatile("" : "+v"(one) : "v"(pv) : "memory");
                const unsigned qv = ATOMV(rl, psig ? (unsigned)scnt + pseg * 4u : OOR, one);
                const unsigned qsig = psig, qsegn = psegn;
                if (pfin) ++mine;
                if (add2) {
#pragma unroll
                    for (int k = 0; k < 8; ++k) {
                        va[k] = (u32x4){add_bf16x2(va[k].x, vb[k].x), add_bf16x2(va[k].y, vb[k].y), add_bf16x2(va[k].z, vb[k].z), add_bf16x2(va[k].w, vb[k].w)};
                        if (!ok) va[k] = (u32x4){0x7fc07fc0u, 0x7fc07fc0u, 0x7fc07fc0u, 0x7fc07fc0u};
                    }
                }
                {
                    const __amdgpu_buffer_rsrc_t rd = __builtin_amdgcn_make_buffer_rsrc(ck == 4 ? P.out[e] : P.ws[e], 0, ck == 4 ? (int)OUT_BYTES : (int)WS_END, 0x00020000);
                    const unsigned dv = (ck ? cdo : OOR) + (ck == 4 ? rlane : llane), ds = ck == 4 ? 32768u : 1024u;
#pragma unroll
                    for (int k = 0; k < 8; ++k) __builtin_amdgcn_raw_buffer_store_b128(va[k], rd, dv + k * ds, 0, 17  );
                }
                pfin = ck == 4; psig = csig; pseg = cseg; psegn = csegn;
                const bool fire = qsig && UNI(qv) == qsegn * call - 1u;
                (void)ATOMV(rq, fire ? qsig : markp ? (unsigned)(L_MARK + e) * 128u : OOR, (fire ? qsegn : 1u) * one);
                if (!fire) markp = false;
            }
            if (mine) (void)ATOM(rq, (L_DONE + e) * 128, mine, 16);
        }
        if (broker) {
            const unsigned long long tw = __builtin_amdgcn_s_memrealtime();
            unsigned seen_at = 0, first = 0; bool seen = (lane & 7) == 3;
            for (unsigned polls = 1; ok; ++polls) {
                const unsigned v = ld_relaxed_sys(ctl + (L_MARK + (lane & 7)) * 32), now = (unsigned)__builtin_amdgcn_s_memrealtime() | 1u;
                if (!seen) seen_at = now;
                seen = seen || (int)(v - call) >= 0;
                const unsigned sm = (unsigned)__ballot(seen) & 255u;
                if (sm != 8u && first == 0u) first = now;
                if (sm == 255u || (first && now - first > 2200u)) break;
                if ((polls & 63u) == 0u && gave_up(tw, tmo, call)) ok = false;
                __builtin_amdgcn_s_sleep(8);
            }
            float lag[8], mean = 0.f;
#pragma unroll
            for (int k = 0; k < 8; ++k) { lag[k] = (float)((unsigned)__builtin_amdgcn_readlane((int)seen_at, k) - first) * 0.0715f + (float)(k < 2 ? FIX_N : k == 2 ? FIX_D : FIX_X); if (k != 3) mean += lag[k]; }
            mean = mean * (1.f / 7.f) + (float)(12 * POOL_ITEMS) / 7.f;
            const int pl = lane % 12, pu = pl >> 2, pvv = pl & 3;
            const float su = mean - (pu == 0 ? lag[0] : pu == 1 ? lag[1] : lag[2]), sv = mean - (pvv == 0 ? lag[4] : pvv == 1 ? lag[5] : pvv == 2 ? lag[6] : lag[7]);
            const float S = 3.f * mean - lag[0] - lag[1] - lag[2];
            int x = (int)(su * 0.25f + ((float)POOL_ITEMS - sv * (1.f / 3.f)) - S * (1.f / 12.f) + 0.5f);
            x = x < 0 ? 0 : x > POOL_ITEMS ? POOL_ITEMS : x;
            if (lane < 28) {
                const int se = lane < 12 ? pu : lane < 24 ? 4 + pvv : 4 + (lane & 3), ct = lane < 24 ? (pu + 1) ^ pvv : (lane & 3);
                const unsigned w = tag << 20 | (lane < 12 ? (unsigned)x : lane < 24 ? (unsigned)x << 10 | (unsigned)(POOL_ITEMS - x) : 0u);
                __hip_atomic_store((unsigned*)P.ws[se] + (L_SHARE + se) * 32 + ct, w, __ATOMIC_RELAXED, __HIP_MEMORY_SCOPE_SYSTEM);
            }
        }
        STAMP(3);
        __syncthreads();
        wg_exit(ctl, call, wid, lane);
        if (broker) {
            const unsigned long long tw = __builtin_amdgcn_s_memrealtime();
            for (unsigned polls = 1; ok; ++polls) {
                const int v = (int)ld_relaxed_sys(ctl + (L_DONE + (lane & 7)) * 32);
                unsigned sum = 0;
#pragma unroll
                for (int k = 0; k < 8; ++k) sum += (unsigned)__builtin_amdgcn_readlane(v, k);
                if ((int)(sum - 12288u * call) >= 0) break;
                if ((polls & 63u) == 0u && gave_up(tw, tmo, call)) ok = false;
                __builtin_amdgcn_s_sleep(6);
            }
        }
        if (!ok && lane == 0) ((unsigned short*)P.out[3])[(size_t)panel_base(grp, a1, 0) * GN + panel_base(grp, a0, 0)] = 0x7fc0;
        return;
    }

    const unsigned call = (unsigned)__builtin_amdgcn_readfirstlane(ld_relaxed_agent(ctl + L_CALL * 32)) + 1u;
    const int njobs = (int)P.njob[xl];
    for (int job = slot - CE_WGS, round = 0; job < njobs; ++round) {
        const unsigned jw = (P.job[(xl * 64 + job) >> 1] >> ((job & 1) * 16)) & 0xffffu;
        if (jw >> 8) job = gemm_job<true>(P, lds, myws, call, (int)(jw & 255u), (int)(jw >> 8) & 3, xl, round);
        else job = gemm_job<false>(P, lds, myws, call, (int)jw, 0, xl, round);
    }
    wg_exit(ctl, call, wid, lane);
}

}

extern "C" void kernel_launch(int rank, int nranks, ncclComm_t comm,
                              hipStream_t stream,
                              void* const* d_in, const int* in_sizes, int n_in,
                              void* d_out, int out_size,
                              const int* in_dtypes, int out_dtype,
                              void* const* d_out_all,
                              void* const* d_ws_all, size_t ws_bytes) {
    if (nranks != NR || (unsigned)rank >= (unsigned)NR || n_in < 2 || ws_bytes < WS_END || (size_t)out_size != (size_t)GM * GN ||
        (size_t)in_sizes[0] != (size_t)GM * KS || (size_t)in_sizes[1] != (size_t)KS * GN) {
        fprintf(stderr, "kernel_launch: unexpected problem shape\n");
        return;
    }
    const int member[2][4] = {{0, 2, 3, 6}, {1, 4, 5, 7}};
    Params P;
    P.x = d_in[0]; P.w = d_in[1]; P.g = 0; P.a = 0;
    for (int g = 0; g < 2; ++g) for (int a = 0; a < 4; ++a) if (member[g][a] == rank) { P.g = g; P.a = a; }
    for (int e = 0; e < 8; ++e) {
        const int peer = e < 3 ? member[P.g][P.a ^ (e + 1)] : (e == 3 ? rank : member[1 - P.g][P.a ^ (e - 4)]);
        P.out[e] = d_out_all[peer]; P.ws[e] = d_ws_all[peer];
    }
    unsigned char* tb = (unsigned char*)P.tab;
    int n = 0, n2[4] = {0, 0, 0, 0}, n3[4] = {0, 0, 0, 0}, nx[8] = {0, 0, 0, 0, 0, 0, 0, 0};
    for (int k = 0; k < 16; ++k)
        for (int r = 0; r < 2 * k + 1; ++r) {
            const int I = r < k ? k : r - k, J = r < k ? r : k;
            const unsigned char tl = (unsigned char)(I << 4 | J);
            const int c = (I & 1) << 1 | (J & 1), x = (I & 3) << 1 | ((J >> 1) & 1);
            tb[T_ORDF + n] = tl; tb[T_ORD + n++] = tl; tb[T_ORD2 + c * 64 + n2[c]++] = tl;
            unsigned short* jb = (unsigned short*)P.job;
            if (n <= K_SMALL) { const int xs = (n - 1) & 7; for (int q = 0; q < 4; ++q) jb[xs * 64 + nx[xs]++] = (unsigned short)(tl | (4 + q) << 8); }
            else jb[x * 64 + nx[x]++] = tl;
        }
    for (int x = 0; x < 8; ++x) P.njob[x] = (unsigned)nx[x];
    {
        const int shell[] = K_SHELLS;
        unsigned rng = 12345u;
        for (unsigned si = 0; si + 1 < sizeof(shell) / sizeof(shell[0]); ++si) {
            const int lo = shell[si] * shell[si], hi = shell[si + 1] * shell[si + 1];
            for (int i = hi - 1; i > lo; --i) { rng = rng * 1664525u + 1013904223u; const int j = lo + (int)((rng >> 8) % (unsigned)(i - lo + 1)); const unsigned char t = tb[T_ORDF + i]; tb[T_ORDF + i] = tb[T_ORDF + j]; tb[T_ORDF + j] = t; }
        }
        int seen[4] = {0, 0, 0, 0}, ns = 0;
        for (int i = 0; i < NTILE; ++i) {
            const unsigned char tl = tb[T_ORDF + i]; const int c = ((tl >> 4) & 1) << 1 | (tl & 1);
            if (seen[c] >= 64 - NP) { tb[T_POOL + c * 40 + seen[c]++ - (64 - NP)] = tl; continue; }
            ++seen[c]; tb[T_ORDF + ns++] = tl;
            for (int o = 0; o < 4; ++o) if (o != c) tb[T_ORD3 + o * 192 + n3[o]++] = tl;
        }
    }
    hipLaunchKernelGGL(gemm_ar_kernel, dim3(GRID), dim3(512), 0, stream, P);
    HIP_CHECK(hipGetLastError());
}
```

```cpp
#include <hip/hip_runtime.h>
#include <rccl/rccl.h>
#include <cstdio>

#define HIP_CHECK(expr) do { hipError_t _e = (expr); if (_e != hipSuccess) { \
    fprintf(stderr, "HIP error %s at %s:%d\n", hipGetErrorString(_e), __FILE__, __LINE__); \
    return; } } while (0)

namespace {

#define LAS __attribute__((address_space(3)))
typedef short bf16x8 __attribute__((ext_vector_type(8)));
typedef short s16x4 __attribute__((ext_vector_type(4)));
typedef float f32x4 __attribute__((ext_vector_type(4)));
typedef unsigned u32x4 __attribute__((ext_vector_type(4)));

constexpr int NR = 8;
constexpr int GM = 8192, GN = 8192;
constexpr int KS = 1024;
constexpr int NTILE = 256;

#define K_TR 10
#define K_NF 56
#define K_NG 0
#define K_LEAD 4
#define K_N0 36
#define K_M 8
#define K_NP 16
#define K_SMALL 64
#define K_LDAUX 17
#define K_DSLEEP 2
#define K_MARK 200
#define K_SHELLS {6, 7, 8, 10, 12, 14, 16}
#define K_SCR 1
#define K_CW 1
#define K_FT 1
#define K_SLEEP 8
#define K_CE 12
constexpr int TR = K_TR, NF = K_NF, NG = K_NG, LEAD = K_LEAD, N0 = K_N0, MS = K_M, NP = K_NP, NS = 256 - 4 * NP;
static_assert(TR % 2 == 0 && TR <= 32 && NF % 4 == 0 && NG % 4 == 0 && NF + NG <= 64 && N0 % 4 == 0 && MS >= 1 && MS < 16 && NP >= 1 && NP <= 39, "knobs");
constexpr int NF1 = (NF + NG) / 4;
constexpr int NS2 = NF1 + NF / 4;
constexpr int ND = 64 - NF - NG, NH = 64 - NF;

constexpr unsigned CTL_BYTES = 1u << 20;
constexpr unsigned XS_OFF = CTL_BYTES, XS_SLOT = 16u * 524288u;
constexpr unsigned WS_OFF = XS_OFF + 3u * XS_SLOT;
constexpr unsigned RL_OFF = WS_OFF + 3u * XS_SLOT;
constexpr unsigned PT_OFF = RL_OFF + 4u * 32u * 16u * 8192u;
constexpr unsigned RT_OFF = PT_OFF + 256u * 131072u;
constexpr unsigned FT_OFF = RT_OFF + 256u * 131072u;
constexpr unsigned WS_END = FT_OFF + 256u * 131072u;
constexpr unsigned OUT_BYTES = (unsigned)((size_t)GM * GN * 2);
constexpr unsigned IN_BYTES = (unsigned)((size_t)GM * KS * 2);

constexpr int L_TMO = 1, L_EXIT = 2, L_CALL = 3  , L_NFIN = 4, L_ALLFIN = 5, L_DONE = 24  , L_START = 32  ,
              L_HEAD = 64  , L_MARK = 40  , L_SHARE = 48  , L_PANEL = 128  , L_RELAY = 256  , L_TILE = 512  ,
              L_SCNT = 1024  ;
constexpr int CE_WGS = K_CE;
constexpr int G_WGS = 32 - CE_WGS;
constexpr int GRID = 256;
constexpr unsigned long long DEADLINE_TICKS = 50000000ull;
constexpr unsigned long long WATCH_DELAY = 1000;

constexpr int Q1X_ITEMS = TR * (NF1 + NS2);
constexpr int Q2_ITEMS = 64 * 16;
constexpr int Q2_AFTER = 128;
constexpr int Q3I_ITEMS = (NS - N0) * MS;
constexpr int Q3X_ITEMS = (N0 * 3 / 4) * 16 + ((NS - N0) * 3 / 4) * (16 - MS);
constexpr int POOL_ITEMS = NP * 16;
constexpr int FIX_N = (TR / 2) * (64 + ND) + (32 - TR) * 64 + Q3I_ITEMS, FIX_D = TR * NH + (32 - TR) * 64 + Q3I_ITEMS, FIX_X = Q1X_ITEMS + Q2_ITEMS + Q3X_ITEMS;
constexpr int MARK_AT = (FIX_N < FIX_D ? (FIX_N < FIX_X ? FIX_N : FIX_X) : (FIX_D < FIX_X ? FIX_D : FIX_X)) - K_MARK;
constexpr int Q3L_ITEMS = NTILE * 4;

constexpr int BK = 64, HALF = 128, HTB = HALF * BK * 2, STAGE_BYTES = 8 * HTB;

#define STAMP(i) do {} while (0)
#define PROG(q, it) do {} while (0)

constexpr int MAXJ = 96;
static_assert((K_SMALL + 7) / 8 * 4 + 32 <= MAXJ, "job table");
struct Params {
    const void* x; const void* w;
    void* out[8]; void* ws[8];
    int g, a;
    unsigned njob[8], job[8 * MAXJ / 2];
    unsigned tab[424];
};
constexpr int T_ORD = 0  , T_ORD2 = 256  , T_ORD3 = 512  ,
              T_ORDF = 1280  ,
              T_POOL = 1536  ;
#define TAB(i) ((P.tab[(unsigned)(i) >> 2] >> (((unsigned)(i) & 3u) * 8u)) & 255u)

__device__ __forceinline__ int lds_byte(int r, int c) { const int st = (r >> 4) * 2 + (c >> 5), rr = r & 15, cc = c & 31, ob = rr * 64 + cc * 2; return st * 1024 + (ob ^ (((ob >> 9) & 1) << 5)); }
__device__ __forceinline__ void stage_rc(int b, int& R, int& C) { const int st = b / 1024, sb = b % 1024, swz = sb ^ (((sb >> 9) & 1) << 5); R = (st >> 1) * 16 + swz / 64; C = (st & 1) * 32 + (swz % 64) / 2; }
__device__ __forceinline__ unsigned wswz(unsigned row) { return ((row & 3u) << 2) | ((row >> 2) & 3u); }
__device__ __forceinline__ unsigned cvt_pk_bf16(float lo, float hi) { unsigned r; asm volatile("v_cvt_pk_bf16_f32 %0, %1, %2" : "=v"(r) : "v"(lo), "v"(hi)); return r; }
__device__ __forceinline__ unsigned panel_base(int g, int ab, int I) {
    const int bb = ab ^ (I & 1), q = g == 0 ? 2 * ab + bb : 2 * bb + ab, j = I >> 1;
    return (unsigned)((j << 2) | ((q + (0xb114 >> (2 * j))) & 3)) * 256u;
}

__device__ __forceinline__ unsigned ld_relaxed_sys(const unsigned* p) { return __hip_atomic_load(p, __ATOMIC_RELAXED, __HIP_MEMORY_SCOPE_SYSTEM); }
__device__ __forceinline__ unsigned ld_relaxed_agent(const unsigned* p) { return __hip_atomic_load(p, __ATOMIC_RELAXED, __HIP_MEMORY_SCOPE_AGENT); }

__device__ __forceinline__ bool gave_up(unsigned long long t0, unsigned* tmo, unsigned call) {
    if ((unsigned)__builtin_amdgcn_readfirstlane(ld_relaxed_agent(tmo)) == call || __builtin_amdgcn_s_memrealtime() - t0 > DEADLINE_TICKS) {
        __hip_atomic_store(tmo, call, __ATOMIC_RELAXED, __HIP_MEMORY_SCOPE_AGENT);
        return true;
    }
    return false;
}
__device__ __forceinline__ unsigned add_bf16x2(unsigned a, unsigned b) {
    return cvt_pk_bf16(__uint_as_float(a << 16) + __uint_as_float(b << 16), __uint_as_float(a & 0xffff0000u) + __uint_as_float(b & 0xffff0000u));
}

__device__ __forceinline__ void wg_exit(unsigned* ctl, unsigned call, int wid, int lane) {
    if (wid != 0) return;
    const unsigned t = (unsigned)__builtin_amdgcn_raw_ptr_buffer_atomic_add_i32(1, __builtin_amdgcn_make_buffer_rsrc(ctl, 0, (int)CTL_BYTES, 0x00020000), lane == 0 ? (unsigned)L_EXIT * 128u : 0x7fffff00u, 0, 0);
    if (((unsigned)__builtin_amdgcn_readfirstlane((int)t) & (unsigned)(GRID - 1)) == (unsigned)(GRID - 1)) {
        if (lane < 25) __hip_atomic_store(ctl + (lane == 24 ? L_CALL : L_HEAD + lane) * 32, lane == 24 ? call : 0u, __ATOMIC_RELAXED, __HIP_MEMORY_SCOPE_AGENT);
    }
}

template <bool SMALL>
__device__ __forceinline__ int gemm_job(const Params& P, LAS unsigned char* lds, char* myws, unsigned call, int tl, int qd, int xl, int round) {
    int tid = threadIdx.x; asm volatile("" : "+v"(tid));
    const int wid = __builtin_amdgcn_readfirstlane(tid >> 6), lane = tid & 63;
    const int grp = P.g, a1 = P.a >> 1, a0 = P.a & 1, ti = tl >> 4, tj = tl & 15;
    unsigned* ctl = (unsigned*)myws;
    unsigned* tmo = ctl + L_TMO * 32;
    const int wr = wid >> 2, wc = wid & 3, fr = lane & 15, fq = lane >> 4;
    unsigned voffA[2], voffB[2];
#pragma unroll
    for (int i = 0; i < 2; ++i) { int R, C; stage_rc(tid * 16 + i * 8192, R, C); voffA[i] = (unsigned)(R * KS + C) * 2u; }
#define SET_VOFFB(sh) do { int tz = tid; asm volatile("" : "+v"(tz)); _Pragma("unroll") for (int i = 0; i < 2; ++i) { \
        const unsigned row = (unsigned)(tz >> 4) + 32u * i, ch = ((unsigned)tz & 15u) ^ wswz(row); voffB[i] = (row << (sh)) + ch * 16u; } \
        kstepB = (size_t)(SMALL ? 2 * BK : BK) << (sh); hstepB = SMALL ? (size_t)BK << (sh) : (size_t)HALF * 2; } while (0)
    size_t kstepB, hstepB;
    constexpr size_t kstepA = SMALL ? BK * 4 : BK * 2, hstepA = SMALL ? BK * 2 : (size_t)HALF * KS * 2;
    const unsigned qa = SMALL ? (unsigned)(qd >> 1) * (unsigned)(HALF * KS * 2) : 0u, qb = SMALL ? (unsigned)(qd & 1) * 256u : 0u;
    const unsigned ldsw = (unsigned)wid * 1024u;
    const int aoff = lds_byte(wr * 64 + fr, fq * 8);
    int boff[2][2];
#pragma unroll
    for (int n = 0; n < 2; ++n)
#pragma unroll
        for (int t = 0; t < 2; ++t) {
            const unsigned q4 = (unsigned)(lane & 15) >> 2, p4 = (unsigned)lane & 3u, row = 8u * fq + 4u * t + q4, ch = 4u * wc + 2u * n + (p4 >> 1);
            boff[n][t] = (int)((unsigned)(size_t)lds + 4u * HTB + 256u * row + 16u * (ch ^ wswz(row)) + 8u * (p4 & 1u));
        }
#define SA(b, h) (((b) * 2 + (h)) * HTB)
#define SB(b, h) ((4 + (b) * 2 + (h)) * HTB)
#define STAGE(bufoff, gbase, voff) do { _Pragma("unroll") for (int _i = 0; _i < 2; ++_i) \
        __builtin_amdgcn_global_load_lds((const unsigned*)((const char*)(gbase) + (voff)[_i]), (LAS unsigned*)(lds + (bufoff) + ldsw + _i * 8192), 16, 0, 0); } while (0)
#define LDA(dst, b, h) do { _Pragma("unroll") for (int m = 0; m < 4; ++m) _Pragma("unroll") for (int k = 0; k < 2; ++k) dst[m][k] = *(const LAS bf16x8*)(lds + SA(b, h) + aoff + m * 2048 + k * 1024); } while (0)
#define TRR(dst, base, off) asm volatile("ds_read_b64_tr_b16 %0, %1 offset:%2" : "=&v"(dst) : "v"(base), "i"(off) : "memory")
#define LDB(raw, b, h) do { \
        TRR(raw[0], boff[0][0], SB(b, h) - 4 * HTB); TRR(raw[1], boff[0][1], SB(b, h) - 4 * HTB); TRR(raw[2], boff[0][0], SB(b, h) - 4 * HTB + 8192); TRR(raw[3], boff[0][1], SB(b, h) - 4 * HTB + 8192); \
        TRR(raw[4], boff[1][0], SB(b, h) - 4 * HTB); TRR(raw[5], boff[1][1], SB(b, h) - 4 * HTB); TRR(raw[6], boff[1][0], SB(b, h) - 4 * HTB + 8192); TRR(raw[7], boff[1][1], SB(b, h) - 4 * HTB + 8192); } while (0)
#define WAIT_B(raw, dst) do { asm volatile("s_waitcnt lgkmcnt(0)" : "+v"(raw[0]), "+v"(raw[1]), "+v"(raw[2]), "+v"(raw[3]), "+v"(raw[4]), "+v"(raw[5]), "+v"(raw[6]), "+v"(raw[7]) :: "memory"); \
        _Pragma("unroll") for (int n = 0; n < 2; ++n) _Pragma("unroll") for (int k = 0; k < 2; ++k) dst[n][k] = __builtin_shufflevector(raw[n * 4 + k * 2], raw[n * 4 + k * 2 + 1], 0, 1, 2, 3, 4, 5, 6, 7); } while (0)
#define MMA(ai, bj, At, Bt) do { if (!SMALL || (ai) == (bj)) { __builtin_amdgcn_s_setprio(1); _Pragma("unroll") for (int m = 0; m < 4; ++m) _Pragma("unroll") for (int n = 0; n < 2; ++n) _Pragma("unroll") for (int k = 0; k < 2; ++k) \
        acc[SMALL ? 0 : ai][SMALL ? 0 : bj][m][n] = __builtin_amdgcn_mfma_f32_16x16x32_bf16(Bt[n][k], At[m][k], acc[SMALL ? 0 : ai][SMALL ? 0 : bj][m][n], 0, 0, 0); __builtin_amdgcn_s_setprio(0); } } while (0)
#define WAIT_V(n) asm volatile("s_waitcnt vmcnt(" #n ")" ::: "memory")
#define WAIT_L(n) asm volatile("s_waitcnt lgkmcnt(" #n ")" ::: "memory")
#define BAR __builtin_amdgcn_s_barrier()
#define SCHED __builtin_amdgcn_sched_barrier(0)
    constexpr int NH2 = SMALL ? 1 : 2;
    f32x4 acc[NH2][NH2][4][2];
#pragma unroll
    for (int a = 0; a < NH2; ++a)
#pragma unroll
        for (int b = 0; b < NH2; ++b)
#pragma unroll
            for (int m = 0; m < 4; ++m)
#pragma unroll
                for (int n = 0; n < 2; ++n) acc[a][b][m][n] = (f32x4){0.f, 0.f, 0.f, 0.f};
    bf16x8 At[4][2], B0[2][2], B1[2][2];
    s16x4 R0[8], R1[8];
    const char* cA = (const char*)P.x + (size_t)panel_base(grp, a1, ti) * (KS * 2) + qa;
    const char* cB = (const char*)P.w + (size_t)panel_base(grp, a0, tj) * 2 + qb;
    constexpr int nt = SMALL ? KS / BK / 2 : KS / BK;
    unsigned todo = 7u;
    SET_VOFFB(14);
    STAGE(SB(0, 0), cB, voffB); STAGE(SA(0, 0), cA, voffA); STAGE(SB(0, 1), cB + hstepB, voffB); STAGE(SA(0, 1), cA + hstepA, voffA);
    if (wr == 1) BAR;
    WAIT_V(4); BAR;
    STAGE(SB(1, 0), cB + kstepB, voffB); STAGE(SA(1, 0), cA + kstepA, voffA); STAGE(SB(1, 1), cB + hstepB + kstepB, voffB);
    WAIT_V(6); BAR;
    for (int s = 0; s < 4; ++s) {
        const char* nA = cA; const char* nB = cB;
        for (int t = 0; t < nt; t += 2) {
            const bool last = (t == nt - 2);
            if (last && s < 3) {
                if (wid == 0) {
                    STAMP(1 + 2 * s + 8 * round);
                    const unsigned* word = ctl + (L_PANEL + ((lane >> 1) % 3) * 32 + ((lane & 1) ? 2 * tj + 1 : 2 * ti)) * 32;
                    const unsigned long long t0 = __builtin_amdgcn_s_memrealtime();
                    unsigned pick;
                    for (unsigned polls = 1;; ++polls) {
                        const unsigned mk = (unsigned)__ballot((int)(ld_relaxed_sys(word) - 64u * call) >= 0);
                        const unsigned both = mk & (mk >> 1);
                        pick = ((both & 1u) | ((both >> 1) & 2u) | ((both >> 2) & 4u)) & todo;
                        if (pick) break;
                        if ((polls & 63u) == 0u && gave_up(t0, tmo, call)) { pick = todo; break; }
                        __builtin_amdgcn_s_sleep(4);
                    }
                    STAMP(2 + 2 * s + 8 * round);
                    *(LAS unsigned*)(lds + STAGE_BYTES + 16) = (unsigned)__builtin_ctz(pick);
                    __builtin_amdgcn_fence(__ATOMIC_ACQUIRE, "");
                    asm volatile("s_waitcnt vmcnt(0) lgkmcnt(0)" ::: "memory");
                }
                asm volatile("" ::: "memory"); BAR; asm volatile("" ::: "memory");
                unsigned slv;
                asm volatile("ds_read_b32 %0, %1\n\ts_waitcnt lgkmcnt(0)" : "=&v"(slv) : "v"((unsigned)(size_t)lds + STAGE_BYTES + 16) : "memory");
                const unsigned sl = (unsigned)__builtin_amdgcn_readfirstlane(slv) & 3u;
                todo &= ~(1u << sl);
                nA = myws + XS_OFF + sl * XS_SLOT + (unsigned)ti * 524288u + qa;
                nB = myws + WS_OFF + sl * XS_SLOT + (unsigned)tj * 524288u + qb;
                if (s == 0) SET_VOFFB(9);
            }
            const char* a1p = cA + (size_t)(t + 1) * kstepA;
            const char* a2 = last ? nA : cA + (size_t)(t + 2) * kstepA; const char* b2 = last ? nB : cB + (size_t)(t + 2) * kstepB;
            const char* a3 = a2 + kstepA; const char* b3 = b2 + kstepB;
            LDB(R0, 0, 0); SCHED; LDA(At, 0, 0); STAGE(SA(1, 1), a1p + hstepA, voffA);
            WAIT_L(8); BAR; WAIT_B(R0, B0); MMA(0, 0, At, B0); BAR; SCHED;
            LDB(R1, 0, 1); STAGE(SB(0, 0), b2, voffB);
            BAR; WAIT_B(R1, B1); MMA(0, 1, At, B1); BAR; if (SMALL) SCHED;
            LDA(At, 0, 1); STAGE(SA(0, 0), a2, voffA);
            BAR; WAIT_L(0); MMA(1, 0, At, B0); BAR; SCHED;
            STAGE(SB(0, 1), b2 + hstepB, voffB);
            WAIT_V(6); BAR; MMA(1, 1, At, B1); BAR;
            LDB(R0, 1, 0); SCHED; LDA(At, 1, 0); STAGE(SA(0, 1), a2 + hstepA, voffA);
            WAIT_L(8); BAR; WAIT_B(R0, B0); MMA(0, 0, At, B0); BAR; SCHED;
            LDB(R1, 1, 1); STAGE(SB(1, 0), b3, voffB);
            BAR; WAIT_B(R1, B1); MMA(0, 1, At, B1); BAR; if (SMALL) SCHED;
            LDA(At, 1, 1); STAGE(SA(1, 0), a3, voffA);
            BAR; WAIT_L(0); MMA(1, 0, At, B0); BAR; SCHED;
            STAGE(SB(1, 1), b3 + hstepB, voffB);
            WAIT_V(6); BAR; MMA(1, 1, At, B1); BAR;
        }
        cA = nA; cB = nB;
    }
    WAIT_V(0);
    if (wr == 0) BAR;
    STAMP(7 + 8 * round);
    const __amdgpu_buffer_rsrc_t rc = __builtin_amdgcn_make_buffer_rsrc(myws, 0, (int)CTL_BYTES, 0x00020000);
    const unsigned nj = (unsigned)__builtin_amdgcn_raw_ptr_buffer_atomic_add_i32(1, rc, tid == 0 ? (unsigned)(L_HEAD + 16 + xl) * 128u : 0x7fffff00u, 0, 0);
    const bool bad = (unsigned)__builtin_amdgcn_readfirstlane(ld_relaxed_agent(tmo)) == call;
    const __amdgpu_buffer_rsrc_t ro = __builtin_amdgcn_make_buffer_rsrc(myws + PT_OFF + (unsigned)tl * 131072u, 0, 131072, 0x00020000);
    const unsigned obase = (unsigned)(64 * wr + fr + (SMALL ? (qd >> 1) * HALF : 0)) * 512u + (unsigned)(32 * wc + (fq & 1) * 16 + (fq >> 1) * 8) * 2u + qb;
#pragma unroll
    for (int ai = 0; ai < NH2; ++ai)
#pragma unroll
        for (int m = 0; m < 4; ++m)
#pragma unroll
            for (int bj = 0; bj < NH2; ++bj) {
                const f32x4 v0 = acc[ai][bj][m][0], v1 = acc[ai][bj][m][1];
                const unsigned x0 = cvt_pk_bf16(v0[0], v0[1]), x1 = cvt_pk_bf16(v0[2], v0[3]), y0 = cvt_pk_bf16(v1[0], v1[1]), y1 = cvt_pk_bf16(v1[2], v1[3]);
                const auto r0 = __builtin_amdgcn_permlane16_swap(x0, y0, false, false);
                const auto r1 = __builtin_amdgcn_permlane16_swap(x1, y1, false, false);
                u32x4 wv = {r0[0], r1[0], r0[1], r1[1]};
                if (bad) wv = (u32x4){0x7fc07fc0u, 0x7fc07fc0u, 0x7fc07fc0u, 0x7fc07fc0u};
                __builtin_amdgcn_raw_buffer_store_b128(wv, ro, obase + (unsigned)(ai * HALF + m * 16) * 512u + (unsigned)bj * 256u, 0, 16  );
            }
    if (wid == 0) *(LAS unsigned*)(lds + STAGE_BYTES) = (unsigned)__builtin_amdgcn_readfirstlane((int)nj);
    asm volatile("s_waitcnt vmcnt(0) lgkmcnt(0)" ::: "memory");
    BAR;
    (void)__builtin_amdgcn_raw_ptr_buffer_atomic_add_i32(SMALL ? 2 : 8, rc, tid == 0 ? (unsigned)(L_TILE + tl) * 128u : 0x7fffff00u, 0, 0);
    STAMP(8 + 8 * round);
    unsigned njv;
    asm volatile("ds_read_b32 %0, %1\n\ts_waitcnt lgkmcnt(0)" : "=&v"(njv) : "v"((unsigned)(size_t)lds + STAGE_BYTES) : "memory");
    return __builtin_amdgcn_readfirstlane((int)njv) + G_WGS;
}

__global__ __launch_bounds__(512, 2) void gemm_ar_kernel(const Params P) {
    __shared__ __attribute__((aligned(16))) unsigned char shm[STAGE_BYTES + 64];
    LAS unsigned char* lds = (LAS unsigned char*)shm;
    const int tid = threadIdx.x, wid = __builtin_amdgcn_readfirstlane(tid >> 6), lane = tid & 63;
    const int grp = P.g, a1 = P.a >> 1, a0 = P.a & 1;
    const int xl = blockIdx.x & 7, slot = blockIdx.x >> 3;
    char* myws = (char*)P.ws[3];
    unsigned* ctl = (unsigned*)myws;
    unsigned* tmo = ctl + L_TMO * 32;
    STAMP(0);

    if (slot < CE_WGS) {
        int e = (int)((unsigned)__builtin_amdgcn_s_getreg(20 | (3 << 11)) & 7u);
        const bool broker = blockIdx.x == 0 && wid == 6;
        bool active = (wid < K_CW || e == 3) && !broker;
        const __amdgpu_buffer_rsrc_t rl = __builtin_amdgcn_make_buffer_rsrc(myws, 0, (int)WS_END, 0x00020000);
        const unsigned l0 = lane == 0 ? 0u : 0x7fffff00u;
#define ATOM(rs, off, v, aux) (unsigned)__builtin_amdgcn_raw_ptr_buffer_atomic_add_i32((int)(v), rs, l0, off, aux)
#define ATOMV(rs, off, v) (unsigned)__builtin_amdgcn_raw_ptr_buffer_atomic_add_i32((int)(v), rs, l0 | (off), 0, 16)
#define UNI(x) (unsigned)__builtin_amdgcn_readfirstlane((int)(x))
        if (wid == 7) {
            if (slot == 0 && xl != 3) (void)ATOM(__builtin_amdgcn_make_buffer_rsrc(P.ws[xl], 0, (int)CTL_BYTES, 0x00020000), (L_START + xl) * 128, 1, 16);
            const unsigned long long t0 = __builtin_amdgcn_s_memrealtime();
            while (__builtin_amdgcn_s_memrealtime() - t0 < WATCH_DELAY) __builtin_amdgcn_s_sleep(16);
            if (UNI(ld_relaxed_agent(ctl + (L_HEAD + xl * 2) * 32)) == 0u) { e = xl; active = true; }
        }
        const __amdgpu_buffer_rsrc_t rq = __builtin_amdgcn_make_buffer_rsrc(P.ws[e], 0, (int)WS_END, 0x00020000);
        const __amdgpu_buffer_rsrc_t rqo = __builtin_amdgcn_make_buffer_rsrc(P.out[e], 0, (int)OUT_BYTES, 0x00020000);
        const unsigned llane = (unsigned)lane * 16u;
        const unsigned rlane = (unsigned)(lane >> 5) * 16384u + (unsigned)(lane & 31) * 16u;
        const int head = (L_HEAD + e * 2) * 128, scnt = (L_SCNT + e * 8) * 128;
        const unsigned cold = (unsigned)(L_START + e) * 128u;
        unsigned tkv = 0, call = 0;
        if (active) tkv = ATOM(rl, head, e == 3 ? 1 : 2, 0);
        if (active || broker) call = UNI(__builtin_amdgcn_raw_buffer_load_b32(rl, L_CALL * 128u, 0, 16)) + 1u;
        STAMP(4);
        const unsigned tag = call % 4095u + 1u;
        bool ok = true;
        unsigned idle = 0; unsigned long long t0 = 0;
        if (active && e == 3) {
            for (unsigned it = UNI(tkv); it < (unsigned)Q3L_ITEMS; it = UNI(tkv)) {
                const unsigned tl = TAB(T_ORD + (it >> 2)), st = (it & 3u) * 4u, dep = (unsigned)(L_TILE + tl) * 128u;
                t0 = __builtin_amdgcn_s_memrealtime();
                for (idle = 1; ok; ++idle) {
                    const unsigned v = __builtin_amdgcn_raw_buffer_load_b32(rl, dep + (unsigned)(lane & 1) * 64u, 0, 17);
                    if (__all((int)(v - ((lane & 1) ? 16u : 8u) * call) >= 0)) break;
                    if ((idle & 63u) == 0u && gave_up(t0, tmo, call)) ok = false;
                    __builtin_amdgcn_s_sleep(K_SLEEP);
                }
                __builtin_amdgcn_fence(__ATOMIC_ACQUIRE, "wavefront");
                PROG(2, it);
                const unsigned so = tl * 131072u + st * 8192u + llane;
                const unsigned dO = (panel_base(grp, a1, (int)(tl >> 4)) + st * 16u) * 16384u + panel_base(grp, a0, (int)(tl & 15u)) * 2u + rlane;
                for (int sub = 0; sub < 4; ++sub) {
                    const unsigned sv = so + (unsigned)sub * 8192u, dv = dO + (unsigned)sub * 262144u;
                    u32x4 va[8], vb[8];
#pragma unroll
                    for (int k = 0; k < 8; ++k) { va[k] = __builtin_amdgcn_raw_buffer_load_b128(rl, PT_OFF + sv + k * 1024u, 0, 16); vb[k] = __builtin_amdgcn_raw_buffer_load_b128(rl, RT_OFF + sv + k * 1024u, 0, K_LDAUX); }
#pragma unroll
                    for (int k = 0; k < 8; ++k) {
                        u32x4 o = {add_bf16x2(va[k].x, vb[k].x), add_bf16x2(va[k].y, vb[k].y), add_bf16x2(va[k].z, vb[k].z), add_bf16x2(va[k].w, vb[k].w)};
                        if (!ok) o = (u32x4){0x7fc07fc0u, 0x7fc07fc0u, 0x7fc07fc0u, 0x7fc07fc0u};
                        __builtin_amdgcn_raw_buffer_store_b128(o, rl, FT_OFF + sv + k * 1024u, 0, 16);
                        __builtin_amdgcn_raw_buffer_store_b128(o, rqo, dv + k * 32768u, 0, 16);
                    }
                }
                tkv = ATOM(rl, head, 1, 0);
                asm volatile("s_waitcnt vmcnt(0)" ::: "memory");
                if ((UNI(ATOM(rl, scnt + (int)(it >> 2) * 4, 1, 0)) & 3u) == 3u) {
                    (void)ATOM(rl, (int)dep + 32, 4, 0);
                    if ((UNI(ATOM(rl, L_NFIN * 128, 1, 0)) & 255u) == 255u) (void)ATOM(rl, L_ALLFIN * 128, 1, 0);
                }
            }
        } else if (active) {
            const int d = e + 1, c = (e - 4) & 3;
            const int nX = (d & 2) ? (d == 3 ? NH : ND) : 64, nW = (d & 1) ? (d == 3 ? NH : ND) : 64;
            const unsigned n1 = e < 3 ? (unsigned)((TR / 2) * (nX + nW) + (32 - TR) * 64) : (unsigned)Q1X_ITEMS;
            const unsigned n13 = n1 + (e < 3 ? (unsigned)Q3I_ITEMS : (unsigned)Q3X_ITEMS);
            bool more = true, spare = true, shk = false, msent = false;
            int tks = 2, tk2s = 0;
            unsigned tk = UNI(tkv), tk2 = 0, sh0 = 0, sh1 = 0, sh2 = 0, sh3 = 0;
            bool q2 = e > 3, allfin = false, started = false;
            int nk = 0; unsigned nso = 0, ndo = 0, ndep = cold, nneed = 0, nsig = 0, nseg = 0, nsegn = 1; bool nrdy = false, nfin = false;
            bool h2 = false, r2 = false; unsigned p2 = 0, tl2 = 0;
            unsigned psig = 0, pseg = 0, psegn = 1, qsig = 0, qsegn = 1, qv = 0;
            constexpr unsigned OOR = 0x7fff0000u;
            unsigned mine = 0; bool pfin = false;
            const unsigned lm0 = lane == 0 ? ~0u : 0u, lm1 = lane == 1 ? ~0u : 0u, lm2 = lane == 2 ? ~0u : 0u, lm3 = lane == 3 ? ~0u : 0u, lm4 = lane == 4 ? ~0u : 0u, lm5 = lane == 5 ? ~0u : 0u, lms = (lane >> 2) == 2 ? ~0u : 0u, lmx = (lane > 5 && (lane >> 2) != 2) ? 0x7fff0000u : 0u;
            const unsigned pneed = (lm1 & (16u * call)) | (lm2 & (8u * call)) | (lm4 & (4u * call)) | ((lm3 | lm5) & call);
            auto refill = [&]() __attribute__((always_inline)) {
                if (nk == 0 && tks == 2 && (tk < n13 || shk)) {
                    tks = 0; nneed = 0; ndep = cold; nfin = false; nsig = 0; nseg = 0;
                    if (tk < n1) {
                        const unsigned it = tk;
                        int t, u, dd; bool first = false, second = false; int cp = 0, m = 0;
                        if (e < 3) {
                            dd = d;
                            const unsigned nrel = (unsigned)((TR / 2) * (nX + nW));
                            if (it < nrel) { const unsigned pr = d == 3 ? it / (unsigned)(2 * NH) : it / (unsigned)(64 + ND), rem = it - pr * (unsigned)(nX + nW);
                                if (rem < (unsigned)nX) { t = 2 * (int)pr; u = 64 - nX + (int)rem; } else { t = 2 * (int)pr + 1; u = 64 - nW + (int)rem - nX; } }
                            else { t = TR + (int)((it - nrel) >> 6); u = (int)((it - nrel) & 63u); }
                            nseg = (unsigned)t; nsegn = t >= TR ? 64u : (unsigned)((t & 1) ? nW : nX);
                        } else {
                            int r;
                            if (it < (unsigned)(LEAD * NF1)) { t = (int)it / NF1; r = (int)it - t * NF1; first = true; }
                            else { const unsigned i2 = it - (unsigned)(LEAD * NF1);
                                if (i2 < (unsigned)((TR - LEAD) * (NF1 + NS2))) { const int sl = (int)(i2 / (unsigned)(NF1 + NS2)); r = (int)i2 - sl * (NF1 + NS2);
                                    if (r < NF1) { t = sl + LEAD; first = true; } else { t = sl; r -= NF1; } }
                                else { const unsigned i3 = i2 - (unsigned)((TR - LEAD) * (NF1 + NS2)); const int sl = (int)(i3 / (unsigned)NS2); t = TR - LEAD + sl; r = (int)i3 - sl * NS2; } }
                            second = !first;
                            if (first) { m = r; cp = c; dd = 3; }
                            else { if (r < NF1) { m = r; dd = (t & 1) ? 1 : 2; } else { m = r - NF1; dd = 3; } cp = c ^ dd; }
                            u = 4 * m + cp;
                            nseg = (unsigned)(t + (first ? 0 : (dd == 3 ? 64 : 32))); nsegn = (second && dd == 3) ? (unsigned)(NF / 4) : (unsigned)NF1;
                        }
                        const bool isw = t & 1; const int I = t >> 1;
                        const unsigned rel = RL_OFF + (unsigned)((cp * 32 + t) * 16 + m) * 8192u;
                        if (second) { nk = 3; nso = rel; ndep = (unsigned)(L_RELAY + cp * 32 + t) * 128u; nneed = (unsigned)NF1 * call; }
                        else if (isw) { nk = 2; nso = (unsigned)u * 16u * 16384u + panel_base(grp, a0 ^ (dd & 1), I) * 2u; }
                        else { nk = 1; nso = panel_base(grp, a1 ^ (dd >> 1), I) * 2048u + (unsigned)u * 8192u; }
                        if (first) { ndo = rel; nsig = (unsigned)(L_RELAY + cp * 32 + t) * 128u; }
                        else { ndo = (isw ? WS_OFF : XS_OFF) + (unsigned)(dd - 1) * XS_SLOT + (unsigned)I * 524288u + (unsigned)u * 8192u; nsig = (unsigned)(L_PANEL + (dd - 1) * 32 + t) * 128u; }
                        nrdy = !second || !ok;
                    } else {
                        const unsigned it = tk - n1;
                        unsigned tl, st; bool none = false;
                        if (tk >= n13) {
                            unsigned r = tk - n13, ct = 0, w = sh0;
                            if (r >= (w & 1023u)) { r -= w & 1023u; ct = 1; w = sh1;
                                if (r >= (w & 1023u)) { r -= w & 1023u; ct = 2; w = sh2;
                                    if (r >= (w & 1023u)) { r -= w & 1023u; ct = 3; w = sh3; if (r >= (w & 1023u)) { none = true; r = 0; w = 0; } } } }
                            r += (w >> 10) & 1023u;
                            tl = (unsigned)T_POOL + ct * 40u + (r % (unsigned)NP); st = r / (unsigned)NP;
                        }
                        else if (e < 3) { const unsigned k = it / (unsigned)MS; tl = T_ORDF + N0 + k; st = it - k * (unsigned)MS; }
                        else {
                            constexpr unsigned B0 = (N0 * 3 / 4) * 16;
                            if (it < B0) { tl = it >> 4; st = it & 15u; }
                            else { const unsigned i2 = it - B0, k = i2 / (unsigned)(16 - MS); tl = N0 * 3 / 4 + k; st = (unsigned)MS + i2 - k * (unsigned)(16 - MS); }
                            tl += (unsigned)(T_ORD3 + c * 192);
                        }
                        tl = TAB(tl);
                        nk = 4; ndep = (unsigned)(L_TILE + tl) * 128u; nneed = 8u * call;
                        nso = tl * 131072u + st * 8192u;
                        ndo = (panel_base(grp, a1, (int)(tl >> 4)) + st * 16u) * 16384u + panel_base(grp, a0, (int)(tl & 15u)) * 2u;
                        nrdy = allfin || !ok; nfin = allfin;
                        if (none) { nk = 0; more = false; spare = false; STAMP(29); }
                    }
                    if (spare) { spare = false; ++tk; tks = 2; }
                }
                if (!h2 && tk2s == 2) { tk2s = 0; h2 = true; p2 = tk2; tl2 = TAB(T_ORD2 + c * 64 + (p2 >> 4)); r2 = !ok; }
            };
            refill();
            for (;;) {
                int ck = 0; unsigned cso = 0, cdo = 0, csig = 0, cseg = 0, csegn = 1; bool cfin = false;
                if (h2 && r2) {
                    const unsigned o = tl2 * 131072u + (p2 & 15u) * 8192u;
                    ck = 3; cso = PT_OFF + o; cdo = RT_OFF + o; csig = (unsigned)(L_TILE + tl2) * 128u + 64u; cseg = 128u + (p2 >> 4); csegn = 16u; h2 = false;
                    PROG(1, p2);
                } else if (nk && nrdy) {
                    ck = nk; cso = nso; cdo = ndo; csig = nsig; cseg = nseg & 255u; csegn = nsegn; cfin = nfin && ok; nk = 0;
                    PROG(0, nseg >> 8);
                }
                if (ck == 0 && nk == 0 && !more && !h2 && tk2s == 0 && !q2 && psig == 0u && qsig == 0u) break;
                __builtin_amdgcn_fence(__ATOMIC_ACQUIRE, "wavefront");
                const bool add2 = ck == 4 && !cfin;
                u32x4 va[8], vb[8];
                {
                    const __amdgpu_buffer_rsrc_t rs = __builtin_amdgcn_make_buffer_rsrc(ck == 1 ? (void*)P.x : ck == 2 ? (void*)P.w : (void*)myws, 0, ck >= 3 ? (int)WS_END : (int)IN_BYTES, 0x00020000);
                    const unsigned sv = (ck ? cso + (ck == 4 ? (cfin ? FT_OFF : PT_OFF) : 0u) : OOR) + (ck == 2 ? rlane : llane), ss = ck == 2 ? 32768u : 1024u;
                    const unsigned sb = (add2 ? RT_OFF + cso : OOR) + llane;
#pragma unroll
                    for (int k = 0; k < 8; ++k) va[k] = __builtin_amdgcn_raw_buffer_load_b128(rs, sv + k * ss, 0, K_LDAUX);
#pragma unroll
                    for (int k = 0; k < 8; ++k) vb[k] = __builtin_amdgcn_raw_buffer_load_b128(rl, sb + k * 1024u, 0, K_LDAUX);
                }
                __builtin_amdgcn_sched_barrier(0);
                refill();
                if (ck) idle = 0;
                else {
                    if (idle == 0u) t0 = __builtin_amdgcn_s_memrealtime(); else __builtin_amdgcn_s_sleep(K_SLEEP);
                    if ((++idle & 63u) == 0u && ok && gave_up(t0, tmo, call)) { ok = false; nrdy = true; r2 = true; shk = true; }
                }
                const bool cl_own = tks == 0 && more, cl2 = q2 && tk2s == 0 && tk >= (unsigned)Q2_AFTER, ws = !shk && tk + 64u >= n13;
                const unsigned tkv1 = ATOMV(rl, cl_own ? (unsigned)head : OOR, 1);
                const unsigned tkv2 = ATOMV(rl, cl2 ? (unsigned)head + 128u : OOR, 1);
                const bool nw = nk && !(nrdy && (nfin || nk != 4)), w2 = h2 && !r2, f4 = nw && nk == 4;
                const unsigned poff = (lm0 & (nw ? ndep : OOR)) | (lm1 & (f4 ? ndep + 64u : OOR)) | (lm2 & (w2 ? (unsigned)(L_TILE + tl2) * 128u : OOR)) | (lm3 & ((w2 && !started) ? cold : OOR)) |
                                      (lm4 & (f4 ? ndep + 32u : OOR)) | (lm5 & (f4 ? (unsigned)(L_ALLFIN * 128) : OOR)) | (lms & (ws ? (unsigned)(L_SHARE + e) * 128u + (unsigned)(lane & 3) * 4u : OOR)) | lmx;
                const unsigned pv = __builtin_amdgcn_raw_buffer_load_b32(rl, poff, 0, 17);
                __builtin_amdgcn_sched_barrier(0);
                (void)ATOMV(rq, (qsig && UNI(qv) == qsegn * call - 1u) ? qsig : OOR, qsegn);
                const bool mark = !msent && tk + (e < 3 ? 0u : q2 ? tk2 : (unsigned)Q2_ITEMS) >= (unsigned)MARK_AT;
                __builtin_amdgcn_raw_buffer_store_b32(call, rq, l0 | (mark ? (unsigned)(L_MARK + e) * 128u : OOR), 0, 17);
                if (mark) msent = true;
                const __amdgpu_buffer_rsrc_t rd = __builtin_amdgcn_make_buffer_rsrc(ck == 4 ? P.out[e] : P.ws[e], 0, ck == 4 ? (int)OUT_BYTES : (int)WS_END, 0x00020000);
                unsigned dv[8], qoff = l0 | (psig ? (unsigned)scnt + pseg * 4u : OOR);
#pragma unroll
                for (int k = 0; k < 8; ++k) dv[k] = (ck ? cdo : OOR) + (ck == 4 ? rlane + k * 32768u : llane + k * 1024u);
                asm volatile("" : "+v"(dv[0]), "+v"(dv[1]), "+v"(dv[2]), "+v"(dv[3]), "+v"(dv[4]), "+v"(dv[5]), "+v"(dv[6]), "+v"(dv[7]), "+v"(qoff));
                __builtin_amdgcn_sched_barrier(0);
                unsigned one = 1u;
                asm volatile("" : "+v"(one) : "v"(pv) : "memory");
                qv = (unsigned)__builtin_amdgcn_raw_ptr_buffer_atomic_add_i32((int)one, rl, qoff, 0, 16);
                if (add2) {
#pragma unroll
                    for (int k = 0; k < 8; ++k) {
                        va[k] = (u32x4){add_bf16x2(va[k].x, vb[k].x), add_bf16x2(va[k].y, vb[k].y), add_bf16x2(va[k].z, vb[k].z), add_bf16x2(va[k].w, vb[k].w)};
                        if (!ok) va[k] = (u32x4){0x7fc07fc0u, 0x7fc07fc0u, 0x7fc07fc0u, 0x7fc07fc0u};
                    }
                }
#pragma unroll
                for (int k = 0; k < 8; ++k) __builtin_amdgcn_raw_buffer_store_b128(va[k], rd, dv[k], 0, 17  );
                __builtin_amdgcn_sched_barrier(0);
                qsig = psig; qsegn = psegn;
                if (pfin) ++mine;
                const unsigned okm = (unsigned)__ballot((int)(pv - ((lm0 & nneed) | pneed)) >= 0);
                if (okm & 32u) allfin = true;
                if (okm & 8u) started = true;
                if (w2 && started && (okm & 4u)) r2 = true;
                if (nw && (okm & 1u) && (!f4 || (okm & 2u))) { nrdy = true; nfin = K_FT && (okm & 16u); }
                if (allfin && nk == 4) nrdy = nfin = true;
                if (ws && (((unsigned)__ballot((pv >> 20) == tag) >> 8) & 15u) == 15u) {
                    shk = true; sh0 = (unsigned)__builtin_amdgcn_readlane((int)pv, 8); sh1 = (unsigned)__builtin_amdgcn_readlane((int)pv, 9);
                    sh2 = (unsigned)__builtin_amdgcn_readlane((int)pv, 10); sh3 = (unsigned)__builtin_amdgcn_readlane((int)pv, 11);
                }
                if (cl_own) { tk = UNI(tkv1); tks = 2; }
                if (cl2) { tk2 = UNI(tkv2); tk2s = 2; if (tk2 >= (unsigned)Q2_ITEMS) { tk2s = 0; q2 = false; STAMP(2); } }
                pfin = ck == 4; psig = csig; pseg = cseg; psegn = csegn;
            }
            asm volatile("s_waitcnt vmcnt(0)" ::: "memory");
            if (pfin) ++mine;
            if (mine) (void)ATOM(rq, (L_DONE + e) * 128, mine, 16);
        }
        if (broker) {
            const unsigned long long tw = __builtin_amdgcn_s_memrealtime();
            unsigned seen_at = 0, first = 0; bool seen = (lane & 7) == 3;
            for (unsigned polls = 1; ok; ++polls) {
                const unsigned v = ld_relaxed_sys(ctl + (L_MARK + (lane & 7)) * 32), now = (unsigned)__builtin_amdgcn_s_memrealtime() | 1u;
                if (!seen) seen_at = now;
                seen = seen || (int)(v - call) >= 0;
                const unsigned sm = (unsigned)__ballot(seen) & 255u;
                if (sm != 8u && first == 0u) first = now;
                if (sm == 255u || (first && now - first > 2200u)) break;
                if ((polls & 63u) == 0u && gave_up(tw, tmo, call)) ok = false;
                __builtin_amdgcn_s_sleep(8);
            }
            float lag[8], mean = 0.f;
#pragma unroll
            for (int k = 0; k < 8; ++k) { lag[k] = (float)((unsigned)__builtin_amdgcn_readlane((int)seen_at, k) - first) * 0.0715f + (float)(k < 2 ? FIX_N : k == 2 ? FIX_D : FIX_X); if (k != 3) mean += lag[k]; }
            mean = mean * (1.f / 7.f) + (float)(12 * POOL_ITEMS) / 7.f;
            const int pl = lane % 12, pu = pl >> 2, pvv = pl & 3;
            const float su = mean - (pu == 0 ? lag[0] : pu == 1 ? lag[1] : lag[2]), sv = mean - (pvv == 0 ? lag[4] : pvv == 1 ? lag[5] : pvv == 2 ? lag[6] : lag[7]);
            const float S = 3.f * mean - lag[0] - lag[1] - lag[2];
            int x = (int)(su * 0.25f + ((float)POOL_ITEMS - sv * (1.f / 3.f)) - S * (1.f / 12.f) + 0.5f);
            x = x < 0 ? 0 : x > POOL_ITEMS ? POOL_ITEMS : x;
            if (lane < 28) {
                const int se = lane < 12 ? pu : lane < 24 ? 4 + pvv : 4 + (lane & 3), ct = lane < 24 ? (pu + 1) ^ pvv : (lane & 3);
                const unsigned w = tag << 20 | (lane < 12 ? (unsigned)x : lane < 24 ? (unsigned)x << 10 | (unsigned)(POOL_ITEMS - x) : 0u);
                __hip_atomic_store((unsigned*)P.ws[se] + (L_SHARE + se) * 32 + ct, w, __ATOMIC_RELAXED, __HIP_MEMORY_SCOPE_SYSTEM);
            }
        }
        STAMP(3);
        __syncthreads();
        wg_exit(ctl, call, wid, lane);
        if (broker) {
            const unsigned long long tw = __builtin_amdgcn_s_memrealtime();
            for (unsigned polls = 1; ok; ++polls) {
                const int v = (int)ld_relaxed_sys(ctl + (L_DONE + (lane & 7)) * 32);
                unsigned sum = 0;
#pragma unroll
                for (int k = 0; k < 8; ++k) sum += (unsigned)__builtin_amdgcn_readlane(v, k);
                if ((int)(sum - 12288u * call) >= 0) break;
                if ((polls & 63u) == 0u && gave_up(tw, tmo, call)) ok = false;
                __builtin_amdgcn_s_sleep(K_DSLEEP);
            }
        }
        if (!ok && lane == 0) ((unsigned short*)P.out[3])[(size_t)panel_base(grp, a1, 0) * GN + panel_base(grp, a0, 0)] = 0x7fc0;
        return;
    }

    const unsigned call = (unsigned)__builtin_amdgcn_readfirstlane(ld_relaxed_agent(ctl + L_CALL * 32)) + 1u;
    const int njobs = (int)P.njob[xl];
    for (int job = slot - CE_WGS, round = 0; job < njobs; ++round) {
        const unsigned jw = (P.job[(xl * MAXJ + job) >> 1] >> ((job & 1) * 16)) & 0xffffu;
        if (jw >> 8) job = gemm_job<true>(P, lds, myws, call, (int)(jw & 255u), (int)(jw >> 8) & 3, xl, round);
        else job = gemm_job<false>(P, lds, myws, call, (int)jw, 0, xl, round);
    }
    wg_exit(ctl, call, wid, lane);
}

}

extern "C" void kernel_launch(int rank, int nranks, ncclComm_t comm,
                              hipStream_t stream,
                              void* const* d_in, const int* in_sizes, int n_in,
                              void* d_out, int out_size,
                              const int* in_dtypes, int out_dtype,
                              void* const* d_out_all,
                              void* const* d_ws_all, size_t ws_bytes) {
    if (nranks != NR || (unsigned)rank >= (unsigned)NR || n_in < 2 || ws_bytes < WS_END || (size_t)out_size != (size_t)GM * GN ||
        (size_t)in_sizes[0] != (size_t)GM * KS || (size_t)in_sizes[1] != (size_t)KS * GN) {
        fprintf(stderr, "kernel_launch: unexpected problem shape\n");
        return;
    }
    const int member[2][4] = {{0, 2, 3, 6}, {1, 4, 5, 7}};
    Params P;
    P.x = d_in[0]; P.w = d_in[1]; P.g = 0; P.a = 0;
    for (int g = 0; g < 2; ++g) for (int a = 0; a < 4; ++a) if (member[g][a] == rank) { P.g = g; P.a = a; }
    for (int e = 0; e < 8; ++e) {
        const int peer = e < 3 ? member[P.g][P.a ^ (e + 1)] : (e == 3 ? rank : member[1 - P.g][P.a ^ (e - 4)]);
        P.out[e] = d_out_all[peer]; P.ws[e] = d_ws_all[peer];
    }
    unsigned char* tb = (unsigned char*)P.tab;
    int n = 0, n2[4] = {0, 0, 0, 0}, n3[4] = {0, 0, 0, 0}, nx[8] = {0, 0, 0, 0, 0, 0, 0, 0};
    for (int k = 0; k < 16; ++k)
        for (int r = 0; r < 2 * k + 1; ++r) {
            const int I = r < k ? k : r - k, J = r < k ? r : k;
            const unsigned char tl = (unsigned char)(I << 4 | J);
            const int c = (I & 1) << 1 | (J & 1), x = (I & 3) << 1 | ((J >> 1) & 1);
            tb[T_ORDF + n] = tl; tb[T_ORD + n++] = tl; tb[T_ORD2 + c * 64 + n2[c]++] = tl;
            unsigned short* jb = (unsigned short*)P.job;
            if (n <= K_SMALL) { const int xs = (n - 1) & 7; for (int q = 0; q < 4; ++q) jb[xs * MAXJ + nx[xs]++] = (unsigned short)(tl | (4 + q) << 8); }
            else jb[x * MAXJ + nx[x]++] = tl;
        }
    for (int x = 0; x < 8; ++x) P.njob[x] = (unsigned)nx[x];
    {
        const int shell[] = K_SHELLS;
        unsigned rng = 12345u;
        for (unsigned si = 0; si + 1 < sizeof(shell) / sizeof(shell[0]); ++si) {
            const int lo = shell[si] * shell[si], hi = shell[si + 1] * shell[si + 1];
            for (int i = hi - 1; i > lo; --i) { rng = rng * 1664525u + 1013904223u; const int j = lo + (int)((rng >> 8) % (unsigned)(i - lo + 1)); const unsigned char t = tb[T_ORDF + i]; tb[T_ORDF + i] = tb[T_ORDF + j]; tb[T_ORDF + j] = t; }
        }
        int seen[4] = {0, 0, 0, 0}, ns = 0;
        for (int i = 0; i < NTILE; ++i) {
            const unsigned char tl = tb[T_ORDF + i]; const int c = ((tl >> 4) & 1) << 1 | (tl & 1);
            if (seen[c] >= 64 - NP) { tb[T_POOL + c * 40 + seen[c]++ - (64 - NP)] = tl; continue; }
            ++seen[c]; tb[T_ORDF + ns++] = tl;
            for (int o = 0; o < 4; ++o) if (o != c) tb[T_ORD3 + o * 192 + n3[o]++] = tl;
        }
    }
    hipLaunchKernelGGL(gemm_ar_kernel, dim3(GRID), dim3(512), 0, stream, P);
    HIP_CHECK(hipGetLastError());
}
```

```cpp
#include <hip/hip_runtime.h>
#include <rccl/rccl.h>
#include <cstdio>

#define HIP_CHECK(expr) do { hipError_t _e = (expr); if (_e != hipSuccess) { \
    fprintf(stderr, "HIP error %s at %s:%d\n", hipGetErrorString(_e), __FILE__, __LINE__); \
    return; } } while (0)

namespace {

#define LAS __attribute__((address_space(3)))
typedef short bf16x8 __attribute__((ext_vector_type(8)));
typedef short s16x4 __attribute__((ext_vector_type(4)));
typedef float f32x4 __attribute__((ext_vector_type(4)));
typedef unsigned u32x4 __attribute__((ext_vector_type(4)));

constexpr int NR = 8;
constexpr int GM = 8192, GN = 8192;
constexpr int KS = 1024;

#define K_NA 36
#define K_NB 8
#define K_NL 118
#define K_SMALL 49
#define K_MARK 200
#define K_SLEEP 8
#define K_CE 12
constexpr int NA = K_NA, NB = K_NB, NL = K_NL;
static_assert(NA >= 1 && NB >= 0 && NA < NL && NL <= 128 - NB, "knobs");

constexpr unsigned CTL_BYTES = 1u << 20;
constexpr unsigned XS_OFF = CTL_BYTES, XS_SLOT = 16u * 524288u;
constexpr unsigned WS_OFF = XS_OFF + 3u * XS_SLOT;
constexpr unsigned PT_OFF = WS_OFF + 3u * XS_SLOT;
constexpr unsigned RT_OFF = PT_OFF + 256u * 131072u;
constexpr unsigned FT_OFF = RT_OFF + 256u * 131072u;
constexpr unsigned WS_END = FT_OFF + 256u * 131072u;
constexpr unsigned OUT_BYTES = (unsigned)((size_t)GM * GN * 2);
constexpr unsigned IN_BYTES = (unsigned)((size_t)GM * KS * 2);

constexpr int L_TMO = 1, L_EXIT = 2, L_CALL = 3  , L_NFIN = 4, L_ALLFIN = 5, L_START = 6, L_DONE = 24  , L_LAND = 32  ,
              L_MARK = 40  , L_SHARE = 48  , L_HEAD = 64  , L_PANEL = 128  ,
              L_TILE = 512  , L_SCNT = 1024  ;
constexpr int CE_WGS = K_CE;
constexpr int G_WGS = 32 - CE_WGS;
constexpr int GRID = 256;
constexpr unsigned long long DEADLINE_TICKS = 50000000ull;
constexpr unsigned long long WATCH_DELAY = 1000;

constexpr int IN_ITEMS = 32 * 32;
constexpr int Q2_ITEMS = 128 * 16;
constexpr int FIX6 = IN_ITEMS + 128 * 16, FIX4 = (NA + NB) * 16;
constexpr int POOL_ITEMS = (128 - NA - NB) * 16, FWD_MAX = (NL - NA) * 16;
constexpr int MARK4 = K_MARK, MARK6 = K_MARK + (44 - NA - NB) * 16;
static_assert(NA + NB <= 44, "marks");
constexpr int Q3L_ITEMS = 128 * 4;
static_assert(POOL_ITEMS < 2048, "share word");

constexpr int BK = 64, HALF = 128, HTB = HALF * BK * 2, STAGE_BYTES = 8 * HTB;

#define STAMP_TK do {} while (0)
#define STAMP(i) do {} while (0)
#define PROG(q, it) do {} while (0)

constexpr int MAXJ = 96;
static_assert((K_SMALL + 7) / 8 * 4 + 32 <= MAXJ, "job table");
struct Params {
    const void* x; const void* w;
    void* out[8]; void* ws[8];
    int g, a;
    unsigned njob[8], job[8 * MAXJ / 2];
    unsigned tab[64];
};
#define TAB(i) ((P.tab[(unsigned)(i) >> 2] >> (((unsigned)(i) & 3u) * 8u)) & 255u)

__device__ __forceinline__ int lds_byte(int r, int c) { const int st = (r >> 4) * 2 + (c >> 5), rr = r & 15, cc = c & 31, ob = rr * 64 + cc * 2; return st * 1024 + (ob ^ (((ob >> 9) & 1) << 5)); }
__device__ __forceinline__ void stage_rc(int b, int& R, int& C) { const int st = b / 1024, sb = b % 1024, swz = sb ^ (((sb >> 9) & 1) << 5); R = (st >> 1) * 16 + swz / 64; C = (st & 1) * 32 + (swz % 64) / 2; }
__device__ __forceinline__ unsigned wswz(unsigned row) { return ((row & 3u) << 2) | ((row >> 2) & 3u); }
__device__ __forceinline__ unsigned cvt_pk_bf16(float lo, float hi) { unsigned r; asm volatile("v_cvt_pk_bf16_f32 %0, %1, %2" : "=v"(r) : "v"(lo), "v"(hi)); return r; }
__device__ __forceinline__ unsigned panel_base(int ab, int I) {
    const int j = I >> 1;
    return (unsigned)((j << 2) | ((2 * ab + (I & 1) + (0xb114 >> (2 * j))) & 3)) * 256u;
}
__device__ __forceinline__ unsigned out_offset(int a, unsigned tl, unsigned st) {
    return (panel_base(a >> 1, (int)(tl >> 4)) + st * 16u) * 16384u + panel_base(a & 1, (int)(tl & 15u)) * 2u;
}

__device__ __forceinline__ unsigned ld_relaxed_sys(const unsigned* p) { return __hip_atomic_load(p, __ATOMIC_RELAXED, __HIP_MEMORY_SCOPE_SYSTEM); }
__device__ __forceinline__ unsigned ld_relaxed_agent(const unsigned* p) { return __hip_atomic_load(p, __ATOMIC_RELAXED, __HIP_MEMORY_SCOPE_AGENT); }

__device__ __forceinline__ bool gave_up(unsigned long long t0, unsigned* tmo, unsigned call) {
    if ((unsigned)__builtin_amdgcn_readfirstlane(ld_relaxed_agent(tmo)) == call || __builtin_amdgcn_s_memrealtime() - t0 > DEADLINE_TICKS) {
        __hip_atomic_store(tmo, call, __ATOMIC_RELAXED, __HIP_MEMORY_SCOPE_AGENT);
        return true;
    }
    return false;
}
__device__ __forceinline__ unsigned add_bf16x2(unsigned a, unsigned b) {
    return cvt_pk_bf16(__uint_as_float(a << 16) + __uint_as_float(b << 16), __uint_as_float(a & 0xffff0000u) + __uint_as_float(b & 0xffff0000u));
}

__device__ __forceinline__ void wg_exit(unsigned* ctl, unsigned call, int wid, int lane) {
    if (wid != 0) return;
    const unsigned t = (unsigned)__builtin_amdgcn_raw_ptr_buffer_atomic_add_i32(1, __builtin_amdgcn_make_buffer_rsrc(ctl, 0, (int)CTL_BYTES, 0x00020000), lane == 0 ? (unsigned)L_EXIT * 128u : 0x7fffff00u, 0, 0);
    if (((unsigned)__builtin_amdgcn_readfirstlane((int)t) & (unsigned)(GRID - 1)) == (unsigned)(GRID - 1)) {
        if (lane < 25) __hip_atomic_store(ctl + (lane == 24 ? L_CALL : L_HEAD + lane) * 32, lane == 24 ? call : 0u, __ATOMIC_RELAXED, __HIP_MEMORY_SCOPE_AGENT);
    }
}

template <bool SMALL>
__device__ __forceinline__ int gemm_job(const Params& P, LAS unsigned char* lds, char* myws, unsigned call, int tl, int qd, int xl, int round) {
    int tid = threadIdx.x; asm volatile("" : "+v"(tid));
    const int wid = __builtin_amdgcn_readfirstlane(tid >> 6), lane = tid & 63;
    const int a1 = P.a >> 1, a0 = P.a & 1, ti = tl >> 4, tj = tl & 15;
    unsigned* ctl = (unsigned*)myws;
    unsigned* tmo = ctl + L_TMO * 32;
    const int wr = wid >> 2, wc = wid & 3, fr = lane & 15, fq = lane >> 4;
    unsigned voffA[2], voffB[2];
#pragma unroll
    for (int i = 0; i < 2; ++i) { int R, C; stage_rc(tid * 16 + i * 8192, R, C); voffA[i] = (unsigned)(R * KS + C) * 2u; }
#define SET_VOFFB(sh) do { int tz = tid; asm volatile("" : "+v"(tz)); _Pragma("unroll") for (int i = 0; i < 2; ++i) { \
        const unsigned row = (unsigned)(tz >> 4) + 32u * i, ch = ((unsigned)tz & 15u) ^ wswz(row); voffB[i] = (row << (sh)) + ch * 16u; } \
        kstepB = (size_t)(SMALL ? 2 * BK : BK) << (sh); hstepB = SMALL ? (size_t)BK << (sh) : (size_t)HALF * 2; } while (0)
    size_t kstepB, hstepB;
    constexpr size_t kstepA = SMALL ? BK * 4 : BK * 2, hstepA = SMALL ? BK * 2 : (size_t)HALF * KS * 2;
    const unsigned qa = SMALL ? (unsigned)(qd >> 1) * (unsigned)(HALF * KS * 2) : 0u, qb = SMALL ? (unsigned)(qd & 1) * 256u : 0u;
    const unsigned ldsw = (unsigned)wid * 1024u;
    const int aoff = lds_byte(wr * 64 + fr, fq * 8);
    int boff[2][2];
#pragma unroll
    for (int n = 0; n < 2; ++n)
#pragma unroll
        for (int t = 0; t < 2; ++t) {
            const unsigned q4 = (unsigned)(lane & 15) >> 2, p4 = (unsigned)lane & 3u, row = 8u * fq + 4u * t + q4, ch = 4u * wc + 2u * n + (p4 >> 1);
            boff[n][t] = (int)((unsigned)(size_t)lds + 4u * HTB + 256u * row + 16u * (ch ^ wswz(row)) + 8u * (p4 & 1u));
        }
#define SA(b, h) (((b) * 2 + (h)) * HTB)
#define SB(b, h) ((4 + (b) * 2 + (h)) * HTB)
#define STAGE(bufoff, gbase, voff) do { _Pragma("unroll") for (int _i = 0; _i < 2; ++_i) \
        __builtin_amdgcn_global_load_lds((const unsigned*)((const char*)(gbase) + (voff)[_i]), (LAS unsigned*)(lds + (bufoff) + ldsw + _i * 8192), 16, 0, 0); } while (0)
#define LDA(dst, b, h) do { _Pragma("unroll") for (int m = 0; m < 4; ++m) _Pragma("unroll") for (int k = 0; k < 2; ++k) dst[m][k] = *(const LAS bf16x8*)(lds + SA(b, h) + aoff + m * 2048 + k * 1024); } while (0)
#define TRR(dst, base, off) asm volatile("ds_read_b64_tr_b16 %0, %1 offset:%2" : "=&v"(dst) : "v"(base), "i"(off) : "memory")
#define LDB(raw, b, h) do { \
        TRR(raw[0], boff[0][0], SB(b, h) - 4 * HTB); TRR(raw[1], boff[0][1], SB(b, h) - 4 * HTB); TRR(raw[2], boff[0][0], SB(b, h) - 4 * HTB + 8192); TRR(raw[3], boff[0][1], SB(b, h) - 4 * HTB + 8192); \
        TRR(raw[4], boff[1][0], SB(b, h) - 4 * HTB); TRR(raw[5], boff[1][1], SB(b, h) - 4 * HTB); TRR(raw[6], boff[1][0], SB(b, h) - 4 * HTB + 8192); TRR(raw[7], boff[1][1], SB(b, h) - 4 * HTB + 8192); } while (0)
#define WAIT_B(raw, dst) do { asm volatile("s_waitcnt lgkmcnt(0)" : "+v"(raw[0]), "+v"(raw[1]), "+v"(raw[2]), "+v"(raw[3]), "+v"(raw[4]), "+v"(raw[5]), "+v"(raw[6]), "+v"(raw[7]) :: "memory"); \
        _Pragma("unroll") for (int n = 0; n < 2; ++n) _Pragma("unroll") for (int k = 0; k < 2; ++k) dst[n][k] = __builtin_shufflevector(raw[n * 4 + k * 2], raw[n * 4 + k * 2 + 1], 0, 1, 2, 3, 4, 5, 6, 7); } while (0)
#define MMA(ai, bj, At, Bt) do { if (!SMALL || (ai) == (bj)) { __builtin_amdgcn_s_setprio(1); _Pragma("unroll") for (int m = 0; m < 4; ++m) _Pragma("unroll") for (int n = 0; n < 2; ++n) _Pragma("unroll") for (int k = 0; k < 2; ++k) \
        acc[SMALL ? 0 : ai][SMALL ? 0 : bj][m][n] = __builtin_amdgcn_mfma_f32_16x16x32_bf16(Bt[n][k], At[m][k], acc[SMALL ? 0 : ai][SMALL ? 0 : bj][m][n], 0, 0, 0); __builtin_amdgcn_s_setprio(0); } } while (0)
#define WAIT_V(n) asm volatile("s_waitcnt vmcnt(" #n ")" ::: "memory")
#define WAIT_L(n) asm volatile("s_waitcnt lgkmcnt(" #n ")" ::: "memory")
#define BAR __builtin_amdgcn_s_barrier()
#define SCHED __builtin_amdgcn_sched_barrier(0)
    constexpr int NH2 = SMALL ? 1 : 2;
    f32x4 acc[NH2][NH2][4][2];
#pragma unroll
    for (int a = 0; a < NH2; ++a)
#pragma unroll
        for (int b = 0; b < NH2; ++b)
#pragma unroll
            for (int m = 0; m < 4; ++m)
#pragma unroll
                for (int n = 0; n < 2; ++n) acc[a][b][m][n] = (f32x4){0.f, 0.f, 0.f, 0.f};
    bf16x8 At[4][2], B0[2][2], B1[2][2];
    s16x4 R0[8], R1[8];
    const char* cA = (const char*)P.x + (size_t)panel_base(a1, ti) * (KS * 2) + qa;
    const char* cB = (const char*)P.w + (size_t)panel_base(a0, tj) * 2 + qb;
    constexpr int nt = SMALL ? KS / BK / 2 : KS / BK;
    unsigned todo = 7u;
    SET_VOFFB(14);
    STAGE(SB(0, 0), cB, voffB); STAGE(SA(0, 0), cA, voffA); STAGE(SB(0, 1), cB + hstepB, voffB); STAGE(SA(0, 1), cA + hstepA, voffA);
    if (wr == 1) BAR;
    WAIT_V(4); BAR;
    STAGE(SB(1, 0), cB + kstepB, voffB); STAGE(SA(1, 0), cA + kstepA, voffA); STAGE(SB(1, 1), cB + hstepB + kstepB, voffB);
    WAIT_V(6); BAR;
    for (int s = 0; s < 4; ++s) {
        const char* nA = cA; const char* nB = cB;
        for (int t = 0; t < nt; t += 2) {
            const bool last = (t == nt - 2);
            if (last && s < 3) {
                if (wid == 0) {
                    STAMP(1 + 2 * s + 8 * round);
                    const unsigned* word = ctl + (L_PANEL + ((lane >> 1) % 3) * 32 + ((lane & 1) ? 2 * tj + 1 : 2 * ti)) * 32;
                    const unsigned long long t0 = __builtin_amdgcn_s_memrealtime();
                    unsigned pick;
                    for (unsigned polls = 1;; ++polls) {
                        const unsigned mk = (unsigned)__ballot((int)(ld_relaxed_sys(word) - 64u * call) >= 0);
                        const unsigned both = mk & (mk >> 1);
                        pick = ((both & 1u) | ((both >> 1) & 2u) | ((both >> 2) & 4u)) & todo;
                        if (pick) break;
                        if ((polls & 63u) == 0u && gave_up(t0, tmo, call)) { pick = todo; break; }
                        __builtin_amdgcn_s_sleep(4);
                    }
                    STAMP(2 + 2 * s + 8 * round);
                    *(LAS unsigned*)(lds + STAGE_BYTES + 16) = (unsigned)__builtin_ctz(pick);
                    __builtin_amdgcn_fence(__ATOMIC_ACQUIRE, "");
                    asm volatile("s_waitcnt vmcnt(0) lgkmcnt(0)" ::: "memory");
                }
                asm volatile("" ::: "memory"); BAR; asm volatile("" ::: "memory");
                unsigned slv;
                asm volatile("ds_read_b32 %0, %1\n\ts_waitcnt lgkmcnt(0)" : "=&v"(slv) : "v"((unsigned)(size_t)lds + STAGE_BYTES + 16) : "memory");
                const unsigned sl = (unsigned)__builtin_amdgcn_readfirstlane(slv) & 3u;
                todo &= ~(1u << sl);
                nA = myws + XS_OFF + sl * XS_SLOT + (unsigned)ti * 524288u + qa;
                nB = myws + WS_OFF + sl * XS_SLOT + (unsigned)tj * 524288u + qb;
                if (s == 0) SET_VOFFB(9);
            }
            const char* a1p = cA + (size_t)(t + 1) * kstepA;
            const char* a2 = last ? nA : cA + (size_t)(t + 2) * kstepA; const char* b2 = last ? nB : cB + (size_t)(t + 2) * kstepB;
            const char* a3 = a2 + kstepA; const char* b3 = b2 + kstepB;
            LDB(R0, 0, 0); SCHED; LDA(At, 0, 0); STAGE(SA(1, 1), a1p + hstepA, voffA);
            WAIT_L(8); BAR; WAIT_B(R0, B0); MMA(0, 0, At, B0); BAR; SCHED;
            LDB(R1, 0, 1); STAGE(SB(0, 0), b2, voffB);
            BAR; WAIT_B(R1, B1); MMA(0, 1, At, B1); BAR; if (SMALL) SCHED;
            LDA(At, 0, 1); STAGE(SA(0, 0), a2, voffA);
            BAR; WAIT_L(0); MMA(1, 0, At, B0); BAR; SCHED;
            STAGE(SB(0, 1), b2 + hstepB, voffB);
            WAIT_V(6); BAR; MMA(1, 1, At, B1); BAR;
            LDB(R0, 1, 0); SCHED; LDA(At, 1, 0); STAGE(SA(0, 1), a2 + hstepA, voffA);
            WAIT_L(8); BAR; WAIT_B(R0, B0); MMA(0, 0, At, B0); BAR; SCHED;
            LDB(R1, 1, 1); STAGE(SB(1, 0), b3, voffB);
            BAR; WAIT_B(R1, B1); MMA(0, 1, At, B1); BAR; if (SMALL) SCHED;
            LDA(At, 1, 1); STAGE(SA(1, 0), a3, voffA);
            BAR; WAIT_L(0); MMA(1, 0, At, B0); BAR; SCHED;
            STAGE(SB(1, 1), b3 + hstepB, voffB);
            WAIT_V(6); BAR; MMA(1, 1, At, B1); BAR;
        }
        cA = nA; cB = nB;
    }
    WAIT_V(0);
    if (wr == 0) BAR;
    STAMP(7 + 8 * round);
    const __amdgpu_buffer_rsrc_t rc = __builtin_amdgcn_make_buffer_rsrc(myws, 0, (int)CTL_BYTES, 0x00020000);
    const unsigned nj = (unsigned)__builtin_amdgcn_raw_ptr_buffer_atomic_add_i32(1, rc, tid == 0 ? (unsigned)(L_HEAD + 16 + xl) * 128u : 0x7fffff00u, 0, 0);
    const bool bad = (unsigned)__builtin_amdgcn_readfirstlane(ld_relaxed_agent(tmo)) == call;
    const __amdgpu_buffer_rsrc_t ro = __builtin_amdgcn_make_buffer_rsrc(myws + PT_OFF + (unsigned)tl * 131072u, 0, 131072, 0x00020000);
    const unsigned obase = (unsigned)(64 * wr + fr + (SMALL ? (qd >> 1) * HALF : 0)) * 512u + (unsigned)(32 * wc + (fq & 1) * 16 + (fq >> 1) * 8) * 2u + qb;
#pragma unroll
    for (int ai = 0; ai < NH2; ++ai)
#pragma unroll
        for (int m = 0; m < 4; ++m)
#pragma unroll
            for (int bj = 0; bj < NH2; ++bj) {
                const f32x4 v0 = acc[ai][bj][m][0], v1 = acc[ai][bj][m][1];
                const unsigned x0 = cvt_pk_bf16(v0[0], v0[1]), x1 = cvt_pk_bf16(v0[2], v0[3]), y0 = cvt_pk_bf16(v1[0], v1[1]), y1 = cvt_pk_bf16(v1[2], v1[3]);
                const auto r0 = __builtin_amdgcn_permlane16_swap(x0, y0, false, false);
                const auto r1 = __builtin_amdgcn_permlane16_swap(x1, y1, false, false);
                u32x4 wv = {r0[0], r1[0], r0[1], r1[1]};
                if (bad) wv = (u32x4){0x7fc07fc0u, 0x7fc07fc0u, 0x7fc07fc0u, 0x7fc07fc0u};
                __builtin_amdgcn_raw_buffer_store_b128(wv, ro, obase + (unsigned)(ai * HALF + m * 16) * 512u + (unsigned)bj * 256u, 0, 16  );
            }
    if (wid == 0) *(LAS unsigned*)(lds + STAGE_BYTES) = (unsigned)__builtin_amdgcn_readfirstlane((int)nj);
    asm volatile("s_waitcnt vmcnt(0) lgkmcnt(0)" ::: "memory");
    BAR;
    (void)__builtin_amdgcn_raw_ptr_buffer_atomic_add_i32(SMALL ? 2 : 8, rc, tid == 0 ? (unsigned)(L_TILE + tl) * 128u : 0x7fffff00u, 0, 0);
    STAMP(8 + 8 * round);
    unsigned njv;
    asm volatile("ds_read_b32 %0, %1\n\ts_waitcnt lgkmcnt(0)" : "=&v"(njv) : "v"((unsigned)(size_t)lds + STAGE_BYTES) : "memory");
    return __builtin_amdgcn_readfirstlane((int)njv) + G_WGS;
}

__global__ __launch_bounds__(512, 2) void gemm_ar_kernel(const Params P) {
    __shared__ __attribute__((aligned(16))) unsigned char shm[STAGE_BYTES + 64];
    LAS unsigned char* lds = (LAS unsigned char*)shm;
    const int tid = threadIdx.x, wid = __builtin_amdgcn_readfirstlane(tid >> 6), lane = tid & 63;
    const int hm = P.g, qa = P.a;
    const int xl = blockIdx.x & 7, slot = blockIdx.x >> 3;
    char* myws = (char*)P.ws[3];
    unsigned* ctl = (unsigned*)myws;
    unsigned* tmo = ctl + L_TMO * 32;
    STAMP(0);

    if (slot < CE_WGS) {
        int e = (int)((unsigned)__builtin_amdgcn_s_getreg(20 | (3 << 11)) & 7u);
        const bool broker = blockIdx.x == 0 && wid == 6;
        bool active = (wid == 0 || e == 3) && !broker;
        const __amdgpu_buffer_rsrc_t rl = __builtin_amdgcn_make_buffer_rsrc(myws, 0, (int)WS_END, 0x00020000);
        const unsigned l0 = lane == 0 ? 0u : 0x7fffff00u;
#define ATOM(rs, off, v, aux) (unsigned)__builtin_amdgcn_raw_ptr_buffer_atomic_add_i32((int)(v), rs, l0, off, aux)
#define ATOMV(rs, off, v) (unsigned)__builtin_amdgcn_raw_ptr_buffer_atomic_add_i32((int)(v), rs, l0 | (off), 0, 16)
#define UNI(x) (unsigned)__builtin_amdgcn_readfirstlane((int)(x))
        if (wid == 7) {
            if (blockIdx.x == 0) (void)ATOM(__builtin_amdgcn_make_buffer_rsrc(P.ws[4], 0, (int)CTL_BYTES, 0x00020000), L_START * 128, 1, 16);
            const unsigned long long t0 = __builtin_amdgcn_s_memrealtime();
            while (__builtin_amdgcn_s_memrealtime() - t0 < WATCH_DELAY) __builtin_amdgcn_s_sleep(16);
            if (UNI(ld_relaxed_agent(ctl + (L_HEAD + xl * 2) * 32)) == 0u) { e = xl; active = true; }
        }
        const __amdgpu_buffer_rsrc_t rq = __builtin_amdgcn_make_buffer_rsrc(P.ws[e], 0, (int)WS_END, 0x00020000);
        const unsigned llane = (unsigned)lane * 16u;
        const unsigned rlane = (unsigned)(lane >> 5) * 16384u + (unsigned)(lane & 31) * 16u;
        const int head = (L_HEAD + e * 2) * 128, scnt = (L_SCNT + e * 8) * 128;
        unsigned tkv = 0, call = 0;
        if (active) tkv = ATOM(rl, head, e == 3 ? 1 : 2, 0);
        if (active || broker) call = UNI(__builtin_amdgcn_raw_buffer_load_b32(rl, L_CALL * 128u, 0, 16)) + 1u;
        STAMP(4);
        const unsigned tag = call % 1023u + 1u;
        bool ok = true;
        unsigned idle = 0; unsigned long long t0 = 0;
        if (active && e == 3) {
            const __amdgpu_buffer_rsrc_t ro = __builtin_amdgcn_make_buffer_rsrc(P.out[3], 0, (int)OUT_BYTES, 0x00020000);
            for (unsigned it = UNI(tkv); it < (unsigned)Q3L_ITEMS; it = UNI(tkv)) {
                const unsigned tl = TAB(hm * 128 + (it >> 2)), st = (it & 3u) * 4u, dep = (unsigned)(L_TILE + tl) * 128u;
                t0 = __builtin_amdgcn_s_memrealtime();
                for (idle = 1; ok; ++idle) {
                    const unsigned v = __builtin_amdgcn_raw_buffer_load_b32(rl, dep + (unsigned)(lane & 1) * 64u, 0, 17);
                    if (__all((int)(v - ((lane & 1) ? 16u : 8u) * call) >= 0)) break;
                    if ((idle & 63u) == 0u && gave_up(t0, tmo, call)) ok = false;
                    __builtin_amdgcn_s_sleep(K_SLEEP);
                }
                __builtin_amdgcn_fence(__ATOMIC_ACQUIRE, "wavefront");
                PROG(2, it);
                const unsigned so = tl * 131072u + st * 8192u + llane, dO = out_offset(qa, tl, st) + rlane;
                for (int sub = 0; sub < 4; ++sub) {
                    const unsigned sv = so + (unsigned)sub * 8192u, dv = dO + (unsigned)sub * 262144u;
                    u32x4 va[8], vb[8];
#pragma unroll
                    for (int k = 0; k < 8; ++k) { va[k] = __builtin_amdgcn_raw_buffer_load_b128(rl, PT_OFF + sv + k * 1024u, 0, 17); vb[k] = __builtin_amdgcn_raw_buffer_load_b128(rl, RT_OFF + sv + k * 1024u, 0, 17); }
#pragma unroll
                    for (int k = 0; k < 8; ++k) {
                        u32x4 o = {add_bf16x2(va[k].x, vb[k].x), add_bf16x2(va[k].y, vb[k].y), add_bf16x2(va[k].z, vb[k].z), add_bf16x2(va[k].w, vb[k].w)};
                        if (!ok) o = (u32x4){0x7fc07fc0u, 0x7fc07fc0u, 0x7fc07fc0u, 0x7fc07fc0u};
                        __builtin_amdgcn_raw_buffer_store_b128(o, rl, FT_OFF + sv + k * 1024u, 0, 16);
                        __builtin_amdgcn_raw_buffer_store_b128(o, ro, dv + k * 32768u, 0, 16);
                    }
                }
                tkv = ATOM(rl, head, 1, 0);
                asm volatile("s_waitcnt vmcnt(0)" ::: "memory");
                if ((UNI(ATOM(rl, scnt + (int)(it >> 2) * 4, 1, 0)) & 3u) == 3u) {
                    (void)ATOM(rl, (int)dep + 32, 4, 0);
                    if ((UNI(ATOM(rl, L_NFIN * 128, 1, 0)) & 127u) == 127u) (void)ATOM(rl, L_ALLFIN * 128, 1, 0);
                }
            }
        } else if (active) {
            const int x = e < 3 ? e + 1 : e - 4, hp = e < 3 ? hm : hm ^ 1, qp = qa ^ x;
            const unsigned nfix = e == 4 ? (unsigned)FIX4 : (unsigned)FIX6;
            constexpr unsigned OOR = 0x7fff0000u;
            bool more = true, spare = true, shk = false, msent = false;
            int tks = 2, tk2s = 0;
            unsigned tk = UNI(tkv), tk2 = 0, share = 0;
            bool q2 = e == 4;
            unsigned known = 0;
            int nk = 0; unsigned nso = 0, ndo = 0, ndep = OOR, nneed = 0, nsig = 0, nseg = 0, nsegn = 1; bool nrdy = false, nfin = false;
            bool h2 = false, r2 = false; unsigned p2 = 0, tl2 = 0;
            unsigned psig = 0, pseg = 0, psegn = 1, qsig = 0, qsegn = 1, qv = 0;
            unsigned mine = 0; bool pfin = false;
            const unsigned lm0 = lane == 0 ? ~0u : 0u, lm1 = lane == 1 ? ~0u : 0u, lm2 = lane == 2 ? ~0u : 0u, lm3 = lane == 3 ? ~0u : 0u, lm4 = lane == 4 ? ~0u : 0u, lm5 = lane == 5 ? ~0u : 0u,
                           lm6 = lane == 6 ? ~0u : 0u, lmx = lane > 6 ? OOR : 0u;
            const unsigned pneed = (lm1 & (16u * call)) | (lm2 & (8u * call)) | (lm4 & (4u * call)) | ((lm3 | lm5) & call);
#define REFILL() do { \
                if (nk == 0 && tks == 2 && (tk < nfix || shk)) { \
                    tks = 0; nneed = 0; nfin = false; nsig = 0; nseg = 0; \
                    if (e != 4 && tk < (unsigned)IN_ITEMS) {    \
                        const unsigned t = tk >> 5, u = tk & 31u, ds = (unsigned)(x - 1) * XS_SLOT + (t >> 1) * 524288u, half = e < 3 ? 0u : 1u; \
                        if (t & 1u) { nk = 2; nso = ((unsigned)hp * 512u + u * 16u) * 16384u + panel_base(qp & 1, (int)(t >> 1)) * 2u; ndo = WS_OFF + ds + half * 262144u + u * 8192u; } \
                        else { nk = 1; nso = (panel_base(qp >> 1, (int)(t >> 1)) + u * 8u) * 2048u + (unsigned)hp * 1024u; ndo = XS_OFF + ds + u * 16384u + half * 1024u; } \
                        nseg = t; nsegn = 32u; nsig = (unsigned)(L_PANEL + (x - 1) * 32 + (int)t) * 128u; \
                        nrdy = true; \
                    } else { \
                        unsigned r; bool fwd = false, none = false;    \
                        if (tk >= nfix) { r = tk - nfix; none = r >= (share & 2047u); r = none ? 0u : r + ((share >> 11) & 2047u) + (unsigned)(NA * 16); fwd = e != 4; } \
                        else if (e == 4) r = tk < (unsigned)(NA * 16) ? tk : tk + (unsigned)POOL_ITEMS; \
                        else { r = tk - (unsigned)IN_ITEMS; if (r < (unsigned)(NL * 16)) { nseg = 64u; nsegn = (unsigned)(NL * 16); nsig = (unsigned)(L_LAND + e) * 128u; } } \
                        const unsigned tl = TAB((fwd ? hp ^ 1 : hm) * 128 + (int)(r >> 4)), st = r & 15u; \
                        ndo = out_offset(fwd ? qp : qa, tl, st); \
                        nk = fwd ? 5 : 4; \
                        nso = fwd ? ndo : tl * 131072u + st * 8192u; \
                        ndep = fwd ? (unsigned)(L_LAND + (e < 3 ? 4 + x : x - 1)) * 128u : (unsigned)(L_TILE + tl) * 128u; \
                        nneed = (fwd ? (unsigned)(NL * 16) : 8u) * call; \
                        nrdy = ((known >> (fwd ? 0 : 5)) & 1u) != 0u || !ok; \
                        nfin = !fwd && (known & 32u) != 0u; \
                        if (none) { nk = 0; more = false; spare = false; STAMP(29); } \
                    } \
                    STAMP_TK; \
                    if (spare) { spare = false; ++tk; tks = 2; } \
                } \
                if (!h2 && tk2s == 2) { tk2s = 0; h2 = true; p2 = tk2; tl2 = TAB((hm ^ 1) * 128 + (int)(p2 >> 4)); r2 = !ok; } \
            } while (0)
            REFILL();
            for (;;) {
                int ck = 0; unsigned cso = 0, cdo = 0, csig = 0, cseg = 0, csegn = 1; bool cfin = false;
                if (h2 && r2) {
                    const unsigned o = tl2 * 131072u + (p2 & 15u) * 8192u;
                    ck = 3; cso = PT_OFF + o; cdo = RT_OFF + o; csig = (unsigned)(L_TILE + tl2) * 128u + 64u; cseg = 128u + (p2 >> 4); csegn = 16u; h2 = false;
                    PROG(1, p2);
                } else if (nk && nrdy) {
                    ck = nk; cso = nso; cdo = ndo; csig = nsig; cseg = nseg & 255u; csegn = nsegn; cfin = nfin && ok; nk = 0;
                    PROG(0, nseg >> 8);
                }
                if (ck == 0 && nk == 0 && !more && !h2 && tk2s == 0 && !q2 && psig == 0u && qsig == 0u) break;
                __builtin_amdgcn_fence(__ATOMIC_ACQUIRE, "wavefront");
                const bool add2 = (ck == 4 && !cfin) || (ck == 5 && !ok);
                u32x4 va[8], vb[8];
                {
                    const __amdgpu_buffer_rsrc_t rs = __builtin_amdgcn_make_buffer_rsrc(ck == 1 ? (void*)P.x : ck == 2 ? (void*)P.w : ck == 5 ? P.out[3] : (void*)myws, 0,
                                                                                        ck == 5 ? (int)OUT_BYTES : ck >= 3 ? (int)WS_END : (int)IN_BYTES, 0x00020000);
                    const bool sr = ck == 2 || ck == 5;
                    const unsigned sv = (ck ? cso + (ck == 4 ? (cfin ? FT_OFF : PT_OFF) : 0u) : OOR) + (sr ? rlane : llane), ss = sr ? 32768u : ck == 1 ? 2048u : 1024u;
                    const unsigned sb = ((add2 && ck == 4) ? RT_OFF + cso : OOR) + llane;
#pragma unroll
                    for (int k = 0; k < 8; ++k) va[k] = __builtin_amdgcn_raw_buffer_load_b128(rs, sv + k * ss, 0, 17  );
#pragma unroll
                    for (int k = 0; k < 8; ++k) vb[k] = __builtin_amdgcn_raw_buffer_load_b128(rl, sb + k * 1024u, 0, 17);
                }
                __builtin_amdgcn_sched_barrier(0);
                REFILL();
                if (ck) idle = 0;
                else {
                    if (idle == 0u) t0 = __builtin_amdgcn_s_memrealtime(); else __builtin_amdgcn_s_sleep(K_SLEEP);
                    if ((++idle & 63u) == 0u && ok && gave_up(t0, tmo, call)) { ok = false; nrdy = true; r2 = true; shk = true; }
                }
                const bool cl_own = tks == 0 && more, cl2 = q2 && tk2s == 0, ws = !shk && tk + 64u >= nfix;
                const unsigned tkv1 = ATOMV(rl, cl_own ? (unsigned)head : OOR, 1);
                const unsigned tkv2 = ATOMV(rl, cl2 ? (unsigned)head + 128u : OOR, 1);
                const bool nw = nk && !(nrdy && (nfin || nk != 4)), w2 = h2 && !r2, f4 = nw && nk == 4;
                const unsigned poff = (lm0 & (nw ? ndep : OOR)) | (lm1 & (f4 ? ndep + 64u : OOR)) | (lm2 & (w2 ? (unsigned)(L_TILE + tl2) * 128u : OOR)) | (lm3 & ((w2 && !(known & 8u)) ? (unsigned)(L_START * 128) : OOR)) |
                                      (lm4 & (f4 ? ndep + 32u : OOR)) | (lm5 & (f4 ? (unsigned)(L_ALLFIN * 128) : OOR)) | (lm6 & (ws ? (unsigned)(L_SHARE + e) * 128u : OOR)) | lmx;
                const unsigned pv = __builtin_amdgcn_raw_buffer_load_b32(rl, poff, 0, 17);
                __builtin_amdgcn_sched_barrier(0);
                (void)ATOMV(rq, (qsig && UNI(qv) == qsegn * call - 1u) ? qsig : OOR, qsegn);
                const bool mark = !msent && tk + (q2 ? tk2 : (unsigned)Q2_ITEMS) + (unsigned)(e == 4 ? MARK4 : MARK6) >= nfix + (unsigned)Q2_ITEMS;
                __builtin_amdgcn_raw_buffer_store_b32(call, rq, l0 | (mark ? (unsigned)(L_MARK + e) * 128u : OOR), 0, 17);
                if (mark) msent = true;
                const __amdgpu_buffer_rsrc_t rd = __builtin_amdgcn_make_buffer_rsrc(ck >= 4 ? P.out[e] : P.ws[e], 0, ck >= 4 ? (int)OUT_BYTES : (int)WS_END, 0x00020000);
                unsigned dv[8], qoff = l0 | (psig ? (unsigned)scnt + pseg * 4u : OOR);
                const unsigned dv0 = (ck ? cdo : OOR) + (ck >= 4 ? rlane : llane), ds = ck >= 4 ? 32768u : ck == 1 ? 2048u : 1024u;
#pragma unroll
                for (int k = 0; k < 8; ++k) dv[k] = dv0 + k * ds;
                asm volatile("" : "+v"(dv[0]), "+v"(dv[1]), "+v"(dv[2]), "+v"(dv[3]), "+v"(dv[4]), "+v"(dv[5]), "+v"(dv[6]), "+v"(dv[7]), "+v"(qoff));
                __builtin_amdgcn_sched_barrier(0);
                unsigned one = 1u;
                asm volatile("" : "+v"(one) : "v"(pv) : "memory");
                qv = (unsigned)__builtin_amdgcn_raw_ptr_buffer_atomic_add_i32((int)one, rl, qoff, 0, 16);
                if (add2) {
#pragma unroll
                    for (int k = 0; k < 8; ++k) {
                        va[k] = (u32x4){add_bf16x2(va[k].x, vb[k].x), add_bf16x2(va[k].y, vb[k].y), add_bf16x2(va[k].z, vb[k].z), add_bf16x2(va[k].w, vb[k].w)};
                        if (!ok) va[k] = (u32x4){0x7fc07fc0u, 0x7fc07fc0u, 0x7fc07fc0u, 0x7fc07fc0u};
                    }
                }
#pragma unroll
                for (int k = 0; k < 8; ++k) __builtin_amdgcn_raw_buffer_store_b128(va[k], rd, dv[k], 0, 17  );
                __builtin_amdgcn_sched_barrier(0);
                qsig = psig; qsegn = psegn;
                if (pfin) ++mine;
                const unsigned okm = (unsigned)__ballot((int)(pv - ((lm0 & nneed) | pneed)) >= 0);
                known |= okm & (40u | ((nw && nk == 5) ? 1u : 0u));
                r2 = r2 || (w2 && (known & 8u) && (okm & 4u));
                const bool af = nk == 4 && (known & 32u), got = nw && (okm & 1u) && (!f4 || (okm & 2u));
                nrdy = nrdy || got || af;
                nfin = nfin || af || (got && (okm & 16u));
                if (ws) { const unsigned w = (unsigned)__builtin_amdgcn_readlane((int)pv, 6); if ((w >> 22) == tag) { shk = true; share = w; } }
                if (cl_own) { tk = UNI(tkv1); tks = 2; }
                if (cl2) { tk2 = UNI(tkv2); tk2s = 2; if (tk2 >= (unsigned)Q2_ITEMS) { tk2s = 0; q2 = false; STAMP(2); } }
                pfin = ck >= 4; psig = csig; pseg = cseg; psegn = csegn;
            }
            asm volatile("s_waitcnt vmcnt(0)" ::: "memory");
            if (pfin) ++mine;
            if (mine) (void)ATOM(rq, (L_DONE + e) * 128, mine, 16);
        }
        if (broker) {
            const unsigned long long tw = __builtin_amdgcn_s_memrealtime();
            unsigned seen_at = 0, first = 0; bool seen = (lane & 7) == 3;
            for (unsigned polls = 1; ok; ++polls) {
                const unsigned v = ld_relaxed_sys(ctl + (L_MARK + (lane & 7)) * 32), now = (unsigned)__builtin_amdgcn_s_memrealtime() | 1u;
                if (!seen) seen_at = now;
                seen = seen || (int)(v - call) >= 0;
                const unsigned sm = (unsigned)__ballot(seen) & 255u;
                if (sm != 8u && first == 0u) first = now;
                if (sm == 255u || (first && now - first > 2200u)) break;
                if ((polls & 63u) == 0u && gave_up(tw, tmo, call)) ok = false;
                __builtin_amdgcn_s_sleep(8);
            }
            float lag[8], mean = (float)POOL_ITEMS;
#pragma unroll
            for (int k = 0; k < 8; ++k) { lag[k] = (float)((unsigned)__builtin_amdgcn_readlane((int)seen_at, k) - first) * 0.0718f + (float)(k == 4 ? MARK4 : MARK6); if (k != 3) mean += lag[k]; }
            mean *= 1.f / 7.f;
            int pos = 0; unsigned w = 0;
#pragma unroll
            for (int k = 0; k < 8; ++k) {
                if (k == 3 || k == 4) continue;
                int n = (int)(mean - lag[k] + 0.5f);
                n = n < 0 ? 0 : n > FWD_MAX - pos ? FWD_MAX - pos : n;
                if (lane == k) w = (unsigned)pos << 11 | (unsigned)n;
                pos += n;
            }
            if (lane == 4) w = (unsigned)pos << 11 | (unsigned)(POOL_ITEMS - pos);
            if (lane < 8 && lane != 3) __hip_atomic_store((unsigned*)P.ws[lane] + (L_SHARE + lane) * 32, tag << 22 | w, __ATOMIC_RELAXED, __HIP_MEMORY_SCOPE_SYSTEM);
        }
        STAMP(3);
        __syncthreads();
        wg_exit(ctl, call, wid, lane);
        if (broker) {
            const unsigned long long tw = __builtin_amdgcn_s_memrealtime();
            for (unsigned polls = 1; ok; ++polls) {
                const int v = (int)ld_relaxed_sys(ctl + (L_DONE + (lane & 7)) * 32);
                unsigned sum = 0;
#pragma unroll
                for (int k = 0; k < 8; ++k) sum += (unsigned)__builtin_amdgcn_readlane(v, k);
                if ((int)(sum - 14336u * call) >= 0) break;
                if ((polls & 63u) == 0u && gave_up(tw, tmo, call)) ok = false;
                __builtin_amdgcn_s_sleep(2);
            }
        }
        if (!ok && lane == 0) *(unsigned short*)((char*)P.out[3] + out_offset(qa, TAB(hm * 128), 0)) = 0x7fc0;
        return;
    }

    const unsigned call = (unsigned)__builtin_amdgcn_readfirstlane(ld_relaxed_agent(ctl + L_CALL * 32)) + 1u;
    const int njobs = (int)P.njob[xl];
    for (int job = slot - CE_WGS, round = 0; job < njobs; ++round) {
        const unsigned jw = (P.job[(xl * MAXJ + job) >> 1] >> ((job & 1) * 16)) & 0xffffu;
        if (jw >> 8) job = gemm_job<true>(P, lds, myws, call, (int)(jw & 255u), (int)(jw >> 8) & 3, xl, round);
        else job = gemm_job<false>(P, lds, myws, call, (int)jw, 0, xl, round);
    }
    wg_exit(ctl, call, wid, lane);
}

}

extern "C" void kernel_launch(int rank, int nranks, ncclComm_t comm,
                              hipStream_t stream,
                              void* const* d_in, const int* in_sizes, int n_in,
                              void* d_out, int out_size,
                              const int* in_dtypes, int out_dtype,
                              void* const* d_out_all,
                              void* const* d_ws_all, size_t ws_bytes) {
    if (nranks != NR || (unsigned)rank >= (unsigned)NR || n_in < 2 || ws_bytes < WS_END || (size_t)out_size != (size_t)GM * GN ||
        (size_t)in_sizes[0] != (size_t)GM * KS || (size_t)in_sizes[1] != (size_t)KS * GN) {
        fprintf(stderr, "kernel_launch: unexpected problem shape\n");
        return;
    }
    const int member[2][4] = {{0, 2, 3, 6}, {1, 4, 5, 7}};
    Params P = {};
    P.x = d_in[0]; P.w = d_in[1]; P.g = 0; P.a = 0;
    for (int g = 0; g < 2; ++g) for (int a = 0; a < 4; ++a) if (member[g][a] == rank) { P.g = g; P.a = a; }
    for (int e = 0; e < 8; ++e) {
        const int peer = e < 3 ? member[P.g][P.a ^ (e + 1)] : (e == 3 ? rank : member[1 - P.g][P.a ^ (e - 4)]);
        P.out[e] = d_out_all[peer]; P.ws[e] = d_ws_all[peer];
    }
    unsigned char* tb = (unsigned char*)P.tab;
    unsigned short* jb = (unsigned short*)P.job;
    int n = 0, np[2] = {0, 0}, nx[8] = {0, 0, 0, 0, 0, 0, 0, 0};
    for (int k = 0; k < 16; ++k)
        for (int r = 0; r < 2 * k + 1; ++r, ++n) {
            const int I = r < k ? k : r - k, J = r < k ? r : k;
            const unsigned char tl = (unsigned char)(I << 4 | J);
            const int par = (I + J) & 1;
            tb[par * 128 + np[par]++] = tl;
            if (n < K_SMALL) { const int xs = n & 7; for (int q = 0; q < 4; ++q) jb[xs * MAXJ + nx[xs]++] = (unsigned short)(tl | (4 + q) << 8); }
            else { const int x = (I & 3) << 1 | ((J >> 1) & 1); jb[x * MAXJ + nx[x]++] = tl; }
        }
    for (int x = 0; x < 8; ++x) P.njob[x] = (unsigned)nx[x];
    hipLaunchKernelGGL(gemm_ar_kernel, dim3(GRID), dim3(512), 0, stream, P);
    HIP_CHECK(hipGetLastError());
}
```

```cpp
#include <hip/hip_runtime.h>
#include <rccl/rccl.h>
#include <cstdio>

#define HIP_CHECK(expr) do { hipError_t _e = (expr); if (_e != hipSuccess) { \
    fprintf(stderr, "HIP error %s at %s:%d\n", hipGetErrorString(_e), __FILE__, __LINE__); \
    return; } } while (0)

namespace {

#define LAS __attribute__((address_space(3)))
typedef short bf16x8 __attribute__((ext_vector_type(8)));
typedef short s16x4 __attribute__((ext_vector_type(4)));
typedef float f32x4 __attribute__((ext_vector_type(4)));
typedef unsigned u32x4 __attribute__((ext_vector_type(4)));

constexpr int NR = 8;
constexpr int GM = 8192, GN = 8192;
constexpr int KS = 1024;

#define K_PUSH 2
#define K_GRP 2
#define K_STW 4
#define K_ST1 2
#define K_STV 4
#define K_GATE 0
#define K_NA 36
#define K_NL 118
#define K_SMALL 49
#define K_SLEEP 8
#define K_CE 11
constexpr int NPUSH = K_PUSH, NA = K_NA, NL = K_NL;
static_assert(NA >= 1 && NA < NL && NL <= 128 && NPUSH >= 0 && NPUSH < 32, "knobs");

constexpr unsigned CTL_BYTES = 1u << 20;
constexpr unsigned XS_OFF = CTL_BYTES, XS_SLOT = 16u * 524288u;
constexpr unsigned WS_OFF = XS_OFF + 3u * XS_SLOT;
constexpr unsigned IN_OFF = WS_OFF + 3u * XS_SLOT;
constexpr unsigned PT_OFF = IN_OFF + 128u * 262144u;
constexpr unsigned RT_OFF = PT_OFF + 256u * 131072u;
constexpr unsigned FT_OFF = RT_OFF + 256u * 131072u;
constexpr unsigned WS_END = FT_OFF + 256u * 131072u;
constexpr unsigned OUT_BYTES = (unsigned)((size_t)GM * GN * 2);
constexpr unsigned IN_BYTES = (unsigned)((size_t)GM * KS * 2);

constexpr int L_TMO = 1, L_EXIT = 2, L_CALL = 3  , L_STG = 16  , L_STGC = 320  ,
              L_LAND = 32  , L_HEAD = 64  ,
              L_PANEL = 128  , L_FIN = 256  ,
              L_TILE = 512  , L_SCNT = 1024  ;
constexpr int CE_WGS = K_CE;
constexpr int G_WGS = 32 - CE_WGS;
constexpr int GRID = 256;
constexpr unsigned long long DEADLINE_TICKS = 50000000ull;
constexpr unsigned long long WATCH_DELAY = 1000;

constexpr int IN_ITEMS = 32 * 32;
constexpr int Q2_ITEMS = 128 * 16;
constexpr int FIX6 = IN_ITEMS + 128 * 16, FIX4 = NA * 16;
constexpr int POOL_ITEMS = (128 - NA) * 16, FWD_MAX = (NL - NA) * 16;
constexpr int Q3L_ITEMS = 128 * 16;
constexpr int STG_ITEMS = (32 - NPUSH) * 8;
constexpr int STG_GRP = K_GRP;
static_assert((32 - NPUSH) % STG_GRP == 0 && (32 - NPUSH) / STG_GRP <= 32, "staging groups");

constexpr int BK = 64, HALF = 128, HTB = HALF * BK * 2, STAGE_BYTES = 8 * HTB;

#define STAMP_TK do {} while (0)
#define STAMP(i) do {} while (0)
#define PROG(q, it) do {} while (0)

constexpr int MAXJ = 96;
static_assert((K_SMALL + 7) / 8 * 4 + 32 <= MAXJ, "job table");
struct Params {
    const void* x; const void* w;
    void* out[8]; void* ws[8];
    int g, a;
    unsigned njob[8], job[8 * MAXJ / 2];
    unsigned tab[64];
};
typedef const __attribute__((address_space(4))) Params CParams;
#define TAB(i) ((P.tab[(unsigned)(i) >> 2] >> (((unsigned)(i) & 3u) * 8u)) & 255u)
#define RSRC(p, n) __builtin_amdgcn_make_buffer_rsrc((void*)(p), 0, (int)(n), 0x00020000)
#define UNI(x) (unsigned)__builtin_amdgcn_readfirstlane((int)(x))

__device__ __forceinline__ int lds_byte(int r, int c) { const int st = (r >> 4) * 2 + (c >> 5), rr = r & 15, cc = c & 31, ob = rr * 64 + cc * 2; return st * 1024 + (ob ^ (((ob >> 9) & 1) << 5)); }
__device__ __forceinline__ void stage_rc(int b, int& R, int& C) { const int st = b / 1024, sb = b % 1024, swz = sb ^ (((sb >> 9) & 1) << 5); R = (st >> 1) * 16 + swz / 64; C = (st & 1) * 32 + (swz % 64) / 2; }
__device__ __forceinline__ unsigned wswz(unsigned row) { return ((row & 3u) << 2) | ((row >> 2) & 3u); }
typedef float f32x2 __attribute__((ext_vector_type(2)));
typedef __bf16 bf16x2 __attribute__((ext_vector_type(2)));
__device__ __forceinline__ unsigned cvt_pk_bf16(float lo, float hi) {
    return __builtin_bit_cast(unsigned, __builtin_convertvector((f32x2){lo, hi}, bf16x2));
}
__device__ __forceinline__ unsigned panel_base(int ab, int I) {
    const int j = I >> 1;
    return (unsigned)((j << 2) | ((2 * ab + (I & 1) + (0xb114 >> (2 * j))) & 3)) * 256u;
}
__device__ __forceinline__ unsigned out_offset(int a, unsigned tl, unsigned st) {
    return (panel_base(a >> 1, (int)(tl >> 4)) + st * 16u) * 16384u + panel_base(a & 1, (int)(tl & 15u)) * 2u;
}

__device__ __forceinline__ unsigned ld_relaxed_sys(const unsigned* p) { return __hip_atomic_load(p, __ATOMIC_RELAXED, __HIP_MEMORY_SCOPE_SYSTEM); }
__device__ __forceinline__ unsigned ld_relaxed_agent(const unsigned* p) { return __hip_atomic_load(p, __ATOMIC_RELAXED, __HIP_MEMORY_SCOPE_AGENT); }

__device__ __forceinline__ bool gave_up(unsigned long long t0, unsigned* tmo, unsigned call) {
    if ((unsigned)__builtin_amdgcn_readfirstlane(ld_relaxed_agent(tmo)) == call || __builtin_amdgcn_s_memrealtime() - t0 > DEADLINE_TICKS) {
        __hip_atomic_store(tmo, call, __ATOMIC_RELAXED, __HIP_MEMORY_SCOPE_AGENT);
        return true;
    }
    return false;
}
__device__ __forceinline__ unsigned add_bf16x2(unsigned a, unsigned b) {
    return cvt_pk_bf16(__uint_as_float(a << 16) + __uint_as_float(b << 16), __uint_as_float(a & 0xffff0000u) + __uint_as_float(b & 0xffff0000u));
}

__device__ __forceinline__ void wg_exit(unsigned* ctl, unsigned call, int wid, int lane) {
    if (wid != 0) return;
    const unsigned t = (unsigned)__builtin_amdgcn_raw_ptr_buffer_atomic_add_i32(1, __builtin_amdgcn_make_buffer_rsrc(ctl, 0, (int)CTL_BYTES, 0x00020000), lane == 0 ? (unsigned)L_EXIT * 128u : 0x7fffff00u, 0, 0);
    if (((unsigned)__builtin_amdgcn_readfirstlane((int)t) & (unsigned)(GRID - 1)) == (unsigned)(GRID - 1)) {
        if (lane < 34) __hip_atomic_store(ctl + (lane == 33 ? L_CALL : L_HEAD + lane) * 32, lane == 33 ? call : 0u, __ATOMIC_RELAXED, __HIP_MEMORY_SCOPE_AGENT);
    }
}

template <bool SMALL>
__device__ __forceinline__ int gemm_job(CParams& P, LAS unsigned char* lds, char* myws, unsigned call, int tl, int qd, int xl, int round) {
    int tid = threadIdx.x; asm volatile("" : "+v"(tid));
    const int wid = __builtin_amdgcn_readfirstlane(tid >> 6), lane = tid & 63;
    const int a1 = P.a >> 1, a0 = P.a & 1, ti = tl >> 4, tj = tl & 15;
    unsigned* ctl = (unsigned*)myws;
    unsigned* tmo = ctl + L_TMO * 32;
    const int wr = wid >> 2, wc = wid & 3, fr = lane & 15, fq = lane >> 4;
    unsigned voffA[2], voffB[2];
#pragma unroll
    for (int i = 0; i < 2; ++i) { int R, C; stage_rc(tid * 16 + i * 8192, R, C); voffA[i] = (unsigned)(R * KS + C) * 2u; }
#define SET_VOFFB(sh) do { int tz = tid; asm volatile("" : "+v"(tz)); _Pragma("unroll") for (int i = 0; i < 2; ++i) { \
        const unsigned row = (unsigned)(tz >> 4) + 32u * i, ch = ((unsigned)tz & 15u) ^ wswz(row); voffB[i] = (row << (sh)) + ch * 16u; } \
        kstepB = (size_t)(SMALL ? 2 * BK : BK) << (sh); hstepB = SMALL ? (size_t)BK << (sh) : (size_t)HALF * 2; } while (0)
    size_t kstepB, hstepB;
    constexpr size_t kstepA = SMALL ? BK * 4 : BK * 2, hstepA = SMALL ? BK * 2 : (size_t)HALF * KS * 2;
    const unsigned qa = SMALL ? (unsigned)(qd >> 1) * (unsigned)(HALF * KS * 2) : 0u, qb = SMALL ? (unsigned)(qd & 1) * 256u : 0u;
    const unsigned ldsw = (unsigned)wid * 1024u;
    const int aoff = lds_byte(wr * 64 + fr, fq * 8);
    int boff[2][2];
#pragma unroll
    for (int n = 0; n < 2; ++n)
#pragma unroll
        for (int t = 0; t < 2; ++t) {
            const unsigned q4 = (unsigned)(lane & 15) >> 2, p4 = (unsigned)lane & 3u, row = 8u * fq + 4u * t + q4, ch = 4u * wc + 2u * n + (p4 >> 1);
            boff[n][t] = (int)((unsigned)(size_t)lds + 4u * HTB + 256u * row + 16u * (ch ^ wswz(row)) + 8u * (p4 & 1u));
        }
#define SA(b, h) (((b) * 2 + (h)) * HTB)
#define SB(b, h) ((4 + (b) * 2 + (h)) * HTB)
#define STAGE(bufoff, gbase, voff) do { _Pragma("unroll") for (int _i = 0; _i < 2; ++_i) \
        __builtin_amdgcn_global_load_lds((const unsigned*)((const char*)(gbase) + (voff)[_i]), (LAS unsigned*)(lds + (bufoff) + ldsw + _i * 8192), 16, 0, 0); } while (0)
#define LDA(dst, b, h) do { _Pragma("unroll") for (int m = 0; m < 4; ++m) _Pragma("unroll") for (int k = 0; k < 2; ++k) dst[m][k] = *(const LAS bf16x8*)(lds + SA(b, h) + aoff + m * 2048 + k * 1024); } while (0)
#define TRR(dst, base, off) asm volatile("ds_read_b64_tr_b16 %0, %1 offset:%2" : "=&v"(dst) : "v"(base), "i"(off) : "memory")
#define LDB(raw, b, h) do { \
        TRR(raw[0], boff[0][0], SB(b, h) - 4 * HTB); TRR(raw[1], boff[0][1], SB(b, h) - 4 * HTB); TRR(raw[2], boff[0][0], SB(b, h) - 4 * HTB + 8192); TRR(raw[3], boff[0][1], SB(b, h) - 4 * HTB + 8192); \
        TRR(raw[4], boff[1][0], SB(b, h) - 4 * HTB); TRR(raw[5], boff[1][1], SB(b, h) - 4 * HTB); TRR(raw[6], boff[1][0], SB(b, h) - 4 * HTB + 8192); TRR(raw[7], boff[1][1], SB(b, h) - 4 * HTB + 8192); } while (0)
#define WAIT_B(raw, dst) do { asm volatile("s_waitcnt lgkmcnt(0)" : "+v"(raw[0]), "+v"(raw[1]), "+v"(raw[2]), "+v"(raw[3]), "+v"(raw[4]), "+v"(raw[5]), "+v"(raw[6]), "+v"(raw[7]) :: "memory"); \
        _Pragma("unroll") for (int n = 0; n < 2; ++n) _Pragma("unroll") for (int k = 0; k < 2; ++k) dst[n][k] = __builtin_shufflevector(raw[n * 4 + k * 2], raw[n * 4 + k * 2 + 1], 0, 1, 2, 3, 4, 5, 6, 7); } while (0)
#define MMA(ai, bj, At, Bt) do { if (!SMALL || (ai) == (bj)) { __builtin_amdgcn_s_setprio(1); _Pragma("unroll") for (int m = 0; m < 4; ++m) _Pragma("unroll") for (int n = 0; n < 2; ++n) _Pragma("unroll") for (int k = 0; k < 2; ++k) \
        acc[SMALL ? 0 : ai][SMALL ? 0 : bj][m][n] = __builtin_amdgcn_mfma_f32_16x16x32_bf16(Bt[n][k], At[m][k], acc[SMALL ? 0 : ai][SMALL ? 0 : bj][m][n], 0, 0, 0); __builtin_amdgcn_s_setprio(0); } } while (0)
#define WAIT_V(n) asm volatile("s_waitcnt vmcnt(" #n ")" ::: "memory")
#define WAIT_L(n) asm volatile("s_waitcnt lgkmcnt(" #n ")" ::: "memory")
#define BAR __builtin_amdgcn_s_barrier()
#define SCHED __builtin_amdgcn_sched_barrier(0)
    constexpr int NH2 = SMALL ? 1 : 2;
    f32x4 acc[NH2][NH2][4][2];
#pragma unroll
    for (int a = 0; a < NH2; ++a)
#pragma unroll
        for (int b = 0; b < NH2; ++b)
#pragma unroll
            for (int m = 0; m < 4; ++m)
#pragma unroll
                for (int n = 0; n < 2; ++n) acc[a][b][m][n] = (f32x4){0.f, 0.f, 0.f, 0.f};
    bf16x8 At[4][2], B0[2][2], B1[2][2];
    s16x4 R0[8], R1[8];
    const char* cA = (const char*)P.x + (size_t)panel_base(a1, ti) * (KS * 2) + qa;
    const char* cB = (const char*)P.w + (size_t)panel_base(a0, tj) * 2 + qb;
    constexpr int nt = SMALL ? KS / BK / 2 : KS / BK;
    unsigned todo = 7u;
    SET_VOFFB(14);
    STAGE(SB(0, 0), cB, voffB); STAGE(SA(0, 0), cA, voffA); STAGE(SB(0, 1), cB + hstepB, voffB); STAGE(SA(0, 1), cA + hstepA, voffA);
    if (wr == 1) BAR;
    WAIT_V(4); BAR;
    STAGE(SB(1, 0), cB + kstepB, voffB); STAGE(SA(1, 0), cA + kstepA, voffA); STAGE(SB(1, 1), cB + hstepB + kstepB, voffB);
    WAIT_V(6); BAR;
    for (int s = 0; s < 4; ++s) {
        const char* nA = cA; const char* nB = cB;
        for (int t = 0; t < nt; t += 2) {
            const bool last = (t == nt - 2);
            if (last && s < 3) {
                if (wid == 0) {
                    STAMP(1 + 2 * s + 8 * round);
                    const unsigned* word = ctl + (L_PANEL + ((lane >> 1) % 3) * 32 + ((lane & 1) ? 2 * tj + 1 : 2 * ti)) * 32;
                    const unsigned long long t0 = __builtin_amdgcn_s_memrealtime();
                    unsigned pick;
                    for (unsigned polls = 1;; ++polls) {
                        const unsigned mk = (unsigned)__ballot((int)(ld_relaxed_sys(word) - 64u * call) >= 0);
                        const unsigned both = mk & (mk >> 1);
                        pick = ((both & 1u) | ((both >> 1) & 2u) | ((both >> 2) & 4u)) & todo;
                        if (pick) break;
                        if ((polls & 63u) == 0u && gave_up(t0, tmo, call)) { pick = todo; break; }
                        __builtin_amdgcn_s_sleep(4);
                    }
                    STAMP(2 + 2 * s + 8 * round);
                    *(LAS unsigned*)(lds + STAGE_BYTES + 16) = (unsigned)__builtin_ctz(pick);
                    __builtin_amdgcn_fence(__ATOMIC_ACQUIRE, "");
                    asm volatile("s_waitcnt vmcnt(0) lgkmcnt(0)" ::: "memory");
                }
                asm volatile("" ::: "memory"); BAR; asm volatile("" ::: "memory");
                unsigned slv;
                asm volatile("ds_read_b32 %0, %1\n\ts_waitcnt lgkmcnt(0)" : "=&v"(slv) : "v"((unsigned)(size_t)lds + STAGE_BYTES + 16) : "memory");
                const unsigned sl = (unsigned)__builtin_amdgcn_readfirstlane(slv) & 3u;
                todo &= ~(1u << sl);
                nA = myws + XS_OFF + sl * XS_SLOT + (unsigned)ti * 524288u + qa;
                nB = myws + WS_OFF + sl * XS_SLOT + (unsigned)tj * 524288u + qb;
                if (s == 0) SET_VOFFB(9);
            }
            const char* a1p = cA + (size_t)(t + 1) * kstepA;
            const char* a2 = last ? nA : cA + (size_t)(t + 2) * kstepA; const char* b2 = last ? nB : cB + (size_t)(t + 2) * kstepB;
            const char* a3 = a2 + kstepA; const char* b3 = b2 + kstepB;
            LDB(R0, 0, 0); SCHED; LDA(At, 0, 0); STAGE(SA(1, 1), a1p + hstepA, voffA);
            WAIT_L(8); BAR; WAIT_B(R0, B0); MMA(0, 0, At, B0); BAR; SCHED;
            LDB(R1, 0, 1); STAGE(SB(0, 0), b2, voffB);
            BAR; WAIT_B(R1, B1); MMA(0, 1, At, B1); BAR; if (SMALL) SCHED;
            LDA(At, 0, 1); STAGE(SA(0, 0), a2, voffA);
            BAR; WAIT_L(0); MMA(1, 0, At, B0); BAR; SCHED;
            STAGE(SB(0, 1), b2 + hstepB, voffB);
            WAIT_V(6); BAR; MMA(1, 1, At, B1); BAR;
            LDB(R0, 1, 0); SCHED; LDA(At, 1, 0); STAGE(SA(0, 1), a2 + hstepA, voffA);
            WAIT_L(8); BAR; WAIT_B(R0, B0); MMA(0, 0, At, B0); BAR; SCHED;
            LDB(R1, 1, 1); STAGE(SB(1, 0), b3, voffB);
            BAR; WAIT_B(R1, B1); MMA(0, 1, At, B1); BAR; if (SMALL) SCHED;
            LDA(At, 1, 1); STAGE(SA(1, 0), a3, voffA);
            BAR; WAIT_L(0); MMA(1, 0, At, B0); BAR; SCHED;
            STAGE(SB(1, 1), b3 + hstepB, voffB);
            WAIT_V(6); BAR; MMA(1, 1, At, B1); BAR;
        }
        cA = nA; cB = nB;
    }
    WAIT_V(0);
    if (wr == 0) BAR;
    STAMP(7 + 8 * round);
    const __amdgpu_buffer_rsrc_t rc = __builtin_amdgcn_make_buffer_rsrc(myws, 0, (int)CTL_BYTES, 0x00020000);
    const unsigned nj = (unsigned)__builtin_amdgcn_raw_ptr_buffer_atomic_add_i32(1, rc, tid == 0 ? (unsigned)(L_HEAD + 16 + xl) * 128u : 0x7fffff00u, 0, 0);
    const bool bad = (unsigned)__builtin_amdgcn_readfirstlane(ld_relaxed_agent(tmo)) == call;
    const __amdgpu_buffer_rsrc_t ro = __builtin_amdgcn_make_buffer_rsrc(myws + PT_OFF + (unsigned)tl * 131072u, 0, 131072, 0x00020000);
    const unsigned obase = (unsigned)(64 * wr + fr + (SMALL ? (qd >> 1) * HALF : 0)) * 512u + (unsigned)(32 * wc + (fq & 1) * 16 + (fq >> 1) * 8) * 2u + qb;
#pragma unroll
    for (int ai = 0; ai < NH2; ++ai)
#pragma unroll
        for (int m = 0; m < 4; ++m)
#pragma unroll
            for (int bj = 0; bj < NH2; ++bj) {
                const f32x4 v0 = acc[ai][bj][m][0], v1 = acc[ai][bj][m][1];
                const unsigned x0 = cvt_pk_bf16(v0[0], v0[1]), x1 = cvt_pk_bf16(v0[2], v0[3]), y0 = cvt_pk_bf16(v1[0], v1[1]), y1 = cvt_pk_bf16(v1[2], v1[3]);
                const auto r0 = __builtin_amdgcn_permlane16_swap(x0, y0, false, false);
                const auto r1 = __builtin_amdgcn_permlane16_swap(x1, y1, false, false);
                u32x4 wv = {r0[0], r1[0], r0[1], r1[1]};
                if (bad) wv = (u32x4){0x7fc07fc0u, 0x7fc07fc0u, 0x7fc07fc0u, 0x7fc07fc0u};
                __builtin_amdgcn_raw_buffer_store_b128(wv, ro, obase + (unsigned)(ai * HALF + m * 16) * 512u + (unsigned)bj * 256u, 0, 17  );
            }
    if (wid == 0) *(LAS unsigned*)(lds + STAGE_BYTES) = (unsigned)__builtin_amdgcn_readfirstlane((int)nj);
    asm volatile("s_waitcnt vmcnt(0) lgkmcnt(0)" ::: "memory");
    BAR;
    const bool mine = ((ti + tj) & 1) == P.g;
    (void)__builtin_amdgcn_raw_ptr_buffer_atomic_add_i32(SMALL ? 2 : 8, __builtin_amdgcn_make_buffer_rsrc(P.ws[mine ? 3 : 4], 0, (int)CTL_BYTES, 0x00020000),
                                                         tid == 0 ? (unsigned)(L_TILE + tl) * 128u + (mine ? 0u : 32u) : 0x7fffff00u, 0, 16);
    STAMP(8 + 8 * round);
    unsigned njv;
    asm volatile("ds_read_b32 %0, %1\n\ts_waitcnt lgkmcnt(0)" : "=&v"(njv) : "v"((unsigned)(size_t)lds + STAGE_BYTES) : "memory");
    return __builtin_amdgcn_readfirstlane((int)njv) + G_WGS;
}

__device__ __forceinline__ unsigned in_offset(unsigned t, int ab, int kh, unsigned u) {
    return (t & 1u) ? ((unsigned)kh * 512u + u * 16u) * 16384u + panel_base(ab, (int)(t >> 1)) * 2u : (panel_base(ab, (int)(t >> 1)) + u * 8u) * 2048u + (unsigned)kh * 1024u;
}
__device__ __forceinline__ void stage_inputs(CParams& P, char* myws, unsigned call, int lb, int lane, int maxn) {
    const __amdgpu_buffer_rsrc_t rl = RSRC(myws, WS_END);
    const unsigned l0 = lane == 0 ? 0u : 0x7fffff00u, llane = (unsigned)lane * 16u, rlane = (unsigned)(lane >> 5) * 16384u + (unsigned)(lane & 31) * 16u;
    for (; maxn > 0; --maxn) {
        const unsigned it = UNI(__builtin_amdgcn_raw_ptr_buffer_atomic_add_i32(1, rl, l0, (L_HEAD + 24 + lb) * 128, 0));
        if (it >= (unsigned)STG_ITEMS) break;
        const unsigned pc = (it * 8u + (unsigned)lb) * 2u + (unsigned)NPUSH * 128u, t = pc >> 7, g = (t - (unsigned)NPUSH) / (unsigned)STG_GRP;
        const unsigned so = in_offset(t, (int)(pc >> 6) & 1, (int)(pc >> 5) & 1, pc & 31u) + ((t & 1u) ? rlane : llane), ss = (t & 1u) ? 32768u : 2048u;
        const __amdgpu_buffer_rsrc_t rs = RSRC((t & 1u) ? (char*)P.w : (char*)P.x, IN_BYTES);
        u32x4 v[16];
#pragma unroll
        for (int k = 0; k < 16; ++k) v[k] = __builtin_amdgcn_raw_buffer_load_b128(rs, so + k * ss, 0, 0);
#pragma unroll
        for (int k = 0; k < 16; ++k) __builtin_amdgcn_raw_buffer_store_b128(v[k], rl, IN_OFF + pc * 8192u + llane + k * 1024u, 0, 17);
        asm volatile("s_waitcnt vmcnt(0)" ::: "memory");
        if (UNI(__builtin_amdgcn_raw_ptr_buffer_atomic_add_i32(1, rl, l0, (L_STGC + (int)g) * 128, 0)) == (unsigned)(STG_GRP * 64) * call - 1u) {
            if (lane < 8 && lane != 3 && lane != 4) __hip_atomic_store((unsigned*)P.ws[lane] + (L_STG + lane) * 32 + g, call, __ATOMIC_RELAXED, __HIP_MEMORY_SCOPE_SYSTEM);
        }
    }
}

__global__ __launch_bounds__(512, 2) void gemm_ar_kernel(const Params) {
    CParams& P = *(CParams*)__builtin_amdgcn_kernarg_segment_ptr();
    __shared__ __attribute__((aligned(16))) unsigned char shm[STAGE_BYTES + 64];
    LAS unsigned char* lds = (LAS unsigned char*)shm;
    const int tid = threadIdx.x, wid = __builtin_amdgcn_readfirstlane(tid >> 6), lane = tid & 63;
    const int hm = P.g, qa = P.a;
    const int xl = blockIdx.x & 7, slot = blockIdx.x >> 3;
    char* myws = (char*)P.ws[3];
    unsigned* ctl = (unsigned*)myws;
    unsigned* tmo = ctl + L_TMO * 32;
    STAMP(0);

    if (slot < CE_WGS) {
        int e = (int)((unsigned)__builtin_amdgcn_s_getreg(20 | (3 << 11)) & 7u);
        bool active = wid == 0 || e == 3;
        const __amdgpu_buffer_rsrc_t rl = RSRC(myws, WS_END);
        const unsigned l0 = lane == 0 ? 0u : 0x7fffff00u;
#define ATOM(rs, off, v, aux) (unsigned)__builtin_amdgcn_raw_ptr_buffer_atomic_add_i32((int)(v), rs, l0, off, aux)
#define ATOMV(rs, off, v) (unsigned)__builtin_amdgcn_raw_ptr_buffer_atomic_add_i32((int)(v), rs, l0 | (off), 0, 16)
        if (wid == 7) {
            const unsigned long long t0 = __builtin_amdgcn_s_memrealtime();
            while (__builtin_amdgcn_s_memrealtime() - t0 < WATCH_DELAY) __builtin_amdgcn_s_sleep(16);
            if (UNI(ld_relaxed_agent(ctl + (L_HEAD + xl * 2) * 32)) == 0u) { e = xl; active = true; }
        }
        const unsigned llane = (unsigned)lane * 16u;
        const unsigned rlane = (unsigned)(lane >> 5) * 16384u + (unsigned)(lane & 31) * 16u;
        const int head = (L_HEAD + e * 2) * 128, scnt = (L_SCNT + e * 8) * 128;
        unsigned tkv = 0, call = 0;
        if (active) { tkv = ATOM(rl, head, e == 3 ? 1 : 2, 0); call = UNI(__builtin_amdgcn_raw_buffer_load_b32(rl, L_CALL * 128u, 0, 16)) + 1u; }
        STAMP(4);
        bool ok = true;
        unsigned idle = 0; unsigned long long t0 = 0;
        if (active && e == 3) {
            if (wid == 7) for (int k = 0; k < 8; ++k) stage_inputs(P, myws, call, (slot + k) & 7, lane, 1 << 20);
            const __amdgpu_buffer_rsrc_t ro = RSRC(P.out[3], OUT_BYTES);
            for (unsigned it = UNI(tkv); it < (unsigned)Q3L_ITEMS; it = UNI(tkv)) {
                const unsigned tl = TAB(hm * 128 + (it >> 4)), st = it & 15u, dep = (unsigned)(L_TILE + tl) * 128u;
                t0 = __builtin_amdgcn_s_memrealtime();
                for (idle = 1; ok; ++idle) {
                    const unsigned v = __builtin_amdgcn_raw_buffer_load_b32(rl, dep + (unsigned)(lane & 1) * 64u, 0, 17);
                    if (__all((int)(v - ((lane & 1) ? 16u : 8u) * call) >= 0)) break;
                    if ((idle & 63u) == 0u && gave_up(t0, tmo, call)) ok = false;
                    __builtin_amdgcn_s_sleep(K_SLEEP);
                }
                __builtin_amdgcn_fence(__ATOMIC_ACQUIRE, "wavefront");
                PROG(2, it);
                const unsigned sv = tl * 131072u + st * 8192u + llane, dv = out_offset(qa, tl, st) + rlane;
                u32x4 va[8], vb[8];
#pragma unroll
                for (int k = 0; k < 8; ++k) { va[k] = __builtin_amdgcn_raw_buffer_load_b128(rl, PT_OFF + sv + k * 1024u, 0, 17); vb[k] = __builtin_amdgcn_raw_buffer_load_b128(rl, RT_OFF + sv + k * 1024u, 0, 17); }
#pragma unroll
                for (int k = 0; k < 8; ++k) {
                    u32x4 o = {add_bf16x2(va[k].x, vb[k].x), add_bf16x2(va[k].y, vb[k].y), add_bf16x2(va[k].z, vb[k].z), add_bf16x2(va[k].w, vb[k].w)};
                    if (!ok) o = (u32x4){0x7fc07fc0u, 0x7fc07fc0u, 0x7fc07fc0u, 0x7fc07fc0u};
                    __builtin_amdgcn_raw_buffer_store_b128(o, rl, FT_OFF + sv + k * 1024u, 0, 17);
                    __builtin_amdgcn_raw_buffer_store_b128(o, ro, dv + k * 32768u, 0, 16);
                }
                tkv = ATOM(rl, head, 1, 0);
                asm volatile("s_waitcnt vmcnt(0)" ::: "memory");
                if ((UNI(ATOM(rl, scnt + (int)(it >> 4) * 4, 1, 0)) & 15u) == 15u) {
                    if (lane < 8 && lane != 3) __hip_atomic_store((unsigned*)P.ws[lane] + (L_FIN + lane * 4) * 32 + (it >> 4), call, __ATOMIC_RELAXED, __HIP_MEMORY_SCOPE_SYSTEM);
                }
            }
        } else if (active) {
            const int x = e < 3 ? e + 1 : e - 4, hp = e < 3 ? hm : hm ^ 1, qp = qa ^ x;
            const unsigned nfix = e == 4 ? (unsigned)FIX4 : (unsigned)FIX6;
            constexpr unsigned OOR = 0x7fff0000u;
            bool more = true, spare = true, pool = false, tkpool = false;
            int tks = 2, tk2s = 0;
            unsigned tk = UNI(tkv), tk2 = 0;
            bool q2 = e == 4;
            int nk = 0; unsigned nso = 0, ndo = 0, ndep = OOR, nneed = 0, nsig = 0, nseg = 0, nsegn = 1; bool nrdy = false;
            bool h2 = false, r2 = false; unsigned p2 = 0, tl2 = 0;
            unsigned psig = 0, pseg = 0, psegn = 1, qsig = 0, qsegn = 1, qv = 0;
            const unsigned lm0 = lane == 0 ? ~0u : 0u, lm1 = lane == 1 ? ~0u : 0u, lm2 = lane == 2 ? ~0u : 0u, lmx = lane > 2 ? OOR : 0u;
            bool gate = K_GATE >= 0 && e != 4;
#define REFILL() do { \
                if (nk == 0 && tks == 2) { \
                    tks = 0; nsig = 0; nseg = 0; nneed = call; \
                    if (!tkpool && tk >= nfix) pool = true; \
                    else if (!tkpool && e != 4 && tk < (unsigned)IN_ITEMS) {       \
                        const unsigned t = tk >> 5, u = tk & 31u, half = e < 3 ? 0u : 1u; const bool push = t < (unsigned)NPUSH; \
                        nk = (push ? 1 : 3) + (int)(t & 1u); \
                        nso = push ? in_offset(t, (t & 1u) ? qp & 1 : qp >> 1, hp, u) : IN_OFF + (((t * 2u + (unsigned)((t & 1u) ? qa & 1 : qa >> 1)) * 2u + (unsigned)hm) * 32u + u) * 8192u; \
                        ndo = (unsigned)(x - 1) * XS_SLOT + (t >> 1) * 524288u + ((t & 1u) ? WS_OFF + half * 262144u + u * 8192u : XS_OFF + u * 16384u + half * 1024u); \
                        nseg = t; nsegn = 32u; nsig = (unsigned)(L_PANEL + (x - 1) * 32 + (int)t) * 128u + (push ? (unsigned)e : 3u); \
                        ndep = (unsigned)(L_STG + e) * 128u + (push ? 0u : (t - (unsigned)NPUSH) / (unsigned)STG_GRP * 4u); nrdy = push || !ok; \
                    } else { \
                        unsigned r; bool none = false;                 \
                        const bool fwd = tkpool && e != 4; \
                        if (tkpool) { const unsigned lo = tk & 0xffffu, hi = tk >> 16, loc = lo < (unsigned)FWD_MAX ? lo : (unsigned)FWD_MAX; \
                            none = loc + hi >= (unsigned)POOL_ITEMS || (fwd && lo >= (unsigned)FWD_MAX); r = none ? 0u : fwd ? (unsigned)(NA * 16) + lo : 2047u - hi; } \
                        else if (e == 4) r = tk; \
                        else { r = tk - (unsigned)IN_ITEMS; if (r < (unsigned)(NL * 16)) { nseg = 64u; nsegn = (unsigned)(NL * 16); nsig = (unsigned)(L_LAND + e2) * 128u + (unsigned)e2; } } \
                        const unsigned tl = TAB((fwd ? hm ^ 1 : hp) * 128 + (int)(r >> 4)), st = r & 15u; \
                        nk = none ? 0 : fwd ? 6 : 5; \
                        ndo = out_offset(fwd ? qa : qp, tl, st); \
                        nso = fwd ? ndo : FT_OFF + tl * 131072u + st * 8192u; \
                        ndep = fwd ? (unsigned)(L_LAND + e) * 128u : (unsigned)(L_FIN + e * 4) * 128u + (r >> 4) * 4u; \
                        nneed = fwd ? (unsigned)(NL * 16) * call : call; \
                        nrdy = !ok; \
                        if (none) { more = false; STAMP(29); } \
                    } \
                    STAMP_TK; \
                    if (spare) { spare = false; ++tk; tks = 2; } \
                } \
                if (!h2 && tk2s == 2) { tk2s = 0; h2 = true; p2 = tk2; tl2 = TAB(hm * 128 + (int)(p2 >> 4)); r2 = !ok; } \
            } while (0)
            const int e2 = e < 3 ? 4 + x : x - 1;
            REFILL();
            for (;;) {
                int ck = 0; unsigned cso = 0, cdo = 0, csig = 0, cseg = 0, csegn = 1;
                if (h2 && r2) {
                    const unsigned o = tl2 * 131072u + (p2 & 15u) * 8192u;
                    ck = 4; cso = PT_OFF + o; cdo = RT_OFF + o; csig = (unsigned)(L_TILE + tl2) * 128u + 64u + 3u; cseg = 128u + (p2 >> 4); csegn = 16u; h2 = false;
                    PROG(1, p2);
                } else if (nk && nrdy) {
                    ck = nk; cso = nso; cdo = ndo; csig = nsig; cseg = nseg & 255u; csegn = nsegn; nk = 0;
                    PROG((nseg >> 28) ? 2 : 0, (nseg >> 8) & 0xfffffu);
                }
                if (ck == 0 && nk == 0 && !more && !h2 && tk2s == 0 && !q2 && psig == 0u && qsig == 0u) break;
                __builtin_amdgcn_fence(__ATOMIC_ACQUIRE, "wavefront");
                u32x4 va[8];
                {
                    const __amdgpu_buffer_rsrc_t rs = RSRC(ck == 1 ? (char*)P.x : ck == 2 ? (char*)P.w : ck == 6 ? (char*)P.out[e] : (char*)P.ws[e], ck == 6 ? OUT_BYTES : ck >= 3 ? WS_END : IN_BYTES);
                    const bool sr = ck == 2 || ck == 6;
                    const unsigned sv = (ck ? cso : OOR) + (sr ? rlane : llane), ss = sr ? 32768u : ck == 1 ? 2048u : 1024u;
#pragma unroll
                    for (int k = 0; k < 8; ++k) va[k] = __builtin_amdgcn_raw_buffer_load_b128(rs, sv + k * ss, 0, 17  );
                }
                __builtin_amdgcn_sched_barrier(0);
                REFILL();
                if (ck) idle = 0;
                else {
                    if (idle == 0u) t0 = __builtin_amdgcn_s_memrealtime(); else __builtin_amdgcn_s_sleep(K_SLEEP);
                    if ((++idle & 63u) == 0u && ok && gave_up(t0, tmo, call)) { ok = false; nrdy = true; r2 = true; }
                }
                (void)ATOMV(RSRC(P.ws[qsig & 7u], CTL_BYTES), (qsig && UNI(qv) == qsegn * call - 1u) ? qsig & ~7u : OOR, qsegn);
                __builtin_amdgcn_sched_barrier(0);
                const bool cl1 = tks == 0 && more, cl2 = q2 && tk2s == 0, nw = nk && !nrdy, w2 = h2 && !r2;
                const unsigned tkv1 = ATOMV(rl, cl1 ? (pool ? (unsigned)(L_HEAD + 32) * 128u : (unsigned)head) : OOR, (pool && e == 4) ? 0x10000u : 1u);
                const unsigned tkv2 = ATOMV(rl, cl2 ? (unsigned)head + 128u : OOR, 1);
                const unsigned pv = __builtin_amdgcn_raw_buffer_load_b32(rl, (lm0 & (nw ? ndep : OOR)) | (lm1 & (w2 ? (unsigned)(L_TILE + tl2) * 128u + 32u : OOR)) | (lm2 & (gate ? (unsigned)(L_PANEL + (x - 1) * 32 + K_GATE) * 128u : OOR)) | lmx, 0, 17);
                const __amdgpu_buffer_rsrc_t rd = RSRC(ck >= 5 ? (char*)P.out[3] : ck >= 3 ? myws : (char*)P.ws[e], ck >= 5 ? OUT_BYTES : WS_END);
                unsigned dv[8], qoff = l0 | (psig ? (unsigned)scnt + pseg * 4u : OOR);
                const unsigned dv0 = (ck ? cdo : OOR) + (ck >= 5 ? rlane : llane), ds = ck >= 5 ? 32768u : (ck & 1) ? 2048u : 1024u;
#pragma unroll
                for (int k = 0; k < 8; ++k) dv[k] = dv0 + k * ds;
                asm volatile("" : "+v"(dv[0]), "+v"(dv[1]), "+v"(dv[2]), "+v"(dv[3]), "+v"(dv[4]), "+v"(dv[5]), "+v"(dv[6]), "+v"(dv[7]), "+v"(qoff));
                __builtin_amdgcn_sched_barrier(0);
                unsigned one = 1u;
                asm volatile("s_waitcnt vmcnt(0)" : "+v"(one) : "v"(pv) : "memory");
                qv = (unsigned)__builtin_amdgcn_raw_ptr_buffer_atomic_add_i32((int)one, rl, qoff, 0, 16);
                if (!ok) {
#pragma unroll
                    for (int k = 0; k < 8; ++k) va[k] = (u32x4){0x7fc07fc0u, 0x7fc07fc0u, 0x7fc07fc0u, 0x7fc07fc0u};
                }
#pragma unroll
                for (int k = 0; k < 8; ++k) __builtin_amdgcn_raw_buffer_store_b128(va[k], rd, dv[k], 0, 17  );
                __builtin_amdgcn_sched_barrier(0);
                qsig = psig; qsegn = psegn;
                const unsigned okm = (unsigned)__ballot((int)(pv - ((lm0 & nneed) | (lm1 & (8u * call)) | (lm2 & (64u * call)))) >= 0);
                r2 = r2 || (w2 && (okm & 2u));
                gate = gate && !(okm & 4u);
                nrdy = nrdy || (nw && !gate && (okm & 1u));
                if (cl1) { tk = UNI(tkv1); tks = 2; tkpool = pool; if (!pool && tk >= nfix) { pool = true; tks = 0; } }
                if (cl2) { tk2 = UNI(tkv2); tk2s = 2; if (tk2 >= (unsigned)Q2_ITEMS) { tk2s = 0; q2 = false; STAMP(2); } }
                psig = csig; pseg = cseg; psegn = csegn;
            }
        }
        STAMP(3);
        __syncthreads();
        wg_exit(ctl, call, wid, lane);
        return;
    }

    const unsigned call = (unsigned)__builtin_amdgcn_readfirstlane(ld_relaxed_agent(ctl + L_CALL * 32)) + 1u;
    stage_inputs(P, myws, call, xl, lane, slot >= 32 - K_STW ? (wid < K_STV ? 1 << 20 : 0) : (wid < K_ST1 ? 1 : 0));
    asm volatile("s_waitcnt vmcnt(0)" ::: "memory");
    const int njobs = (int)P.njob[xl];
    for (int job = slot - CE_WGS, round = 0; job < njobs; ++round) {
        const unsigned jw = (P.job[(xl * MAXJ + job) >> 1] >> ((job & 1) * 16)) & 0xffffu;
        if (jw >> 8) job = gemm_job<true>(P, lds, myws, call, (int)(jw & 255u), (int)(jw >> 8) & 3, xl, round);
        else job = gemm_job<false>(P, lds, myws, call, (int)jw, 0, xl, round);
    }
    wg_exit(ctl, call, wid, lane);
}

}

extern "C" void kernel_launch(int rank, int nranks, ncclComm_t comm,
                              hipStream_t stream,
                              void* const* d_in, const int* in_sizes, int n_in,
                              void* d_out, int out_size,
                              const int* in_dtypes, int out_dtype,
                              void* const* d_out_all,
                              void* const* d_ws_all, size_t ws_bytes) {
    if (nranks != NR || (unsigned)rank >= (unsigned)NR || n_in < 2 || ws_bytes < WS_END || (size_t)out_size != (size_t)GM * GN ||
        (size_t)in_sizes[0] != (size_t)GM * KS || (size_t)in_sizes[1] != (size_t)KS * GN) {
        fprintf(stderr, "kernel_launch: unexpected problem shape\n");
        return;
    }
    const int member[2][4] = {{0, 2, 3, 6}, {1, 4, 5, 7}};
    Params P = {};
    P.x = d_in[0]; P.w = d_in[1]; P.g = 0; P.a = 0;
    for (int g = 0; g < 2; ++g) for (int a = 0; a < 4; ++a) if (member[g][a] == rank) { P.g = g; P.a = a; }
    for (int e = 0; e < 8; ++e) {
        const int peer = e < 3 ? member[P.g][P.a ^ (e + 1)] : (e == 3 ? rank : member[1 - P.g][P.a ^ (e - 4)]);
        P.out[e] = d_out_all[peer]; P.ws[e] = d_ws_all[peer];
    }
    unsigned char* tb = (unsigned char*)P.tab;
    unsigned short* jb = (unsigned short*)P.job;
    int n = 0, np[2] = {0, 0}, nx[8] = {0, 0, 0, 0, 0, 0, 0, 0};
    for (int k = 0; k < 16; ++k)
        for (int r = 0; r < 2 * k + 1; ++r, ++n) {
            const int I = r < k ? k : r - k, J = r < k ? r : k;
            const unsigned char tl = (unsigned char)(I << 4 | J);
            const int par = (I + J) & 1;
            tb[par * 128 + np[par]++] = tl;
            if (n < K_SMALL) { const int xs = n & 7; for (int q = 0; q < 4; ++q) jb[xs * MAXJ + nx[xs]++] = (unsigned short)(tl | (4 + q) << 8); }
            else { const int x = (I & 3) << 1 | ((J >> 1) & 1); jb[x * MAXJ + nx[x]++] = tl; }
        }
    for (int x = 0; x < 8; ++x) P.njob[x] = (unsigned)nx[x];
    hipLaunchKernelGGL(gemm_ar_kernel, dim3(GRID), dim3(512), 0, stream, P);
    HIP_CHECK(hipGetLastError());
}
```

```cpp
#include <hip/hip_runtime.h>
#include <rccl/rccl.h>
#include <cstdio>

#define HIP_CHECK(expr) do { hipError_t _e = (expr); if (_e != hipSuccess) { \
    fprintf(stderr, "HIP error %s at %s:%d\n", hipGetErrorString(_e), __FILE__, __LINE__); \
    return; } } while (0)

namespace {

#define LAS __attribute__((address_space(3)))
typedef short bf16x8 __attribute__((ext_vector_type(8)));
typedef short s16x4 __attribute__((ext_vector_type(4)));
typedef float f32x4 __attribute__((ext_vector_type(4)));
typedef unsigned u32x4 __attribute__((ext_vector_type(4)));

constexpr int NR = 8;
constexpr int GM = 8192, GN = 8192;
constexpr int KS = 1024;

#define K_PUSH 2
#define K_GATE 0
#define K_GRP 2
#define K_ST1 2
#define K_STW 4
#define K_STV 4
#define K_NA 36
#define K_NL 118
#define K_SMALL 49
#define K_SLEEP 8
#define K_CE 14
constexpr int NPUSH = K_PUSH, NA = K_NA, NL = K_NL;
static_assert(NA >= 1 && NA < NL && NL <= 128 && NPUSH >= 0 && NPUSH < 32 && K_GATE < NPUSH, "knobs");

constexpr unsigned CTL_BYTES = 1u << 20;
constexpr unsigned XS_OFF = CTL_BYTES, XS_SLOT = 16u * 524288u;
constexpr unsigned WS_OFF = XS_OFF + 3u * XS_SLOT;
constexpr unsigned IN_OFF = WS_OFF + 3u * XS_SLOT;
constexpr unsigned PT_OFF = IN_OFF + 128u * 262144u;
constexpr unsigned RT_OFF = PT_OFF + 256u * 131072u;
constexpr unsigned FT_OFF = RT_OFF + 256u * 131072u;
constexpr unsigned WS_END = FT_OFF + 256u * 131072u;
constexpr unsigned OUT_BYTES = (unsigned)((size_t)GM * GN * 2);
constexpr unsigned IN_BYTES = (unsigned)((size_t)GM * KS * 2);

constexpr int L_TMO = 1, L_EXIT = 2, L_CALL = 3  , L_STG = 16  , L_STGC = 320  ,
              L_LAND = 32  , L_HEAD = 64  ,
              L_PANEL = 128  , L_FIN = 256  ,
              L_TILE = 512  , L_SCNT = 1024  ;
constexpr int CE_WGS = K_CE;
constexpr int G_WGS = 32 - CE_WGS;
constexpr int GRID = 256;
constexpr unsigned long long DEADLINE_TICKS = 50000000ull;

constexpr int IN_ITEMS = 32 * 32;
constexpr int Q2_ITEMS = 128 * 16;
constexpr int FIX6 = IN_ITEMS + 128 * 16, FIX4 = NA * 16;
constexpr int POOL_ITEMS = (128 - NA) * 16, FWD_MAX = (NL - NA) * 16;
constexpr int Q3L_ITEMS = 128 * 16;
constexpr int STG_ITEMS = (32 - NPUSH) * 8;
constexpr int STG_GRP = K_GRP;
static_assert((32 - NPUSH) % STG_GRP == 0 && (32 - NPUSH) / STG_GRP <= 32, "staging groups");

constexpr int BK = 64, HALF = 128, HTB = HALF * BK * 2, STAGE_BYTES = 8 * HTB;

#define STAMP_TK do {} while (0)
#define STAMP(i) do {} while (0)
#define PROG(q, it) do {} while (0)

constexpr int MAXJ = 96;
static_assert((K_SMALL + 7) / 8 * 4 + 32 <= MAXJ, "job table");
struct Params {
    const void* x; const void* w;
    void* out[8]; void* ws[8];
    int g, a;
    unsigned njob[8], job[8 * MAXJ / 2];
    unsigned tab[64];
};
typedef const __attribute__((address_space(4))) Params CParams;
#define TAB(i) ((P.tab[(unsigned)(i) >> 2] >> (((unsigned)(i) & 3u) * 8u)) & 255u)
#define RSRC(p, n) __builtin_amdgcn_make_buffer_rsrc((void*)(p), 0, (int)(n), 0x00020000)
#define UNI(x) (unsigned)__builtin_amdgcn_readfirstlane((int)(x))

__device__ __forceinline__ int lds_byte(int r, int c) { const int st = (r >> 4) * 2 + (c >> 5), rr = r & 15, cc = c & 31, ob = rr * 64 + cc * 2; return st * 1024 + (ob ^ (((ob >> 9) & 1) << 5)); }
__device__ __forceinline__ void stage_rc(int b, int& R, int& C) { const int st = b / 1024, sb = b % 1024, swz = sb ^ (((sb >> 9) & 1) << 5); R = (st >> 1) * 16 + swz / 64; C = (st & 1) * 32 + (swz % 64) / 2; }
__device__ __forceinline__ unsigned wswz(unsigned row) { return ((row & 3u) << 2) | ((row >> 2) & 3u); }
typedef float f32x2 __attribute__((ext_vector_type(2)));
typedef __bf16 bf16x2 __attribute__((ext_vector_type(2)));
__device__ __forceinline__ unsigned cvt_pk_bf16(float lo, float hi) {
    return __builtin_bit_cast(unsigned, __builtin_convertvector((f32x2){lo, hi}, bf16x2));
}
__device__ __forceinline__ unsigned panel_base(int ab, int I) {
    const int j = I >> 1;
    return (unsigned)((j << 2) | ((2 * ab + (I & 1) + (0xb114 >> (2 * j))) & 3)) * 256u;
}
__device__ __forceinline__ unsigned out_offset(int a, unsigned tl, unsigned st) {
    return (panel_base(a >> 1, (int)(tl >> 4)) + st * 16u) * 16384u + panel_base(a & 1, (int)(tl & 15u)) * 2u;
}

__device__ __forceinline__ unsigned ld_relaxed_sys(const unsigned* p) { return __hip_atomic_load(p, __ATOMIC_RELAXED, __HIP_MEMORY_SCOPE_SYSTEM); }
__device__ __forceinline__ unsigned ld_relaxed_agent(const unsigned* p) { return __hip_atomic_load(p, __ATOMIC_RELAXED, __HIP_MEMORY_SCOPE_AGENT); }

__device__ __forceinline__ bool gave_up(unsigned long long t0, unsigned* tmo, unsigned call) {
    if ((unsigned)__builtin_amdgcn_readfirstlane(ld_relaxed_agent(tmo)) == call || __builtin_amdgcn_s_memrealtime() - t0 > DEADLINE_TICKS) {
        __hip_atomic_store(tmo, call, __ATOMIC_RELAXED, __HIP_MEMORY_SCOPE_AGENT);
        return true;
    }
    return false;
}
__device__ __forceinline__ unsigned add_bf16x2(unsigned a, unsigned b) {
    return cvt_pk_bf16(__uint_as_float(a << 16) + __uint_as_float(b << 16), __uint_as_float(a & 0xffff0000u) + __uint_as_float(b & 0xffff0000u));
}

__device__ __forceinline__ void wg_exit(unsigned* ctl, unsigned call, int wid, int lane) {
    if (wid != 0) return;
    const unsigned t = (unsigned)__builtin_amdgcn_raw_ptr_buffer_atomic_add_i32(1, __builtin_amdgcn_make_buffer_rsrc(ctl, 0, (int)CTL_BYTES, 0x00020000), lane == 0 ? (unsigned)L_EXIT * 128u : 0x7fffff00u, 0, 0);
    if (((unsigned)__builtin_amdgcn_readfirstlane((int)t) & (unsigned)(GRID - 1)) == (unsigned)(GRID - 1)) {
        if (lane < 34) __hip_atomic_store(ctl + (lane == 33 ? L_CALL : L_HEAD + lane) * 32, lane == 33 ? call : 0u, __ATOMIC_RELAXED, __HIP_MEMORY_SCOPE_AGENT);
    }
}

template <bool SMALL>
__device__ __forceinline__ int gemm_job(CParams& P, LAS unsigned char* lds, char* myws, unsigned call, int tl, int qd, int xl, int round) {
    int tid = threadIdx.x; asm volatile("" : "+v"(tid));
    const int wid = __builtin_amdgcn_readfirstlane(tid >> 6), lane = tid & 63;
    const int a1 = P.a >> 1, a0 = P.a & 1, ti = tl >> 4, tj = tl & 15;
    unsigned* ctl = (unsigned*)myws;
    unsigned* tmo = ctl + L_TMO * 32;
    const int wr = wid >> 2, wc = wid & 3, fr = lane & 15, fq = lane >> 4;
    unsigned voffA[2], voffB[2];
#pragma unroll
    for (int i = 0; i < 2; ++i) { int R, C; stage_rc(tid * 16 + i * 8192, R, C); voffA[i] = (unsigned)(R * KS + C) * 2u; }
#define SET_VOFFB(sh) do { int tz = tid; asm volatile("" : "+v"(tz)); _Pragma("unroll") for (int i = 0; i < 2; ++i) { \
        const unsigned row = (unsigned)(tz >> 4) + 32u * i, ch = ((unsigned)tz & 15u) ^ wswz(row); voffB[i] = (row << (sh)) + ch * 16u; } \
        kstepB = (size_t)(SMALL ? 2 * BK : BK) << (sh); hstepB = SMALL ? (size_t)BK << (sh) : (size_t)HALF * 2; } while (0)
    size_t kstepB, hstepB;
    constexpr size_t kstepA = SMALL ? BK * 4 : BK * 2, hstepA = SMALL ? BK * 2 : (size_t)HALF * KS * 2;
    const unsigned qa = SMALL ? (unsigned)(qd >> 1) * (unsigned)(HALF * KS * 2) : 0u, qb = SMALL ? (unsigned)(qd & 1) * 256u : 0u;
    const unsigned ldsw = (unsigned)wid * 1024u;
    const int aoff = lds_byte(wr * 64 + fr, fq * 8);
    int boff[2][2];
#pragma unroll
    for (int n = 0; n < 2; ++n)
#pragma unroll
        for (int t = 0; t < 2; ++t) {
            const unsigned q4 = (unsigned)(lane & 15) >> 2, p4 = (unsigned)lane & 3u, row = 8u * fq + 4u * t + q4, ch = 4u * wc + 2u * n + (p4 >> 1);
            boff[n][t] = (int)((unsigned)(size_t)lds + 4u * HTB + 256u * row + 16u * (ch ^ wswz(row)) + 8u * (p4 & 1u));
        }
#define SA(b, h) (((b) * 2 + (h)) * HTB)
#define SB(b, h) ((4 + (b) * 2 + (h)) * HTB)
#define STAGE(bufoff, gbase, voff) do { _Pragma("unroll") for (int _i = 0; _i < 2; ++_i) \
        __builtin_amdgcn_global_load_lds((const unsigned*)((const char*)(gbase) + (voff)[_i]), (LAS unsigned*)(lds + (bufoff) + ldsw + _i * 8192), 16, 0, 0); } while (0)
#define LDA(dst, b, h) do { _Pragma("unroll") for (int m = 0; m < 4; ++m) _Pragma("unroll") for (int k = 0; k < 2; ++k) dst[m][k] = *(const LAS bf16x8*)(lds + SA(b, h) + aoff + m * 2048 + k * 1024); } while (0)
#define TRR(dst, base, off) asm volatile("ds_read_b64_tr_b16 %0, %1 offset:%2" : "=&v"(dst) : "v"(base), "i"(off) : "memory")
#define LDB(raw, b, h) do { \
        TRR(raw[0], boff[0][0], SB(b, h) - 4 * HTB); TRR(raw[1], boff[0][1], SB(b, h) - 4 * HTB); TRR(raw[2], boff[0][0], SB(b, h) - 4 * HTB + 8192); TRR(raw[3], boff[0][1], SB(b, h) - 4 * HTB + 8192); \
        TRR(raw[4], boff[1][0], SB(b, h) - 4 * HTB); TRR(raw[5], boff[1][1], SB(b, h) - 4 * HTB); TRR(raw[6], boff[1][0], SB(b, h) - 4 * HTB + 8192); TRR(raw[7], boff[1][1], SB(b, h) - 4 * HTB + 8192); } while (0)
#define WAIT_B(raw, dst) do { asm volatile("s_waitcnt lgkmcnt(0)" : "+v"(raw[0]), "+v"(raw[1]), "+v"(raw[2]), "+v"(raw[3]), "+v"(raw[4]), "+v"(raw[5]), "+v"(raw[6]), "+v"(raw[7]) :: "memory"); \
        _Pragma("unroll") for (int n = 0; n < 2; ++n) _Pragma("unroll") for (int k = 0; k < 2; ++k) dst[n][k] = __builtin_shufflevector(raw[n * 4 + k * 2], raw[n * 4 + k * 2 + 1], 0, 1, 2, 3, 4, 5, 6, 7); } while (0)
#define MMA(ai, bj, At, Bt) do { if (!SMALL || (ai) == (bj)) { __builtin_amdgcn_s_setprio(1); _Pragma("unroll") for (int m = 0; m < 4; ++m) _Pragma("unroll") for (int n = 0; n < 2; ++n) _Pragma("unroll") for (int k = 0; k < 2; ++k) \
        acc[SMALL ? 0 : ai][SMALL ? 0 : bj][m][n] = __builtin_amdgcn_mfma_f32_16x16x32_bf16(Bt[n][k], At[m][k], acc[SMALL ? 0 : ai][SMALL ? 0 : bj][m][n], 0, 0, 0); __builtin_amdgcn_s_setprio(0); } } while (0)
#define WAIT_V(n) asm volatile("s_waitcnt vmcnt(" #n ")" ::: "memory")
#define WAIT_L(n) asm volatile("s_waitcnt lgkmcnt(" #n ")" ::: "memory")
#define BAR __builtin_amdgcn_s_barrier()
#define SCHED __builtin_amdgcn_sched_barrier(0)
    constexpr int NH2 = SMALL ? 1 : 2;
    f32x4 acc[NH2][NH2][4][2];
#pragma unroll
    for (int a = 0; a < NH2; ++a)
#pragma unroll
        for (int b = 0; b < NH2; ++b)
#pragma unroll
            for (int m = 0; m < 4; ++m)
#pragma unroll
                for (int n = 0; n < 2; ++n) acc[a][b][m][n] = (f32x4){0.f, 0.f, 0.f, 0.f};
    bf16x8 At[4][2], B0[2][2], B1[2][2];
    s16x4 R0[8], R1[8];
    const char* cA = (const char*)P.x + (size_t)panel_base(a1, ti) * (KS * 2) + qa;
    const char* cB = (const char*)P.w + (size_t)panel_base(a0, tj) * 2 + qb;
    constexpr int nt = SMALL ? KS / BK / 2 : KS / BK;
    unsigned todo = 7u;
    SET_VOFFB(14);
    STAGE(SB(0, 0), cB, voffB); STAGE(SA(0, 0), cA, voffA); STAGE(SB(0, 1), cB + hstepB, voffB); STAGE(SA(0, 1), cA + hstepA, voffA);
    if (wr == 1) BAR;
    WAIT_V(4); BAR;
    STAGE(SB(1, 0), cB + kstepB, voffB); STAGE(SA(1, 0), cA + kstepA, voffA); STAGE(SB(1, 1), cB + hstepB + kstepB, voffB);
    WAIT_V(6); BAR;
    for (int s = 0; s < 4; ++s) {
        const char* nA = cA; const char* nB = cB;
        for (int t = 0; t < nt; t += 2) {
            const bool last = (t == nt - 2);
            if (last && s < 3) {
                if (wid == 0) {
                    STAMP(1 + 2 * s + 8 * round);
                    const unsigned* word = ctl + (L_PANEL + ((lane >> 1) % 3) * 32 + ((lane & 1) ? 2 * tj + 1 : 2 * ti)) * 32;
                    const unsigned long long t0 = __builtin_amdgcn_s_memrealtime();
                    unsigned pick;
                    for (unsigned polls = 1;; ++polls) {
                        const unsigned mk = (unsigned)__ballot((int)(ld_relaxed_sys(word) - 64u * call) >= 0);
                        const unsigned both = mk & (mk >> 1);
                        pick = ((both & 1u) | ((both >> 1) & 2u) | ((both >> 2) & 4u)) & todo;
                        if (pick) break;
                        if ((polls & 63u) == 0u && gave_up(t0, tmo, call)) { pick = todo; break; }
                        __builtin_amdgcn_s_sleep(4);
                    }
                    STAMP(2 + 2 * s + 8 * round);
                    *(LAS unsigned*)(lds + STAGE_BYTES + 16) = (unsigned)__builtin_ctz(pick);
                    __builtin_amdgcn_fence(__ATOMIC_ACQUIRE, "");
                    asm volatile("s_waitcnt vmcnt(0) lgkmcnt(0)" ::: "memory");
                }
                asm volatile("" ::: "memory"); BAR; asm volatile("" ::: "memory");
                unsigned slv;
                asm volatile("ds_read_b32 %0, %1\n\ts_waitcnt lgkmcnt(0)" : "=&v"(slv) : "v"((unsigned)(size_t)lds + STAGE_BYTES + 16) : "memory");
                const unsigned sl = (unsigned)__builtin_amdgcn_readfirstlane(slv) & 3u;
                todo &= ~(1u << sl);
                nA = myws + XS_OFF + sl * XS_SLOT + (unsigned)ti * 524288u + qa;
                nB = myws + WS_OFF + sl * XS_SLOT + (unsigned)tj * 524288u + qb;
                if (s == 0) SET_VOFFB(9);
            }
            const char* a1p = cA + (size_t)(t + 1) * kstepA;
            const char* a2 = last ? nA : cA + (size_t)(t + 2) * kstepA; const char* b2 = last ? nB : cB + (size_t)(t + 2) * kstepB;
            const char* a3 = a2 + kstepA; const char* b3 = b2 + kstepB;
            LDB(R0, 0, 0); SCHED; LDA(At, 0, 0); STAGE(SA(1, 1), a1p + hstepA, voffA);
            WAIT_L(8); BAR; WAIT_B(R0, B0); MMA(0, 0, At, B0); BAR; SCHED;
            LDB(R1, 0, 1); STAGE(SB(0, 0), b2, voffB);
            BAR; WAIT_B(R1, B1); MMA(0, 1, At, B1); BAR; if (SMALL) SCHED;
            LDA(At, 0, 1); STAGE(SA(0, 0), a2, voffA);
            BAR; WAIT_L(0); MMA(1, 0, At, B0); BAR; SCHED;
            STAGE(SB(0, 1), b2 + hstepB, voffB);
            WAIT_V(6); BAR; MMA(1, 1, At, B1); BAR;
            LDB(R0, 1, 0); SCHED; LDA(At, 1, 0); STAGE(SA(0, 1), a2 + hstepA, voffA);
            WAIT_L(8); BAR; WAIT_B(R0, B0); MMA(0, 0, At, B0); BAR; SCHED;
            LDB(R1, 1, 1); STAGE(SB(1, 0), b3, voffB);
            BAR; WAIT_B(R1, B1); MMA(0, 1, At, B1); BAR; if (SMALL) SCHED;
            LDA(At, 1, 1); STAGE(SA(1, 0), a3, voffA);
            BAR; WAIT_L(0); MMA(1, 0, At, B0); BAR; SCHED;
            STAGE(SB(1, 1), b3 + hstepB, voffB);
            WAIT_V(6); BAR; MMA(1, 1, At, B1); BAR;
        }
        cA = nA; cB = nB;
    }
    WAIT_V(0);
    if (wr == 0) BAR;
    STAMP(7 + 8 * round);
    const __amdgpu_buffer_rsrc_t rc = __builtin_amdgcn_make_buffer_rsrc(myws, 0, (int)CTL_BYTES, 0x00020000);
    const unsigned nj = (unsigned)__builtin_amdgcn_raw_ptr_buffer_atomic_add_i32(1, rc, tid == 0 ? (unsigned)(L_HEAD + 16 + xl) * 128u : 0x7fffff00u, 0, 0);
    const bool bad = (unsigned)__builtin_amdgcn_readfirstlane(ld_relaxed_agent(tmo)) == call;
    const __amdgpu_buffer_rsrc_t ro = __builtin_amdgcn_make_buffer_rsrc(myws + PT_OFF + (unsigned)tl * 131072u, 0, 131072, 0x00020000);
    const unsigned obase = (unsigned)(64 * wr + fr + (SMALL ? (qd >> 1) * HALF : 0)) * 512u + (unsigned)(32 * wc + (fq & 1) * 16 + (fq >> 1) * 8) * 2u + qb;
#pragma unroll
    for (int ai = 0; ai < NH2; ++ai)
#pragma unroll
        for (int m = 0; m < 4; ++m)
#pragma unroll
            for (int bj = 0; bj < NH2; ++bj) {
                const f32x4 v0 = acc[ai][bj][m][0], v1 = acc[ai][bj][m][1];
                const unsigned x0 = cvt_pk_bf16(v0[0], v0[1]), x1 = cvt_pk_bf16(v0[2], v0[3]), y0 = cvt_pk_bf16(v1[0], v1[1]), y1 = cvt_pk_bf16(v1[2], v1[3]);
                const auto r0 = __builtin_amdgcn_permlane16_swap(x0, y0, false, false);
                const auto r1 = __builtin_amdgcn_permlane16_swap(x1, y1, false, false);
                u32x4 wv = {r0[0], r1[0], r0[1], r1[1]};
                if (bad) wv = (u32x4){0x7fc07fc0u, 0x7fc07fc0u, 0x7fc07fc0u, 0x7fc07fc0u};
                __builtin_amdgcn_raw_buffer_store_b128(wv, ro, obase + (unsigned)(ai * HALF + m * 16) * 512u + (unsigned)bj * 256u, 0, 17  );
            }
    if (wid == 0) *(LAS unsigned*)(lds + STAGE_BYTES) = (unsigned)__builtin_amdgcn_readfirstlane((int)nj);
    asm volatile("s_waitcnt vmcnt(0) lgkmcnt(0)" ::: "memory");
    BAR;
    const bool mine = ((ti + tj) & 1) == P.g;
    (void)__builtin_amdgcn_raw_ptr_buffer_atomic_add_i32(SMALL ? 2 : 8, __builtin_amdgcn_make_buffer_rsrc(P.ws[mine ? 3 : 4], 0, (int)CTL_BYTES, 0x00020000),
                                                         tid == 0 ? (unsigned)(L_TILE + tl) * 128u + (mine ? 0u : 32u) : 0x7fffff00u, 0, 16);
    STAMP(8 + 8 * round);
    unsigned njv;
    asm volatile("ds_read_b32 %0, %1\n\ts_waitcnt lgkmcnt(0)" : "=&v"(njv) : "v"((unsigned)(size_t)lds + STAGE_BYTES) : "memory");
    return __builtin_amdgcn_readfirstlane((int)njv) + G_WGS;
}

__device__ __forceinline__ unsigned in_offset(unsigned t, int ab, int kh, unsigned u) {
    return (t & 1u) ? ((unsigned)kh * 512u + u * 16u) * 16384u + panel_base(ab, (int)(t >> 1)) * 2u : (panel_base(ab, (int)(t >> 1)) + u * 8u) * 2048u + (unsigned)kh * 1024u;
}
__device__ __forceinline__ void stage_inputs(CParams& P, char* myws, unsigned call, int lb, int lane, int maxn) {
    const __amdgpu_buffer_rsrc_t rl = RSRC(myws, WS_END);
    const unsigned l0 = lane == 0 ? 0u : 0x7fffff00u, llane = (unsigned)lane * 16u, rlane = (unsigned)(lane >> 5) * 16384u + (unsigned)(lane & 31) * 16u;
    for (; maxn > 0; --maxn) {
        const unsigned it = UNI(__builtin_amdgcn_raw_ptr_buffer_atomic_add_i32(1, rl, l0, (L_HEAD + 24 + lb) * 128, 0));
        if (it >= (unsigned)STG_ITEMS) break;
        const unsigned pc = (it * 8u + (unsigned)lb) * 2u + (unsigned)NPUSH * 128u, t = pc >> 7, g = (t - (unsigned)NPUSH) / (unsigned)STG_GRP;
        const unsigned so = in_offset(t, (int)(pc >> 6) & 1, (int)(pc >> 5) & 1, pc & 31u) + ((t & 1u) ? rlane : llane), ss = (t & 1u) ? 32768u : 2048u;
        const __amdgpu_buffer_rsrc_t rs = RSRC((t & 1u) ? (char*)P.w : (char*)P.x, IN_BYTES);
        u32x4 v[16];
#pragma unroll
        for (int k = 0; k < 16; ++k) v[k] = __builtin_amdgcn_raw_buffer_load_b128(rs, so + k * ss, 0, 0);
#pragma unroll
        for (int k = 0; k < 16; ++k) __builtin_amdgcn_raw_buffer_store_b128(v[k], rl, IN_OFF + pc * 8192u + llane + k * 1024u, 0, 17);
        asm volatile("s_waitcnt vmcnt(0)" ::: "memory");
        if (UNI(__builtin_amdgcn_raw_ptr_buffer_atomic_add_i32(1, rl, l0, (L_STGC + (int)g) * 128, 0)) == (unsigned)(STG_GRP * 64) * call - 1u) {
            if (lane < 8 && lane != 3 && lane != 4) __hip_atomic_store((unsigned*)P.ws[lane] + (L_STG + lane) * 32 + g, call, __ATOMIC_RELAXED, __HIP_MEMORY_SCOPE_SYSTEM);
        }
    }
}

__global__ __launch_bounds__(512, 2) void gemm_ar_kernel(const Params) {
    CParams& P = *(CParams*)__builtin_amdgcn_kernarg_segment_ptr();
    __shared__ __attribute__((aligned(16))) unsigned char shm[STAGE_BYTES + 64];
    LAS unsigned char* lds = (LAS unsigned char*)shm;
    const int tid = threadIdx.x, wid = __builtin_amdgcn_readfirstlane(tid >> 6), lane = tid & 63;
    const int hm = P.g, qa = P.a;
    const int xl = blockIdx.x & 7, slot = blockIdx.x >> 3;
    char* myws = (char*)P.ws[3];
    unsigned* ctl = (unsigned*)myws;
    unsigned* tmo = ctl + L_TMO * 32;
    STAMP(0);

    if (slot < CE_WGS) {
        const int e = xl;
        const bool active = wid == 0 || e == 3;
        const __amdgpu_buffer_rsrc_t rl = RSRC(myws, WS_END);
        const unsigned l0 = lane == 0 ? 0u : 0x7fffff00u;
#define ATOM(rs, off, v, aux) (unsigned)__builtin_amdgcn_raw_ptr_buffer_atomic_add_i32((int)(v), rs, l0, off, aux)
#define ATOMV(rs, off, v) (unsigned)__builtin_amdgcn_raw_ptr_buffer_atomic_add_i32((int)(v), rs, l0 | (off), 0, 16)
        const unsigned llane = (unsigned)lane * 16u;
        const unsigned rlane = (unsigned)(lane >> 5) * 16384u + (unsigned)(lane & 31) * 16u;
        const int head = (L_HEAD + e * 2) * 128, scnt = (L_SCNT + e * 8) * 128;
        unsigned call = 0;
        const unsigned callv = __builtin_amdgcn_raw_buffer_load_b32(rl, L_CALL * 128u, 0, 16);
        STAMP(4);
        bool ok = true;
        unsigned idle = 0; unsigned long long t0 = 0;
        if (active && e == 3) {
            call = UNI(callv) + 1u;
            if (wid == 7) for (int k = 0; k < 8; ++k) stage_inputs(P, myws, call, (slot + k) & 7, lane, 1 << 20);
            unsigned tkv = ATOM(rl, head, 1, 0);
            const __amdgpu_buffer_rsrc_t ro = RSRC(P.out[3], OUT_BYTES);
            for (unsigned it = UNI(tkv); it < (unsigned)Q3L_ITEMS; it = UNI(tkv)) {
                const unsigned tl = TAB(hm * 128 + (it >> 4)), st = it & 15u, dep = (unsigned)(L_TILE + tl) * 128u;
                t0 = __builtin_amdgcn_s_memrealtime();
                for (idle = 1; ok; ++idle) {
                    const unsigned v = __builtin_amdgcn_raw_buffer_load_b32(rl, dep + (unsigned)(lane & 1) * 64u, 0, 17);
                    if (__all((int)(v - ((lane & 1) ? 16u : 8u) * call) >= 0)) break;
                    if ((idle & 63u) == 0u && gave_up(t0, tmo, call)) ok = false;
                    __builtin_amdgcn_s_sleep(K_SLEEP);
                }
                __builtin_amdgcn_fence(__ATOMIC_ACQUIRE, "wavefront");
                PROG(2, it);
                const unsigned sv = tl * 131072u + st * 8192u + llane, dv = out_offset(qa, tl, st) + rlane;
                u32x4 va[8], vb[8];
#pragma unroll
                for (int k = 0; k < 8; ++k) { va[k] = __builtin_amdgcn_raw_buffer_load_b128(rl, PT_OFF + sv + k * 1024u, 0, 17); vb[k] = __builtin_amdgcn_raw_buffer_load_b128(rl, RT_OFF + sv + k * 1024u, 0, 17); }
#pragma unroll
                for (int k = 0; k < 8; ++k) {
                    u32x4 o = {add_bf16x2(va[k].x, vb[k].x), add_bf16x2(va[k].y, vb[k].y), add_bf16x2(va[k].z, vb[k].z), add_bf16x2(va[k].w, vb[k].w)};
                    if (!ok) o = (u32x4){0x7fc07fc0u, 0x7fc07fc0u, 0x7fc07fc0u, 0x7fc07fc0u};
                    __builtin_amdgcn_raw_buffer_store_b128(o, rl, FT_OFF + sv + k * 1024u, 0, 17);
                    __builtin_amdgcn_raw_buffer_store_b128(o, ro, dv + k * 32768u, 0, 16);
                }
                tkv = ATOM(rl, head, 1, 0);
                asm volatile("s_waitcnt vmcnt(0)" ::: "memory");
                if ((UNI(ATOM(rl, scnt + (int)(it >> 4) * 4, 1, 0)) & 15u) == 15u) {
                    if (lane < 8 && lane != 3) __hip_atomic_store((unsigned*)P.ws[lane] + (L_FIN + lane * 4) * 32 + (it >> 4), call, __ATOMIC_RELAXED, __HIP_MEMORY_SCOPE_SYSTEM);
                }
            }
        } else if (active) {
            const int x = e < 3 ? e + 1 : e - 4, hp = e < 3 ? hm : hm ^ 1, qp = qa ^ x;
            const unsigned nfix = e == 4 ? (unsigned)FIX4 : (unsigned)FIX6;
            constexpr unsigned OOR = 0x7fff0000u;
            bool more = true, spare = true, pool = false, tkpool = false;
            int tks = 2, tk2s = 0;
            unsigned tk = 2u * (unsigned)slot, tk2 = 0;
            bool q2 = e == 4;
            int nk = 0; unsigned nso = 0, ndo = 0, ndep = OOR, nneed = 0, nsig = 0, nseg = 0, nsegn = 1; bool nrdy = false;
            bool h2 = false, r2 = false; unsigned p2 = 0, tl2 = 0;
            unsigned psig = 0, pseg = 0, psegn = 1, qsig = 0, qsegn = 1, qv = 0;
            const unsigned lm0 = lane == 0 ? ~0u : 0u, lm1 = lane == 1 ? ~0u : 0u, lm2 = lane == 2 ? ~0u : 0u, lmx = lane > 2 ? OOR : 0u;
            bool gate = K_GATE >= 0 && e != 4;
#define REFILL() do { \
                if (nk == 0 && tks == 2) { \
                    tks = 0; nsig = 0; nseg = 0; nneed = 1u; \
                    if (!tkpool && e != 4 && tk < (unsigned)IN_ITEMS) {       \
                        const unsigned t = tk >> 5, u = tk & 31u, half = e < 3 ? 0u : 1u; const bool push = t < (unsigned)NPUSH; \
                        nk = (push ? 1 : 3) + (int)(t & 1u); \
                        nso = push ? in_offset(t, (t & 1u) ? qp & 1 : qp >> 1, hp, u) : IN_OFF + (((t * 2u + (unsigned)((t & 1u) ? qa & 1 : qa >> 1)) * 2u + (unsigned)hm) * 32u + u) * 8192u; \
                        ndo = (unsigned)(x - 1) * XS_SLOT + (t >> 1) * 524288u + ((t & 1u) ? WS_OFF + half * 262144u + u * 8192u : XS_OFF + u * 16384u + half * 1024u); \
                        nseg = t; nsegn = 32u; nsig = (unsigned)(L_PANEL + (x - 1) * 32 + (int)t) * 128u + (push ? (unsigned)e : 3u); \
                        ndep = (unsigned)(L_STG + e) * 128u + (push ? 0u : (t - (unsigned)NPUSH) / (unsigned)STG_GRP * 4u); nrdy = push || !ok; \
                    } else { \
                        unsigned r; bool none = false;                 \
                        const bool fwd = tkpool && e != 4; \
                        if (tkpool) { const unsigned lo = tk & 0xffffu, hi = tk >> 16, loc = lo < (unsigned)FWD_MAX ? lo : (unsigned)FWD_MAX; \
                            none = loc + hi >= (unsigned)POOL_ITEMS || (fwd && lo >= (unsigned)FWD_MAX); r = none ? 0u : fwd ? (unsigned)(NA * 16) + lo : 2047u - hi; } \
                        else if (e == 4) r = tk; \
                        else { r = tk - (unsigned)IN_ITEMS; if (r < (unsigned)(NL * 16)) { nseg = 64u; nsegn = (unsigned)(NL * 16); nsig = (unsigned)(L_LAND + e2) * 128u + (unsigned)e2; } } \
                        const unsigned tl = TAB((fwd ? hm ^ 1 : hp) * 128 + (int)(r >> 4)), st = r & 15u; \
                        nk = none ? 0 : fwd ? 6 : 5; \
                        ndo = out_offset(fwd ? qa : qp, tl, st); \
                        nso = fwd ? ndo : FT_OFF + tl * 131072u + st * 8192u; \
                        ndep = fwd ? (unsigned)(L_LAND + e) * 128u : (unsigned)(L_FIN + e * 4) * 128u + (r >> 4) * 4u; \
                        nneed = fwd ? (unsigned)(NL * 16) : 1u; \
                        nrdy = !ok; \
                        if (none) { more = false; STAMP(29); } \
                    } \
                    STAMP_TK; \
                    if (spare) { spare = false; ++tk; tks = 2; } \
                } \
                if (!h2 && tk2s == 2) { tk2s = 0; h2 = true; p2 = tk2; tl2 = TAB(hm * 128 + (int)(p2 >> 4)); r2 = !ok; } \
            } while (0)
            const int e2 = e < 3 ? 4 + x : x - 1;
            int ck = 0; unsigned cso = 0, cdo = 0, csig = 0, cseg = 0, csegn = 1;
            u32x4 va[4], vb[4] = {};
            unsigned sa[4], sb[4], da[4], db[4] = {OOR, OOR, OOR, OOR};
            __amdgpu_buffer_rsrc_t rs, rd = rl;
#define CHOOSE() do { \
                ck = 0; csig = 0; \
                if (h2 && r2) { \
                    const unsigned o = tl2 * 131072u + (p2 & 15u) * 8192u; \
                    ck = 4; cso = PT_OFF + o; cdo = RT_OFF + o; csig = (unsigned)(L_TILE + tl2) * 128u + 64u + 3u; cseg = 128u + (p2 >> 4); csegn = 16u; h2 = false; \
                    PROG(1, p2); \
                } else if (nk && nrdy) { \
                    ck = nk; cso = nso; cdo = ndo; csig = nsig; cseg = nseg & 255u; csegn = nsegn; nk = 0; \
                    PROG((nseg >> 28) ? 2 : 0, (nseg >> 8) & 0xfffffu); \
                } \
                rs = RSRC(ck == 1 ? (char*)P.x : ck == 2 ? (char*)P.w : ck == 6 ? (char*)P.out[e] : (char*)P.ws[e], ck == 6 ? OUT_BYTES : ck >= 3 ? WS_END : IN_BYTES); \
                const bool sr = ck == 2 || ck == 6; \
                const unsigned sv = (ck ? cso : OOR) + (sr ? rlane : llane), ss = sr ? 32768u : ck == 1 ? 2048u : 1024u; \
                _Pragma("unroll") for (int k = 0; k < 4; ++k) { sa[k] = sv + k * ss; sb[k] = sv + (k + 4) * ss; } \
                asm volatile("" : "+v"(sa[0]), "+v"(sa[1]), "+v"(sa[2]), "+v"(sa[3]), "+v"(sb[0]), "+v"(sb[1]), "+v"(sb[2]), "+v"(sb[3])); \
            } while (0)
            REFILL();
            CHOOSE();
            for (;;) {
#pragma unroll
                for (int k = 0; k < 4; ++k) va[k] = __builtin_amdgcn_raw_buffer_load_b128(rs, sa[k], 0, 17  );
                __builtin_amdgcn_sched_barrier(0);
                if (!ok) {
#pragma unroll
                    for (int k = 0; k < 4; ++k) vb[k] = (u32x4){0x7fc07fc0u, 0x7fc07fc0u, 0x7fc07fc0u, 0x7fc07fc0u};
                }
#pragma unroll
                for (int k = 0; k < 4; ++k) __builtin_amdgcn_raw_buffer_store_b128(vb[k], rd, db[k], 0, 17  );
                __builtin_amdgcn_sched_barrier(0);
                call = UNI(callv) + 1u;
                if (ck == 0 && nk == 0 && !more && !h2 && tk2s == 0 && !q2 && psig == 0u && qsig == 0u) break;
                REFILL();
                if (ck) idle = 0;
                else {
                    if (idle == 0u) t0 = __builtin_amdgcn_s_memrealtime(); else __builtin_amdgcn_s_sleep(K_SLEEP);
                    if ((++idle & 63u) == 0u && ok && gave_up(t0, tmo, call)) { ok = false; nrdy = true; r2 = true; }
                }
                (void)ATOMV(RSRC(P.ws[qsig & 7u], CTL_BYTES), (qsig && UNI(qv) == qsegn * call - 1u) ? qsig & ~7u : OOR, qsegn);
                __builtin_amdgcn_sched_barrier(0);
                const bool cl1 = tks == 0 && more, cl2 = q2 && tk2s == 0, nw = nk && !nrdy, w2 = h2 && !r2;
                const unsigned tkv1 = ATOMV(rl, cl1 ? (pool ? (unsigned)(L_HEAD + 32) * 128u : (unsigned)head) : OOR, (pool && e == 4) ? 0x10000u : 1u);
                const unsigned tkv2 = ATOMV(rl, cl2 ? (unsigned)head + 128u : OOR, 1);
                const unsigned pv = __builtin_amdgcn_raw_buffer_load_b32(rl, (lm0 & (nw ? ndep : OOR)) | (lm1 & (w2 ? (unsigned)(L_TILE + tl2) * 128u + 32u : OOR)) |
                                                                             (lm2 & (gate ? (unsigned)(L_PANEL + (x - 1) * 32 + K_GATE) * 128u : OOR)) | lmx, 0, 17);
                rd = RSRC(ck >= 5 ? (char*)P.out[3] : ck >= 3 ? myws : (char*)P.ws[e], ck >= 5 ? OUT_BYTES : WS_END);
                unsigned qoff = l0 | (psig ? (unsigned)scnt + pseg * 4u : OOR);
                const unsigned dv0 = (ck ? cdo : OOR) + (ck >= 5 ? rlane : llane), ds = ck >= 5 ? 32768u : (ck & 1) ? 2048u : 1024u;
#pragma unroll
                for (int k = 0; k < 4; ++k) { da[k] = dv0 + k * ds; db[k] = dv0 + (k + 4) * ds; }
                asm volatile("" : "+v"(da[0]), "+v"(da[1]), "+v"(da[2]), "+v"(da[3]), "+v"(db[0]), "+v"(db[1]), "+v"(db[2]), "+v"(db[3]), "+v"(qoff));
                __builtin_amdgcn_sched_barrier(0);
                unsigned one = 1u;
                asm volatile("s_waitcnt vmcnt(0)" : "+v"(one) : "v"(pv) : "memory");
#pragma unroll
                for (int k = 0; k < 4; ++k) vb[k] = __builtin_amdgcn_raw_buffer_load_b128(rs, sb[k], 0, 17);
                __builtin_amdgcn_sched_barrier(0);
                if (!ok) {
#pragma unroll
                    for (int k = 0; k < 4; ++k) va[k] = (u32x4){0x7fc07fc0u, 0x7fc07fc0u, 0x7fc07fc0u, 0x7fc07fc0u};
                }
#pragma unroll
                for (int k = 0; k < 4; ++k) __builtin_amdgcn_raw_buffer_store_b128(va[k], rd, da[k], 0, 17);
                __builtin_amdgcn_sched_barrier(0);
                qv = (unsigned)__builtin_amdgcn_raw_ptr_buffer_atomic_add_i32((int)one, rl, qoff, 0, 16);
                __builtin_amdgcn_sched_barrier(0);
                qsig = psig; qsegn = psegn;
                psig = csig; pseg = cseg; psegn = csegn;
                const unsigned okm = (unsigned)__ballot((int)(pv - ((lm0 & (nneed * call)) | (lm1 & (8u * call)) | (lm2 & (64u * call)))) >= 0);
                r2 = r2 || (w2 && (okm & 2u));
                gate = gate && !(okm & 4u);
                nrdy = nrdy || (nw && !gate && (okm & 1u));
                if (cl1) { tk = UNI(tkv1) + (pool ? 0u : 2u * CE_WGS); tks = 2; tkpool = pool; if (!pool && tk >= nfix) { pool = true; tks = 0; } }
                if (cl2) { tk2 = UNI(tkv2); tk2s = 2; if (tk2 >= (unsigned)Q2_ITEMS) { tk2s = 0; q2 = false; STAMP(2); } }
                CHOOSE();
                __builtin_amdgcn_sched_barrier(0);
                asm volatile("" :: "v"(vb[3].x) : "memory");
                __builtin_amdgcn_sched_barrier(0);
            }
        }
        STAMP(3);
        __syncthreads();
        wg_exit(ctl, call, wid, lane);
        return;
    }

    const unsigned call = (unsigned)__builtin_amdgcn_readfirstlane(ld_relaxed_agent(ctl + L_CALL * 32)) + 1u;
    stage_inputs(P, myws, call, xl, lane, slot >= 32 - K_STW ? (wid < K_STV ? 1 << 20 : 0) : (wid < K_ST1 ? 1 : 0));
    asm volatile("s_waitcnt vmcnt(0)" ::: "memory");
    const int njobs = (int)P.njob[xl];
    for (int job = slot - CE_WGS, round = 0; job < njobs; ++round) {
        const unsigned jw = (P.job[(xl * MAXJ + job) >> 1] >> ((job & 1) * 16)) & 0xffffu;
        if (jw >> 8) job = gemm_job<true>(P, lds, myws, call, (int)(jw & 255u), (int)(jw >> 8) & 3, xl, round);
        else job = gemm_job<false>(P, lds, myws, call, (int)jw, 0, xl, round);
    }
    wg_exit(ctl, call, wid, lane);
}

}

extern "C" void kernel_launch(int rank, int nranks, ncclComm_t comm,
                              hipStream_t stream,
                              void* const* d_in, const int* in_sizes, int n_in,
                              void* d_out, int out_size,
                              const int* in_dtypes, int out_dtype,
                              void* const* d_out_all,
                              void* const* d_ws_all, size_t ws_bytes) {
    if (nranks != NR || (unsigned)rank >= (unsigned)NR || n_in < 2 || ws_bytes < WS_END || (size_t)out_size != (size_t)GM * GN ||
        (size_t)in_sizes[0] != (size_t)GM * KS || (size_t)in_sizes[1] != (size_t)KS * GN) {
        fprintf(stderr, "kernel_launch: unexpected problem shape\n");
        return;
    }
    const int member[2][4] = {{0, 2, 3, 6}, {1, 4, 5, 7}};
    Params P = {};
    P.x = d_in[0]; P.w = d_in[1]; P.g = 0; P.a = 0;
    for (int g = 0; g < 2; ++g) for (int a = 0; a < 4; ++a) if (member[g][a] == rank) { P.g = g; P.a = a; }
    for (int e = 0; e < 8; ++e) {
        const int peer = e < 3 ? member[P.g][P.a ^ (e + 1)] : (e == 3 ? rank : member[1 - P.g][P.a ^ (e - 4)]);
        P.out[e] = d_out_all[peer]; P.ws[e] = d_ws_all[peer];
    }
    unsigned char* tb = (unsigned char*)P.tab;
    unsigned short* jb = (unsigned short*)P.job;
    int n = 0, np[2] = {0, 0}, nx[8] = {0, 0, 0, 0, 0, 0, 0, 0};
    for (int k = 0; k < 16; ++k)
        for (int r = 0; r < 2 * k + 1; ++r, ++n) {
            const int I = r < k ? k : r - k, J = r < k ? r : k;
            const unsigned char tl = (unsigned char)(I << 4 | J);
            const int par = (I + J) & 1;
            tb[par * 128 + np[par]++] = tl;
            if (n < K_SMALL) { const int xs = n & 7; for (int q = 0; q < 4; ++q) jb[xs * MAXJ + nx[xs]++] = (unsigned short)(tl | (4 + q) << 8); }
            else { const int x = (I & 3) << 1 | ((J >> 1) & 1); jb[x * MAXJ + nx[x]++] = tl; }
        }
    for (int x = 0; x < 8; ++x) P.njob[x] = (unsigned)nx[x];
    hipLaunchKernelGGL(gemm_ar_kernel, dim3(GRID), dim3(512), 0, stream, P);
    HIP_CHECK(hipGetLastError());
}
```

```cpp
#include <hip/hip_runtime.h>
#include <rccl/rccl.h>
#include <cstdio>

#define HIP_CHECK(expr) do { hipError_t _e = (expr); if (_e != hipSuccess) { \
    fprintf(stderr, "HIP error %s at %s:%d\n", hipGetErrorString(_e), __FILE__, __LINE__); \
    return; } } while (0)

namespace {

#define LAS __attribute__((address_space(3)))
typedef short bf16x8 __attribute__((ext_vector_type(8)));
typedef short s16x4 __attribute__((ext_vector_type(4)));
typedef float f32x4 __attribute__((ext_vector_type(4)));
typedef unsigned u32x4 __attribute__((ext_vector_type(4)));

constexpr int NR = 8;
constexpr int GM = 8192, GN = 8192;
constexpr int KS = 1024;

#define K_PUSH 2
#define K_GATE 0
#define K_GRP 2
#define K_ST1 2
#define K_STW 4
#define K_STV 4
#define K_NA 36
#define K_NL 118
#define K_SMALL 49
#define K_SLEEP 8
#define K_CE 14
constexpr int NPUSH = K_PUSH, NA = K_NA, NL = K_NL;
static_assert(NA >= 1 && NA < NL && NL <= 128 && NPUSH >= 0 && NPUSH < 32 && K_GATE < NPUSH, "knobs");

constexpr unsigned CTL_BYTES = 1u << 20;
constexpr unsigned XS_OFF = CTL_BYTES, XS_SLOT = 16u * 524288u;
constexpr unsigned WS_OFF = XS_OFF + 3u * XS_SLOT;
constexpr unsigned IN_OFF = WS_OFF + 3u * XS_SLOT;
constexpr unsigned PT_OFF = IN_OFF + 128u * 262144u;
constexpr unsigned RT_OFF = PT_OFF + 256u * 131072u;
constexpr unsigned FT_OFF = RT_OFF + 256u * 131072u;
constexpr unsigned WS_END = FT_OFF + 256u * 131072u;
constexpr unsigned OUT_BYTES = (unsigned)((size_t)GM * GN * 2);
constexpr unsigned IN_BYTES = (unsigned)((size_t)GM * KS * 2);

constexpr int L_TMO = 1, L_EXIT = 2, L_CALL = 3  , L_STG = 16  , L_STGC = 320  ,
              L_LAND = 32  , L_HEAD = 64  ,
              L_PANEL = 128  , L_FIN = 256  ,
              L_TILE = 512  , L_SCNT = 1024  ;
constexpr int CE_WGS = K_CE;
constexpr int G_WGS = 32 - CE_WGS;
constexpr int GRID = 256;
constexpr unsigned long long DEADLINE_TICKS = 50000000ull;
constexpr unsigned long long WATCH_DELAY = 1000;

constexpr int IN_ITEMS = 32 * 32;
constexpr int Q2_ITEMS = 128 * 16;
constexpr int FIX6 = IN_ITEMS + 128 * 16, FIX4 = NA * 16;
constexpr int POOL_ITEMS = (128 - NA) * 16, FWD_MAX = (NL - NA) * 16;
constexpr int Q3L_ITEMS = 128 * 16;
constexpr int STG_ITEMS = (32 - NPUSH) * 8;
constexpr int STG_GRP = K_GRP;
static_assert((32 - NPUSH) % STG_GRP == 0 && (32 - NPUSH) / STG_GRP <= 32, "staging groups");

constexpr int BK = 64, HALF = 128, HTB = HALF * BK * 2, STAGE_BYTES = 8 * HTB;

#define STAMP_TK do {} while (0)
#define STAMP(i) do {} while (0)
#define PROG(q, it) do {} while (0)

constexpr int MAXJ = 96;
static_assert((K_SMALL + 7) / 8 * 4 + 32 <= MAXJ, "job table");
struct Params {
    const void* x; const void* w;
    void* out[8]; void* ws[8];
    int g, a;
    unsigned njob[8], job[8 * MAXJ / 2];
    unsigned tab[64];
};
typedef const __attribute__((address_space(4))) Params CParams;
#define TAB(i) ((P.tab[(unsigned)(i) >> 2] >> (((unsigned)(i) & 3u) * 8u)) & 255u)
#define RSRC(p, n) __builtin_amdgcn_make_buffer_rsrc((void*)(p), 0, (int)(n), 0x00020000)
#define UNI(x) (unsigned)__builtin_amdgcn_readfirstlane((int)(x))

__device__ __forceinline__ int lds_byte(int r, int c) { const int st = (r >> 4) * 2 + (c >> 5), rr = r & 15, cc = c & 31, ob = rr * 64 + cc * 2; return st * 1024 + (ob ^ (((ob >> 9) & 1) << 5)); }
__device__ __forceinline__ void stage_rc(int b, int& R, int& C) { const int st = b / 1024, sb = b % 1024, swz = sb ^ (((sb >> 9) & 1) << 5); R = (st >> 1) * 16 + swz / 64; C = (st & 1) * 32 + (swz % 64) / 2; }
__device__ __forceinline__ unsigned wswz(unsigned row) { return ((row & 3u) << 2) | ((row >> 2) & 3u); }
typedef float f32x2 __attribute__((ext_vector_type(2)));
typedef __bf16 bf16x2 __attribute__((ext_vector_type(2)));
__device__ __forceinline__ unsigned cvt_pk_bf16(float lo, float hi) {
    return __builtin_bit_cast(unsigned, __builtin_convertvector((f32x2){lo, hi}, bf16x2));
}
__device__ __forceinline__ unsigned panel_base(int ab, int I) {
    const int j = I >> 1;
    return (unsigned)((j << 2) | ((2 * ab + (I & 1) + (0xb114 >> (2 * j))) & 3)) * 256u;
}
__device__ __forceinline__ unsigned out_offset(int a, unsigned tl, unsigned st) {
    return (panel_base(a >> 1, (int)(tl >> 4)) + st * 16u) * 16384u + panel_base(a & 1, (int)(tl & 15u)) * 2u;
}

__device__ __forceinline__ unsigned ld_relaxed_sys(const unsigned* p) { return __hip_atomic_load(p, __ATOMIC_RELAXED, __HIP_MEMORY_SCOPE_SYSTEM); }
__device__ __forceinline__ unsigned ld_relaxed_agent(const unsigned* p) { return __hip_atomic_load(p, __ATOMIC_RELAXED, __HIP_MEMORY_SCOPE_AGENT); }

__device__ __forceinline__ bool gave_up(unsigned long long t0, unsigned* tmo, unsigned call) {
    if ((unsigned)__builtin_amdgcn_readfirstlane(ld_relaxed_agent(tmo)) == call || __builtin_amdgcn_s_memrealtime() - t0 > DEADLINE_TICKS) {
        __hip_atomic_store(tmo, call, __ATOMIC_RELAXED, __HIP_MEMORY_SCOPE_AGENT);
        return true;
    }
    return false;
}
__device__ __forceinline__ unsigned add_bf16x2(unsigned a, unsigned b) {
    return cvt_pk_bf16(__uint_as_float(a << 16) + __uint_as_float(b << 16), __uint_as_float(a & 0xffff0000u) + __uint_as_float(b & 0xffff0000u));
}

__device__ __forceinline__ void wg_exit(unsigned* ctl, unsigned call, int wid, int lane) {
    if (wid != 0) return;
    const unsigned t = (unsigned)__builtin_amdgcn_raw_ptr_buffer_atomic_add_i32(1, __builtin_amdgcn_make_buffer_rsrc(ctl, 0, (int)CTL_BYTES, 0x00020000), lane == 0 ? (unsigned)L_EXIT * 128u : 0x7fffff00u, 0, 0);
    if (((unsigned)__builtin_amdgcn_readfirstlane((int)t) & (unsigned)(GRID - 1)) == (unsigned)(GRID - 1)) {
        if (lane < 34) __hip_atomic_store(ctl + (lane == 33 ? L_CALL : L_HEAD + lane) * 32, lane == 33 ? call : 0u, __ATOMIC_RELAXED, __HIP_MEMORY_SCOPE_AGENT);
    }
}

template <bool SMALL>
__device__ __forceinline__ int gemm_job(CParams& P, LAS unsigned char* lds, char* myws, unsigned call, int tl, int qd, int xl, int round) {
    int tid = threadIdx.x; asm volatile("" : "+v"(tid));
    const int wid = __builtin_amdgcn_readfirstlane(tid >> 6), lane = tid & 63;
    const int a1 = P.a >> 1, a0 = P.a & 1, ti = tl >> 4, tj = tl & 15;
    unsigned* ctl = (unsigned*)myws;
    unsigned* tmo = ctl + L_TMO * 32;
    const int wr = wid >> 2, wc = wid & 3, fr = lane & 15, fq = lane >> 4;
    unsigned voffA[2], voffB[2];
#pragma unroll
    for (int i = 0; i < 2; ++i) { int R, C; stage_rc(tid * 16 + i * 8192, R, C); voffA[i] = (unsigned)(R * KS + C) * 2u; }
#define SET_VOFFB(sh) do { int tz = tid; asm volatile("" : "+v"(tz)); _Pragma("unroll") for (int i = 0; i < 2; ++i) { \
        const unsigned row = (unsigned)(tz >> 4) + 32u * i, ch = ((unsigned)tz & 15u) ^ wswz(row); voffB[i] = (row << (sh)) + ch * 16u; } \
        kstepB = (size_t)(SMALL ? 2 * BK : BK) << (sh); hstepB = SMALL ? (size_t)BK << (sh) : (size_t)HALF * 2; } while (0)
    size_t kstepB, hstepB;
    constexpr size_t kstepA = SMALL ? BK * 4 : BK * 2, hstepA = SMALL ? BK * 2 : (size_t)HALF * KS * 2;
    const unsigned qa = SMALL ? (unsigned)(qd >> 1) * (unsigned)(HALF * KS * 2) : 0u, qb = SMALL ? (unsigned)(qd & 1) * 256u : 0u;
    const unsigned ldsw = (unsigned)wid * 1024u;
    const int aoff = lds_byte(wr * 64 + fr, fq * 8);
    int boff[2][2];
#pragma unroll
    for (int n = 0; n < 2; ++n)
#pragma unroll
        for (int t = 0; t < 2; ++t) {
            const unsigned q4 = (unsigned)(lane & 15) >> 2, p4 = (unsigned)lane & 3u, row = 8u * fq + 4u * t + q4, ch = 4u * wc + 2u * n + (p4 >> 1);
            boff[n][t] = (int)((unsigned)(size_t)lds + 4u * HTB + 256u * row + 16u * (ch ^ wswz(row)) + 8u * (p4 & 1u));
        }
#define SA(b, h) (((b) * 2 + (h)) * HTB)
#define SB(b, h) ((4 + (b) * 2 + (h)) * HTB)
#define STAGE(bufoff, gbase, voff) do { _Pragma("unroll") for (int _i = 0; _i < 2; ++_i) \
        __builtin_amdgcn_global_load_lds((const unsigned*)((const char*)(gbase) + (voff)[_i]), (LAS unsigned*)(lds + (bufoff) + ldsw + _i * 8192), 16, 0, 0); } while (0)
#define LDA(dst, b, h) do { _Pragma("unroll") for (int m = 0; m < 4; ++m) _Pragma("unroll") for (int k = 0; k < 2; ++k) dst[m][k] = *(const LAS bf16x8*)(lds + SA(b, h) + aoff + m * 2048 + k * 1024); } while (0)
#define TRR(dst, base, off) asm volatile("ds_read_b64_tr_b16 %0, %1 offset:%2" : "=&v"(dst) : "v"(base), "i"(off) : "memory")
#define LDB(raw, b, h) do { \
        TRR(raw[0], boff[0][0], SB(b, h) - 4 * HTB); TRR(raw[1], boff[0][1], SB(b, h) - 4 * HTB); TRR(raw[2], boff[0][0], SB(b, h) - 4 * HTB + 8192); TRR(raw[3], boff[0][1], SB(b, h) - 4 * HTB + 8192); \
        TRR(raw[4], boff[1][0], SB(b, h) - 4 * HTB); TRR(raw[5], boff[1][1], SB(b, h) - 4 * HTB); TRR(raw[6], boff[1][0], SB(b, h) - 4 * HTB + 8192); TRR(raw[7], boff[1][1], SB(b, h) - 4 * HTB + 8192); } while (0)
#define WAIT_B(raw, dst) do { asm volatile("s_waitcnt lgkmcnt(0)" : "+v"(raw[0]), "+v"(raw[1]), "+v"(raw[2]), "+v"(raw[3]), "+v"(raw[4]), "+v"(raw[5]), "+v"(raw[6]), "+v"(raw[7]) :: "memory"); \
        _Pragma("unroll") for (int n = 0; n < 2; ++n) _Pragma("unroll") for (int k = 0; k < 2; ++k) dst[n][k] = __builtin_shufflevector(raw[n * 4 + k * 2], raw[n * 4 + k * 2 + 1], 0, 1, 2, 3, 4, 5, 6, 7); } while (0)
#define MMA(ai, bj, At, Bt) do { if (!SMALL || (ai) == (bj)) { __builtin_amdgcn_s_setprio(1); _Pragma("unroll") for (int m = 0; m < 4; ++m) _Pragma("unroll") for (int n = 0; n < 2; ++n) _Pragma("unroll") for (int k = 0; k < 2; ++k) \
        acc[SMALL ? 0 : ai][SMALL ? 0 : bj][m][n] = __builtin_amdgcn_mfma_f32_16x16x32_bf16(Bt[n][k], At[m][k], acc[SMALL ? 0 : ai][SMALL ? 0 : bj][m][n], 0, 0, 0); __builtin_amdgcn_s_setprio(0); } } while (0)
#define WAIT_V(n) asm volatile("s_waitcnt vmcnt(" #n ")" ::: "memory")
#define WAIT_L(n) asm volatile("s_waitcnt lgkmcnt(" #n ")" ::: "memory")
#define BAR __builtin_amdgcn_s_barrier()
#define SCHED __builtin_amdgcn_sched_barrier(0)
    constexpr int NH2 = SMALL ? 1 : 2;
    f32x4 acc[NH2][NH2][4][2];
#pragma unroll
    for (int a = 0; a < NH2; ++a)
#pragma unroll
        for (int b = 0; b < NH2; ++b)
#pragma unroll
            for (int m = 0; m < 4; ++m)
#pragma unroll
                for (int n = 0; n < 2; ++n) acc[a][b][m][n] = (f32x4){0.f, 0.f, 0.f, 0.f};
    bf16x8 At[4][2], B0[2][2], B1[2][2];
    s16x4 R0[8], R1[8];
    const char* cA = (const char*)P.x + (size_t)panel_base(a1, ti) * (KS * 2) + qa;
    const char* cB = (const char*)P.w + (size_t)panel_base(a0, tj) * 2 + qb;
    constexpr int nt = SMALL ? KS / BK / 2 : KS / BK;
    unsigned todo = 7u;
    SET_VOFFB(14);
    STAGE(SB(0, 0), cB, voffB); STAGE(SA(0, 0), cA, voffA); STAGE(SB(0, 1), cB + hstepB, voffB); STAGE(SA(0, 1), cA + hstepA, voffA);
    if (wr == 1) BAR;
    WAIT_V(4); BAR;
    STAGE(SB(1, 0), cB + kstepB, voffB); STAGE(SA(1, 0), cA + kstepA, voffA); STAGE(SB(1, 1), cB + hstepB + kstepB, voffB);
    WAIT_V(6); BAR;
    for (int s = 0; s < 4; ++s) {
        const char* nA = cA; const char* nB = cB;
        for (int t = 0; t < nt; t += 2) {
            const bool last = (t == nt - 2);
            if (last && s < 3) {
                if (wid == 0) {
                    STAMP(1 + 2 * s + 8 * round);
                    const unsigned* word = ctl + (L_PANEL + ((lane >> 1) % 3) * 32 + ((lane & 1) ? 2 * tj + 1 : 2 * ti)) * 32;
                    const unsigned long long t0 = __builtin_amdgcn_s_memrealtime();
                    unsigned pick;
                    for (unsigned polls = 1;; ++polls) {
                        const unsigned mk = (unsigned)__ballot((int)(ld_relaxed_sys(word) - 64u * call) >= 0);
                        const unsigned both = mk & (mk >> 1);
                        pick = ((both & 1u) | ((both >> 1) & 2u) | ((both >> 2) & 4u)) & todo;
                        if (pick) break;
                        if ((polls & 63u) == 0u && gave_up(t0, tmo, call)) { pick = todo; break; }
                        __builtin_amdgcn_s_sleep(4);
                    }
                    STAMP(2 + 2 * s + 8 * round);
                    *(LAS unsigned*)(lds + STAGE_BYTES + 16) = (unsigned)__builtin_ctz(pick);
                    __builtin_amdgcn_fence(__ATOMIC_ACQUIRE, "");
                    asm volatile("s_waitcnt vmcnt(0) lgkmcnt(0)" ::: "memory");
                }
                asm volatile("" ::: "memory"); BAR; asm volatile("" ::: "memory");
                unsigned slv;
                asm volatile("ds_read_b32 %0, %1\n\ts_waitcnt lgkmcnt(0)" : "=&v"(slv) : "v"((unsigned)(size_t)lds + STAGE_BYTES + 16) : "memory");
                const unsigned sl = (unsigned)__builtin_amdgcn_readfirstlane(slv) & 3u;
                todo &= ~(1u << sl);
                nA = myws + XS_OFF + sl * XS_SLOT + (unsigned)ti * 524288u + qa;
                nB = myws + WS_OFF + sl * XS_SLOT + (unsigned)tj * 524288u + qb;
                if (s == 0) SET_VOFFB(9);
            }
            const char* a1p = cA + (size_t)(t + 1) * kstepA;
            const char* a2 = last ? nA : cA + (size_t)(t + 2) * kstepA; const char* b2 = last ? nB : cB + (size_t)(t + 2) * kstepB;
            const char* a3 = a2 + kstepA; const char* b3 = b2 + kstepB;
            LDB(R0, 0, 0); SCHED; LDA(At, 0, 0); STAGE(SA(1, 1), a1p + hstepA, voffA);
            WAIT_L(8); BAR; WAIT_B(R0, B0); MMA(0, 0, At, B0); BAR; SCHED;
            LDB(R1, 0, 1); STAGE(SB(0, 0), b2, voffB);
            BAR; WAIT_B(R1, B1); MMA(0, 1, At, B1); BAR; if (SMALL) SCHED;
            LDA(At, 0, 1); STAGE(SA(0, 0), a2, voffA);
            BAR; WAIT_L(0); MMA(1, 0, At, B0); BAR; SCHED;
            STAGE(SB(0, 1), b2 + hstepB, voffB);
            WAIT_V(6); BAR; MMA(1, 1, At, B1); BAR;
            LDB(R0, 1, 0); SCHED; LDA(At, 1, 0); STAGE(SA(0, 1), a2 + hstepA, voffA);
            WAIT_L(8); BAR; WAIT_B(R0, B0); MMA(0, 0, At, B0); BAR; SCHED;
            LDB(R1, 1, 1); STAGE(SB(1, 0), b3, voffB);
            BAR; WAIT_B(R1, B1); MMA(0, 1, At, B1); BAR; if (SMALL) SCHED;
            LDA(At, 1, 1); STAGE(SA(1, 0), a3, voffA);
            BAR; WAIT_L(0); MMA(1, 0, At, B0); BAR; SCHED;
            STAGE(SB(1, 1), b3 + hstepB, voffB);
            WAIT_V(6); BAR; MMA(1, 1, At, B1); BAR;
        }
        cA = nA; cB = nB;
    }
    WAIT_V(0);
    if (wr == 0) BAR;
    STAMP(7 + 8 * round);
    const __amdgpu_buffer_rsrc_t rc = __builtin_amdgcn_make_buffer_rsrc(myws, 0, (int)CTL_BYTES, 0x00020000);
    const unsigned nj = (unsigned)__builtin_amdgcn_raw_ptr_buffer_atomic_add_i32(1, rc, tid == 0 ? (unsigned)(L_HEAD + 16 + xl) * 128u : 0x7fffff00u, 0, 0);
    const bool bad = (unsigned)__builtin_amdgcn_readfirstlane(ld_relaxed_agent(tmo)) == call;
    const __amdgpu_buffer_rsrc_t ro = __builtin_amdgcn_make_buffer_rsrc(myws + PT_OFF + (unsigned)tl * 131072u, 0, 131072, 0x00020000);
    const unsigned obase = (unsigned)(64 * wr + fr + (SMALL ? (qd >> 1) * HALF : 0)) * 512u + (unsigned)(32 * wc + (fq & 1) * 16 + (fq >> 1) * 8) * 2u + qb;
#pragma unroll
    for (int ai = 0; ai < NH2; ++ai)
#pragma unroll
        for (int m = 0; m < 4; ++m)
#pragma unroll
            for (int bj = 0; bj < NH2; ++bj) {
                const f32x4 v0 = acc[ai][bj][m][0], v1 = acc[ai][bj][m][1];
                const unsigned x0 = cvt_pk_bf16(v0[0], v0[1]), x1 = cvt_pk_bf16(v0[2], v0[3]), y0 = cvt_pk_bf16(v1[0], v1[1]), y1 = cvt_pk_bf16(v1[2], v1[3]);
                const auto r0 = __builtin_amdgcn_permlane16_swap(x0, y0, false, false);
                const auto r1 = __builtin_amdgcn_permlane16_swap(x1, y1, false, false);
                u32x4 wv = {r0[0], r1[0], r0[1], r1[1]};
                if (bad) wv = (u32x4){0x7fc07fc0u, 0x7fc07fc0u, 0x7fc07fc0u, 0x7fc07fc0u};
                __builtin_amdgcn_raw_buffer_store_b128(wv, ro, obase + (unsigned)(ai * HALF + m * 16) * 512u + (unsigned)bj * 256u, 0, 17  );
            }
    if (wid == 0) *(LAS unsigned*)(lds + STAGE_BYTES) = (unsigned)__builtin_amdgcn_readfirstlane((int)nj);
    asm volatile("s_waitcnt vmcnt(0) lgkmcnt(0)" ::: "memory");
    BAR;
    const bool mine = ((ti + tj) & 1) == P.g;
    (void)__builtin_amdgcn_raw_ptr_buffer_atomic_add_i32(SMALL ? 2 : 8, __builtin_amdgcn_make_buffer_rsrc(P.ws[mine ? 3 : 4], 0, (int)CTL_BYTES, 0x00020000),
                                                         tid == 0 ? (unsigned)(L_TILE + tl) * 128u + (mine ? 0u : 32u) : 0x7fffff00u, 0, 16);
    STAMP(8 + 8 * round);
    unsigned njv;
    asm volatile("ds_read_b32 %0, %1\n\ts_waitcnt lgkmcnt(0)" : "=&v"(njv) : "v"((unsigned)(size_t)lds + STAGE_BYTES) : "memory");
    return __builtin_amdgcn_readfirstlane((int)njv) + G_WGS;
}

__device__ __forceinline__ unsigned in_offset(unsigned t, int ab, int kh, unsigned u) {
    return (t & 1u) ? ((unsigned)kh * 512u + u * 16u) * 16384u + panel_base(ab, (int)(t >> 1)) * 2u : (panel_base(ab, (int)(t >> 1)) + u * 8u) * 2048u + (unsigned)kh * 1024u;
}
__device__ __forceinline__ void stage_inputs(CParams& P, char* myws, unsigned call, int lb, int lane, int maxn) {
    const __amdgpu_buffer_rsrc_t rl = RSRC(myws, WS_END);
    const unsigned l0 = lane == 0 ? 0u : 0x7fffff00u, llane = (unsigned)lane * 16u, rlane = (unsigned)(lane >> 5) * 16384u + (unsigned)(lane & 31) * 16u;
    for (; maxn > 0; --maxn) {
        const unsigned it = UNI(__builtin_amdgcn_raw_ptr_buffer_atomic_add_i32(1, rl, l0, (L_HEAD + 24 + lb) * 128, 0));
        if (it >= (unsigned)STG_ITEMS) break;
        const unsigned pc = (it * 8u + (unsigned)lb) * 2u + (unsigned)NPUSH * 128u, t = pc >> 7, g = (t - (unsigned)NPUSH) / (unsigned)STG_GRP;
        const unsigned so = in_offset(t, (int)(pc >> 6) & 1, (int)(pc >> 5) & 1, pc & 31u) + ((t & 1u) ? rlane : llane), ss = (t & 1u) ? 32768u : 2048u;
        const __amdgpu_buffer_rsrc_t rs = RSRC((t & 1u) ? (char*)P.w : (char*)P.x, IN_BYTES);
        u32x4 v[16];
#pragma unroll
        for (int k = 0; k < 16; ++k) v[k] = __builtin_amdgcn_raw_buffer_load_b128(rs, so + k * ss, 0, 0);
#pragma unroll
        for (int k = 0; k < 16; ++k) __builtin_amdgcn_raw_buffer_store_b128(v[k], rl, IN_OFF + pc * 8192u + llane + k * 1024u, 0, 17);
        asm volatile("s_waitcnt vmcnt(0)" ::: "memory");
        if (UNI(__builtin_amdgcn_raw_ptr_buffer_atomic_add_i32(1, rl, l0, (L_STGC + (int)g) * 128, 0)) == (unsigned)(STG_GRP * 64) * call - 1u) {
            if (lane < 8 && lane != 3 && lane != 4) __hip_atomic_store((unsigned*)P.ws[lane] + (L_STG + lane) * 32 + g, call, __ATOMIC_RELAXED, __HIP_MEMORY_SCOPE_SYSTEM);
        }
    }
}

__global__ __launch_bounds__(512, 2) void gemm_ar_kernel(const Params) {
    CParams& P = *(CParams*)__builtin_amdgcn_kernarg_segment_ptr();
    __shared__ __attribute__((aligned(16))) unsigned char shm[STAGE_BYTES + 64];
    LAS unsigned char* lds = (LAS unsigned char*)shm;
    const int tid = threadIdx.x, wid = __builtin_amdgcn_readfirstlane(tid >> 6), lane = tid & 63;
    const int hm = P.g, qa = P.a;
    const int xl = blockIdx.x & 7, slot = blockIdx.x >> 3;
    char* myws = (char*)P.ws[3];
    unsigned* ctl = (unsigned*)myws;
    unsigned* tmo = ctl + L_TMO * 32;
    STAMP(0);

    if (slot < CE_WGS) {
        int e = (int)((unsigned)__builtin_amdgcn_s_getreg(20 | (3 << 11)) & 7u);
        bool active = wid == 0 || e == 3;
        const __amdgpu_buffer_rsrc_t rl = RSRC(myws, WS_END);
        const unsigned l0 = lane == 0 ? 0u : 0x7fffff00u;
#define ATOM(rs, off, v, aux) (unsigned)__builtin_amdgcn_raw_ptr_buffer_atomic_add_i32((int)(v), rs, l0, off, aux)
#define ATOMV(rs, off, v) (unsigned)__builtin_amdgcn_raw_ptr_buffer_atomic_add_i32((int)(v), rs, l0 | (off), 0, 16)
        if (wid == 7) {
            const unsigned long long t0 = __builtin_amdgcn_s_memrealtime();
            while (__builtin_amdgcn_s_memrealtime() - t0 < WATCH_DELAY) __builtin_amdgcn_s_sleep(16);
            if (UNI(ld_relaxed_agent(ctl + (L_HEAD + xl * 2) * 32)) == 0u) { e = xl; active = true; }
        }
        const unsigned llane = (unsigned)lane * 16u;
        const unsigned rlane = (unsigned)(lane >> 5) * 16384u + (unsigned)(lane & 31) * 16u;
        const int head = (L_HEAD + e * 2) * 128, scnt = (L_SCNT + e * 8) * 128;
        unsigned tkv = 0, call = 0;
        if (active) { if (e != 3) tkv = ATOM(rl, head, 2, 0); call = UNI(__builtin_amdgcn_raw_buffer_load_b32(rl, L_CALL * 128u, 0, 16)) + 1u; }
        STAMP(4);
        bool ok = true;
        unsigned idle = 0; unsigned long long t0 = 0;
        if (active && e == 3) {
            if (wid == 7) for (int k = 0; k < 8; ++k) stage_inputs(P, myws, call, (slot + k) & 7, lane, 1 << 20);
            tkv = ATOM(rl, head, 1, 0);
            const __amdgpu_buffer_rsrc_t ro = RSRC(P.out[3], OUT_BYTES);
            for (unsigned it = UNI(tkv); it < (unsigned)Q3L_ITEMS; it = UNI(tkv)) {
                const unsigned tl = TAB(hm * 128 + (it >> 4)), st = it & 15u, dep = (unsigned)(L_TILE + tl) * 128u;
                t0 = __builtin_amdgcn_s_memrealtime();
                for (idle = 1; ok; ++idle) {
                    const unsigned v = __builtin_amdgcn_raw_buffer_load_b32(rl, dep + (unsigned)(lane & 1) * 64u, 0, 17);
                    if (__all((int)(v - ((lane & 1) ? 16u : 8u) * call) >= 0)) break;
                    if ((idle & 63u) == 0u && gave_up(t0, tmo, call)) ok = false;
                    __builtin_amdgcn_s_sleep(K_SLEEP);
                }
                __builtin_amdgcn_fence(__ATOMIC_ACQUIRE, "wavefront");
                PROG(2, it);
                const unsigned sv = tl * 131072u + st * 8192u + llane, dv = out_offset(qa, tl, st) + rlane;
                u32x4 va[8], vb[8];
#pragma unroll
                for (int k = 0; k < 8; ++k) { va[k] = __builtin_amdgcn_raw_buffer_load_b128(rl, PT_OFF + sv + k * 1024u, 0, 17); vb[k] = __builtin_amdgcn_raw_buffer_load_b128(rl, RT_OFF + sv + k * 1024u, 0, 17); }
#pragma unroll
                for (int k = 0; k < 8; ++k) {
                    u32x4 o = {add_bf16x2(va[k].x, vb[k].x), add_bf16x2(va[k].y, vb[k].y), add_bf16x2(va[k].z, vb[k].z), add_bf16x2(va[k].w, vb[k].w)};
                    if (!ok) o = (u32x4){0x7fc07fc0u, 0x7fc07fc0u, 0x7fc07fc0u, 0x7fc07fc0u};
                    __builtin_amdgcn_raw_buffer_store_b128(o, rl, FT_OFF + sv + k * 1024u, 0, 17);
                    __builtin_amdgcn_raw_buffer_store_b128(o, ro, dv + k * 32768u, 0, 16);
                }
                tkv = ATOM(rl, head, 1, 0);
                asm volatile("s_waitcnt vmcnt(0)" ::: "memory");
                if ((UNI(ATOM(rl, scnt + (int)(it >> 4) * 4, 1, 0)) & 15u) == 15u) {
                    if (lane < 8 && lane != 3) __hip_atomic_store((unsigned*)P.ws[lane] + (L_FIN + lane * 4) * 32 + (it >> 4), call, __ATOMIC_RELAXED, __HIP_MEMORY_SCOPE_SYSTEM);
                }
            }
        } else if (active) {
            const int x = e < 3 ? e + 1 : e - 4, hp = e < 3 ? hm : hm ^ 1, qp = qa ^ x;
            const unsigned nfix = e == 4 ? (unsigned)FIX4 : (unsigned)FIX6;
            constexpr unsigned OOR = 0x7fff0000u;
            bool more = true, spare = true, pool = false, tkpool = false;
            int tks = 2, tk2s = 0;
            unsigned tk = UNI(tkv), tk2 = 0;
            bool q2 = e == 4;
            int nk = 0; unsigned nso = 0, ndo = 0, ndep = OOR, nneed = 0, nsig = 0, nseg = 0, nsegn = 1; bool nrdy = false;
            bool h2 = false, r2 = false; unsigned p2 = 0, tl2 = 0;
            unsigned psig = 0, pseg = 0, psegn = 1, qsig = 0, qsegn = 1, qv = 0;
            const unsigned lm0 = lane == 0 ? ~0u : 0u, lm1 = lane == 1 ? ~0u : 0u, lm2 = lane == 2 ? ~0u : 0u, lmx = lane > 2 ? OOR : 0u;
            bool gate = K_GATE >= 0 && e != 4;
#define REFILL() do { \
                if (nk == 0 && tks == 2) { \
                    tks = 0; nsig = 0; nseg = 0; nneed = call; \
                    if (!tkpool && tk >= nfix) pool = true; \
                    else if (!tkpool && e != 4 && tk < (unsigned)IN_ITEMS) {       \
                        const unsigned t = tk >> 5, u = tk & 31u, half = e < 3 ? 0u : 1u; const bool push = t < (unsigned)NPUSH; \
                        nk = (push ? 1 : 3) + (int)(t & 1u); \
                        nso = push ? in_offset(t, (t & 1u) ? qp & 1 : qp >> 1, hp, u) : IN_OFF + (((t * 2u + (unsigned)((t & 1u) ? qa & 1 : qa >> 1)) * 2u + (unsigned)hm) * 32u + u) * 8192u; \
                        ndo = (unsigned)(x - 1) * XS_SLOT + (t >> 1) * 524288u + ((t & 1u) ? WS_OFF + half * 262144u + u * 8192u : XS_OFF + u * 16384u + half * 1024u); \
                        nseg = t; nsegn = 32u; nsig = (unsigned)(L_PANEL + (x - 1) * 32 + (int)t) * 128u + (push ? (unsigned)e : 3u); \
                        ndep = (unsigned)(L_STG + e) * 128u + (push ? 0u : (t - (unsigned)NPUSH) / (unsigned)STG_GRP * 4u); nrdy = push || !ok; \
                    } else { \
                        unsigned r; bool none = false;                 \
                        const bool fwd = tkpool && e != 4; \
                        if (tkpool) { const unsigned lo = tk & 0xffffu, hi = tk >> 16, loc = lo < (unsigned)FWD_MAX ? lo : (unsigned)FWD_MAX; \
                            none = loc + hi >= (unsigned)POOL_ITEMS || (fwd && lo >= (unsigned)FWD_MAX); r = none ? 0u : fwd ? (unsigned)(NA * 16) + lo : 2047u - hi; } \
                        else if (e == 4) r = tk; \
                        else { r = tk - (unsigned)IN_ITEMS; if (r < (unsigned)(NL * 16)) { nseg = 64u; nsegn = (unsigned)(NL * 16); nsig = (unsigned)(L_LAND + e2) * 128u + (unsigned)e2; } } \
                        const unsigned tl = TAB((fwd ? hm ^ 1 : hp) * 128 + (int)(r >> 4)), st = r & 15u; \
                        nk = none ? 0 : fwd ? 6 : 5; \
                        ndo = out_offset(fwd ? qa : qp, tl, st); \
                        nso = fwd ? ndo : FT_OFF + tl * 131072u + st * 8192u; \
                        ndep = fwd ? (unsigned)(L_LAND + e) * 128u : (unsigned)(L_FIN + e * 4) * 128u + (r >> 4) * 4u; \
                        nneed = fwd ? (unsigned)(NL * 16) * call : call; \
                        nrdy = !ok; \
                        if (none) { more = false; STAMP(29); } \
                    } \
                    STAMP_TK; \
                    if (spare) { spare = false; ++tk; tks = 2; } \
                } \
                if (!h2 && tk2s == 2) { tk2s = 0; h2 = true; p2 = tk2; tl2 = TAB(hm * 128 + (int)(p2 >> 4)); r2 = !ok; } \
            } while (0)
            const int e2 = e < 3 ? 4 + x : x - 1;
            int ck = 0; unsigned cso = 0, cdo = 0, csig = 0, cseg = 0, csegn = 1;
            u32x4 va[4], vb[4] = {};
            unsigned sa[4], sb[4], da[4], db[4] = {OOR, OOR, OOR, OOR};
            __amdgpu_buffer_rsrc_t rs, rd = rl;
#define CHOOSE() do { \
                ck = 0; csig = 0; \
                if (h2 && r2) { \
                    const unsigned o = tl2 * 131072u + (p2 & 15u) * 8192u; \
                    ck = 4; cso = PT_OFF + o; cdo = RT_OFF + o; csig = (unsigned)(L_TILE + tl2) * 128u + 64u + 3u; cseg = 128u + (p2 >> 4); csegn = 16u; h2 = false; \
                    PROG(1, p2); \
                } else if (nk && nrdy) { \
                    ck = nk; cso = nso; cdo = ndo; csig = nsig; cseg = nseg & 255u; csegn = nsegn; nk = 0; \
                    PROG((nseg >> 28) ? 2 : 0, (nseg >> 8) & 0xfffffu); \
                } \
                rs = RSRC(ck == 1 ? (char*)P.x : ck == 2 ? (char*)P.w : ck == 6 ? (char*)P.out[e] : (char*)P.ws[e], ck == 6 ? OUT_BYTES : ck >= 3 ? WS_END : IN_BYTES); \
                const bool sr = ck == 2 || ck == 6; \
                const unsigned sv = (ck ? cso : OOR) + (sr ? rlane : llane), ss = sr ? 32768u : ck == 1 ? 2048u : 1024u; \
                _Pragma("unroll") for (int k = 0; k < 4; ++k) { sa[k] = sv + k * ss; sb[k] = sv + (k + 4) * ss; } \
                asm volatile("" : "+v"(sa[0]), "+v"(sa[1]), "+v"(sa[2]), "+v"(sa[3]), "+v"(sb[0]), "+v"(sb[1]), "+v"(sb[2]), "+v"(sb[3])); \
            } while (0)
            REFILL();
            CHOOSE();
            for (;;) {
#pragma unroll
                for (int k = 0; k < 4; ++k) va[k] = __builtin_amdgcn_raw_buffer_load_b128(rs, sa[k], 0, 17  );
                __builtin_amdgcn_sched_barrier(0);
                if (!ok) {
#pragma unroll
                    for (int k = 0; k < 4; ++k) vb[k] = (u32x4){0x7fc07fc0u, 0x7fc07fc0u, 0x7fc07fc0u, 0x7fc07fc0u};
                }
#pragma unroll
                for (int k = 0; k < 4; ++k) __builtin_amdgcn_raw_buffer_store_b128(vb[k], rd, db[k], 0, 17  );
                __builtin_amdgcn_sched_barrier(0);
                if (ck == 0 && nk == 0 && !more && !h2 && tk2s == 0 && !q2 && psig == 0u && qsig == 0u) break;
                REFILL();
                if (ck) idle = 0;
                else {
                    if (idle == 0u) t0 = __builtin_amdgcn_s_memrealtime(); else __builtin_amdgcn_s_sleep(K_SLEEP);
                    if ((++idle & 63u) == 0u && ok && gave_up(t0, tmo, call)) { ok = false; nrdy = true; r2 = true; }
                }
                (void)ATOMV(RSRC(P.ws[qsig & 7u], CTL_BYTES), (qsig && UNI(qv) == qsegn * call - 1u) ? qsig & ~7u : OOR, qsegn);
                __builtin_amdgcn_sched_barrier(0);
                const bool cl1 = tks == 0 && more, cl2 = q2 && tk2s == 0, nw = nk && !nrdy, w2 = h2 && !r2;
                const unsigned tkv1 = ATOMV(rl, cl1 ? (pool ? (unsigned)(L_HEAD + 32) * 128u : (unsigned)head) : OOR, (pool && e == 4) ? 0x10000u : 1u);
                const unsigned tkv2 = ATOMV(rl, cl2 ? (unsigned)head + 128u : OOR, 1);
                const unsigned pv = __builtin_amdgcn_raw_buffer_load_b32(rl, (lm0 & (nw ? ndep : OOR)) | (lm1 & (w2 ? (unsigned)(L_TILE + tl2) * 128u + 32u : OOR)) |
                                                                             (lm2 & (gate ? (unsigned)(L_PANEL + (x - 1) * 32 + K_GATE) * 128u : OOR)) | lmx, 0, 17);
                rd = RSRC(ck >= 5 ? (char*)P.out[3] : ck >= 3 ? myws : (char*)P.ws[e], ck >= 5 ? OUT_BYTES : WS_END);
                unsigned qoff = l0 | (psig ? (unsigned)scnt + pseg * 4u : OOR);
                const unsigned dv0 = (ck ? cdo : OOR) + (ck >= 5 ? rlane : llane), ds = ck >= 5 ? 32768u : (ck & 1) ? 2048u : 1024u;
#pragma unroll
                for (int k = 0; k < 4; ++k) { da[k] = dv0 + k * ds; db[k] = dv0 + (k + 4) * ds; }
                asm volatile("" : "+v"(da[0]), "+v"(da[1]), "+v"(da[2]), "+v"(da[3]), "+v"(db[0]), "+v"(db[1]), "+v"(db[2]), "+v"(db[3]), "+v"(qoff));
                __builtin_amdgcn_sched_barrier(0);
                unsigned one = 1u;
                asm volatile("s_waitcnt vmcnt(0)" : "+v"(one) : "v"(pv) : "memory");
#pragma unroll
                for (int k = 0; k < 4; ++k) vb[k] = __builtin_amdgcn_raw_buffer_load_b128(rs, sb[k], 0, 17);
                __builtin_amdgcn_sched_barrier(0);
                if (!ok) {
#pragma unroll
                    for (int k = 0; k < 4; ++k) va[k] = (u32x4){0x7fc07fc0u, 0x7fc07fc0u, 0x7fc07fc0u, 0x7fc07fc0u};
                }
#pragma unroll
                for (int k = 0; k < 4; ++k) __builtin_amdgcn_raw_buffer_store_b128(va[k], rd, da[k], 0, 17);
                __builtin_amdgcn_sched_barrier(0);
                qv = (unsigned)__builtin_amdgcn_raw_ptr_buffer_atomic_add_i32((int)one, rl, qoff, 0, 16);
                __builtin_amdgcn_sched_barrier(0);
                qsig = psig; qsegn = psegn;
                psig = csig; pseg = cseg; psegn = csegn;
                const unsigned okm = (unsigned)__ballot((int)(pv - ((lm0 & nneed) | (lm1 & (8u * call)) | (lm2 & (64u * call)))) >= 0);
                r2 = r2 || (w2 && (okm & 2u));
                gate = gate && !(okm & 4u);
                nrdy = nrdy || (nw && !gate && (okm & 1u));
                if (cl1) { tk = UNI(tkv1); tks = 2; tkpool = pool; if (!pool && tk >= nfix) { pool = true; tks = 0; } }
                if (cl2) { tk2 = UNI(tkv2); tk2s = 2; if (tk2 >= (unsigned)Q2_ITEMS) { tk2s = 0; q2 = false; STAMP(2); } }
                CHOOSE();
                __builtin_amdgcn_sched_barrier(0);
                asm volatile("" :: "v"(vb[3].x) : "memory");
                __builtin_amdgcn_sched_barrier(0);
            }
        }
        STAMP(3);
        __syncthreads();
        wg_exit(ctl, call, wid, lane);
        return;
    }

    const unsigned call = (unsigned)__builtin_amdgcn_readfirstlane(ld_relaxed_agent(ctl + L_CALL * 32)) + 1u;
    stage_inputs(P, myws, call, xl, lane, slot >= 32 - K_STW ? (wid < K_STV ? 1 << 20 : 0) : (wid < K_ST1 ? 1 : 0));
    asm volatile("s_waitcnt vmcnt(0)" ::: "memory");
    const int njobs = (int)P.njob[xl];
    for (int job = slot - CE_WGS, round = 0; job < njobs; ++round) {
        const unsigned jw = (P.job[(xl * MAXJ + job) >> 1] >> ((job & 1) * 16)) & 0xffffu;
        if (jw >> 8) job = gemm_job<true>(P, lds, myws, call, (int)(jw & 255u), (int)(jw >> 8) & 3, xl, round);
        else job = gemm_job<false>(P, lds, myws, call, (int)jw, 0, xl, round);
    }
    wg_exit(ctl, call, wid, lane);
}

}

extern "C" void kernel_launch(int rank, int nranks, ncclComm_t comm,
                              hipStream_t stream,
                              void* const* d_in, const int* in_sizes, int n_in,
                              void* d_out, int out_size,
                              const int* in_dtypes, int out_dtype,
                              void* const* d_out_all,
                              void* const* d_ws_all, size_t ws_bytes) {
    if (nranks != NR || (unsigned)rank >= (unsigned)NR || n_in < 2 || ws_bytes < WS_END || (size_t)out_size != (size_t)GM * GN ||
        (size_t)in_sizes[0] != (size_t)GM * KS || (size_t)in_sizes[1] != (size_t)KS * GN) {
        fprintf(stderr, "kernel_launch: unexpected problem shape\n");
        return;
    }
    const int member[2][4] = {{0, 2, 3, 6}, {1, 4, 5, 7}};
    Params P = {};
    P.x = d_in[0]; P.w = d_in[1]; P.g = 0; P.a = 0;
    for (int g = 0; g < 2; ++g) for (int a = 0; a < 4; ++a) if (member[g][a] == rank) { P.g = g; P.a = a; }
    for (int e = 0; e < 8; ++e) {
        const int peer = e < 3 ? member[P.g][P.a ^ (e + 1)] : (e == 3 ? rank : member[1 - P.g][P.a ^ (e - 4)]);
        P.out[e] = d_out_all[peer]; P.ws[e] = d_ws_all[peer];
    }
    unsigned char* tb = (unsigned char*)P.tab;
    unsigned short* jb = (unsigned short*)P.job;
    int n = 0, np[2] = {0, 0}, nx[8] = {0, 0, 0, 0, 0, 0, 0, 0};
    for (int k = 0; k < 16; ++k)
        for (int r = 0; r < 2 * k + 1; ++r, ++n) {
            const int I = r < k ? k : r - k, J = r < k ? r : k;
            const unsigned char tl = (unsigned char)(I << 4 | J);
            const int par = (I + J) & 1;
            tb[par * 128 + np[par]++] = tl;
            if (n < K_SMALL) { const int xs = n & 7; for (int q = 0; q < 4; ++q) jb[xs * MAXJ + nx[xs]++] = (unsigned short)(tl | (4 + q) << 8); }
            else { const int x = (I & 3) << 1 | ((J >> 1) & 1); jb[x * MAXJ + nx[x]++] = tl; }
        }
    for (int x = 0; x < 8; ++x) P.njob[x] = (unsigned)nx[x];
    hipLaunchKernelGGL(gemm_ar_kernel, dim3(GRID), dim3(512), 0, stream, P);
    HIP_CHECK(hipGetLastError());
}
```
